```python
import numpy as np

kernel_name = "gpu_gemm_rs_m4096_k4096_n4096_f32_relu"


ABSMAX_RTOL = 0.001

COMPUTE_DTYPE = "float32"
EPILOGUE = "relu"
IN_QUANT = None
OUT_QUANT = None
IN_DTYPES = {"x": "float32", "w_mat": "float32"}
WORLD_SIZE = 8

def setup_inputs():

    x = np.random.randn(4096, 4096).astype(np.float32)
    w_mat = (np.random.randn(4096, 4096) / np.sqrt(4096)).astype(np.float32)
    return {"x": x, "w_mat": w_mat}

def _epilogue(y):
    return np.maximum(y, 0.0).astype(np.float32)

def reference(x, w_mat):
    y = (x @ w_mat).astype(np.float32)
    return _epilogue(y)
```

```cpp
#include <hip/hip_runtime.h>
#include <rccl/rccl.h>
#include <cstdio>

namespace {

constexpr int NR = 8;
constexpr int NCOL = 4096;
constexpr int KSH = 512;
constexpr int LATE = 8;

constexpr size_t OFF_EPOCH = 0;
constexpr size_t OFF_ENDCNT = 128;
constexpr size_t OFF_TMO = 256;
constexpr size_t OFF_OFLAG = 1024;
constexpr size_t OFF_FLAGS = 4096;
constexpr size_t OFF_STAGE = 1u << 20;
constexpr size_t PIECE = 16384;
constexpr size_t A_BYTES = 64 * 8 * PIECE;
constexpr size_t B_BYTES = 64 * 16 * PIECE;
constexpr size_t PAR_BYTES = A_BYTES + B_BYTES;
constexpr size_t WS_NEED = OFF_STAGE + 2 * PAR_BYTES;

constexpr int NPROD = 64, NCONS = 128, GRID = NPROD + NCONS + 1;
constexpr int THREADS = 576;
constexpr unsigned long long DEADLINE = 30000000ull;

typedef unsigned u32x4 __attribute__((ext_vector_type(4)));
typedef float f32x4 __attribute__((ext_vector_type(4)));
typedef float f32x16 __attribute__((ext_vector_type(16)));
typedef _Float16 h2 __attribute__((ext_vector_type(2)));
typedef _Float16 h8 __attribute__((ext_vector_type(8)));
#define LAS __attribute__((address_space(3)))

struct Args {
  char* ws[NR];
  float* out[NR];
  const float* x;
  const float* w;
  int rank;
  int pad;
};

__device__ __forceinline__ unsigned ld_sys(const void* p) {
  return __hip_atomic_load((const unsigned*)p, __ATOMIC_RELAXED, __HIP_MEMORY_SCOPE_SYSTEM);
}
__device__ __forceinline__ unsigned ld_agent(const void* p) {
  return __hip_atomic_load((const unsigned*)p, __ATOMIC_RELAXED, __HIP_MEMORY_SCOPE_AGENT);
}
__device__ __forceinline__ void st_sys(void* p, unsigned v) {
  __hip_atomic_store((unsigned*)p, v, __ATOMIC_RELAXED, __HIP_MEMORY_SCOPE_SYSTEM);
}
__device__ __forceinline__ void st_agent(void* p, unsigned v) {
  __hip_atomic_store((unsigned*)p, v, __ATOMIC_RELAXED, __HIP_MEMORY_SCOPE_AGENT);
}
__device__ __forceinline__ unsigned pk(float a, float b) {
  h2 h; h.x = (_Float16)a; h.y = (_Float16)b; return __builtin_bit_cast(unsigned, h);
}
#define BAR() do { asm volatile("" ::: "memory"); __builtin_amdgcn_s_barrier(); asm volatile("" ::: "memory"); } while (0)

struct Piece { int kc, tile; };
__device__ __forceinline__ Piece decode(int n) {
  Piece p;
  constexpr int nf = LATE * 24, n2 = nf + (8 - LATE) * 20;
  int i;
  if (n < nf) { p.kc = n / 24; i = n % 24; }
  else if (n < n2) { const int v = n - nf; p.kc = LATE + v / 20; i = v % 20; }
  else { const int v = n - n2; p.kc = LATE + v / 4; i = 20 + v % 4; }
  p.tile = i < 16 ? 8 + i : (i < 20 ? i - 12 : i - 20);
  return p;
}
struct Src { const char* base; unsigned ustep, loff, js; };
__device__ __forceinline__ Src src_of(const Args& a, int d, const Piece& p, int lane) {
  Src s;
  const unsigned i = lane & 31, h = lane >> 5;
  if (p.tile >= 8) {
    s.base = (const char*)a.w + ((size_t)(p.kc * 64) * NCOL + (d >> 2) * 2048 + (p.tile - 8) * 128) * 4;
    s.ustep = NCOL * 4; s.loff = h * (32 * NCOL * 4) + i * 16; s.js = 4 * NCOL * 4;
  } else {
    const int row0 = p.tile < 4 ? d * 512 + p.tile * 128 : (((d >> 2) ^ 1) * 4 + (p.tile - 4)) * 512 + (d & 3) * 128;
    s.base = (const char*)a.x + ((size_t)row0 * KSH + p.kc * 64) * 4;
    s.ustep = 32 * KSH * 4; s.loff = i * (KSH * 4) + h * 128; s.js = 16;
  }
  return s;
}
__device__ __forceinline__ void load8(f32x4 (&r)[8], const char* base, unsigned loff, unsigned js) {
  unsigned o[8];
#pragma unroll
  for (int j = 0; j < 8; ++j) o[j] = loff + j * js;
  asm volatile(
      "s_nop 4\n\t"
      "global_load_dwordx4 %0, %8, %16\n\t"
      "global_load_dwordx4 %1, %9, %16\n\t"
      "global_load_dwordx4 %2, %10, %16\n\t"
      "global_load_dwordx4 %3, %11, %16\n\t"
      "global_load_dwordx4 %4, %12, %16\n\t"
      "global_load_dwordx4 %5, %13, %16\n\t"
      "global_load_dwordx4 %6, %14, %16\n\t"
      "global_load_dwordx4 %7, %15, %16"
      : "=&v"(r[0]), "=&v"(r[1]), "=&v"(r[2]), "=&v"(r[3]), "=&v"(r[4]), "=&v"(r[5]), "=&v"(r[6]), "=&v"(r[7])
      : "v"(o[0]), "v"(o[1]), "v"(o[2]), "v"(o[3]), "v"(o[4]), "v"(o[5]), "v"(o[6]), "v"(o[7]), "s"(base)
      : "memory");
}
#define WAIT_R(N, r) asm volatile("s_waitcnt vmcnt(" #N ")" : "+v"(r[0]), "+v"(r[1]), "+v"(r[2]), "+v"(r[3]), \
    "+v"(r[4]), "+v"(r[5]), "+v"(r[6]), "+v"(r[7]) :: "memory")
__device__ __forceinline__ void cvt4(const f32x4 (&r)[8], u32x4 (&fr)[4]) {
#pragma unroll
  for (int c = 0; c < 4; ++c)
    fr[c] = (u32x4){pk(r[0][c], r[1][c]), pk(r[2][c], r[3][c]), pk(r[4][c], r[5][c]), pk(r[6][c], r[7][c])};
}
__device__ __forceinline__ void store4(char* dbase, unsigned voff, const u32x4 (&fr)[4]) {
  asm volatile(
      "s_nop 4\n\t"
      "global_store_dwordx4 %0, %1, %5 sc0 sc1\n\t"
      "global_store_dwordx4 %0, %2, %5 offset:1024 sc0 sc1\n\t"
      "global_store_dwordx4 %0, %3, %5 offset:2048 sc0 sc1\n\t"
      "global_store_dwordx4 %0, %4, %5 offset:3072 sc0 sc1\n\t"
      "s_nop 1"
      :: "v"(voff), "v"(fr[0]), "v"(fr[1]), "v"(fr[2]), "v"(fr[3]), "s"(dbase) : "memory");
}

__device__ __forceinline__ void producer(const Args& a, int d, int slot, int lane) {
  constexpr int NP = 192, NSLOT = NPROD / NR;
  char* dws = a.ws[d];
  unsigned call_v;
  asm volatile("s_nop 4\n\tglobal_load_dword %0, %1, %2 sc1" : "=&v"(call_v) : "v"(0u), "s"(a.ws[a.rank] + OFF_EPOCH) : "memory");
  const unsigned voff = lane * 16;
  f32x4 r0[8], r1[8];
  u32x4 fr[4];
  Piece p = decode(slot);
  Src s = src_of(a, d, p, lane);
  load8(r0, s.base, s.loff, s.js);
  load8(r1, s.base + s.ustep, s.loff, s.js);
  WAIT_R(0, r0); WAIT_R(0, r1);
  asm volatile("s_nop 0" : "+v"(call_v));
  const unsigned call = __builtin_amdgcn_readfirstlane(call_v) + 1u;
  char* stage = dws + OFF_STAGE + (size_t)(call & 1u) * PAR_BYTES;
  char* prevflag = nullptr;
  for (int n = slot; n < NP; n += NSLOT) {
    const int nn = n + NSLOT < NP ? n + NSLOT : n;
    const Piece pn = decode(nn);
    const Src sn = src_of(a, d, pn, lane);
    const int cs = a.rank * 8 + p.kc;
    char* dst = p.tile >= 8 ? stage + A_BYTES + ((size_t)cs * 16 + (p.tile - 8)) * PIECE : stage + ((size_t)cs * 8 + p.tile) * PIECE;
    WAIT_R(16, r0); cvt4(r0, fr); load8(r0, s.base + 2 * s.ustep, s.loff, s.js); store4(dst, voff, fr);
    WAIT_R(16, r1); cvt4(r1, fr); load8(r1, s.base + 3 * s.ustep, s.loff, s.js); store4(dst + 4096, voff, fr);
    WAIT_R(16, r0);
    if (prevflag != nullptr && lane == 0) st_sys(prevflag, call);
    cvt4(r0, fr); load8(r0, sn.base, sn.loff, sn.js); store4(dst + 8192, voff, fr);
    WAIT_R(16, r1); cvt4(r1, fr); load8(r1, sn.base + sn.ustep, sn.loff, sn.js); store4(dst + 12288, voff, fr);
    prevflag = dws + OFF_FLAGS + (size_t)(p.tile * 64 + cs) * 4;
    p = pn; s = sn;
  }
  asm volatile("s_waitcnt vmcnt(0)" : "+v"(r0[0]), "+v"(r1[0]) :: "memory");
  if (lane == 0) st_sys(prevflag, call);
}

__global__ void __launch_bounds__(THREADS) gemm_rs_kernel(Args a) {
  __shared__ __attribute__((aligned(16))) char lds[4 * 32768 + 64];
  const int tid = threadIdx.x, lane = tid & 63;
  const int wv = __builtin_amdgcn_readfirstlane(tid >> 6);
  const int b = blockIdx.x, xcd = b & 7, jb = b >> 3;
  char* myws = a.ws[a.rank];
  unsigned call = 0;

  if (jb < 8) {
    if (wv == 0) {
      producer(a, xcd, jb, lane);
      call = ld_agent(myws + OFF_EPOCH) + 1u;
    }
  } else if (jb < 24) {
    const int jc = jb - 8;
    const int rt = (xcd & 3) + (jc >> 3) * 4;
    const int ct = (xcd >> 2) * 8 + (jc & 7);
    call = __builtin_amdgcn_readfirstlane(ld_agent(myws + OFF_EPOCH)) + 1u;
    LAS unsigned* lds_bad = (LAS unsigned*)((LAS char*)lds + 4 * 32768);
    if (wv == 8) {
      const char* fa = myws + OFF_FLAGS + (size_t)(rt * 64 + lane) * 4;
      const char* fb = myws + OFF_FLAGS + (size_t)((8 + ct) * 64 + lane) * 4;
      unsigned long long ready = 0;
      for (int c = 0; c < 64; ++c) {
        const int cs = ((a.rank + (c & 7)) & 7) * 8 + (c >> 3);
        if (!((ready >> cs) & 1ull)) {
          const unsigned long long t0 = __builtin_amdgcn_s_memrealtime();
          for (;;) {
            const unsigned va = ld_sys(fa), vb = ld_sys(fb);
            ready = __ballot(va >= call && vb >= call);
            if ((ready >> cs) & 1ull) break;
            if (__builtin_amdgcn_readfirstlane(ld_agent(myws + OFF_TMO)) == call ||
                __builtin_amdgcn_s_memrealtime() - t0 > DEADLINE) {
              if (lane == 0) st_agent(myws + OFF_TMO, call);
              ready = ~0ull;
              break;
            }
            __builtin_amdgcn_s_sleep(1);
          }
        }
        if (c == 63) {
          const unsigned t = __builtin_amdgcn_readfirstlane(ld_agent(myws + OFF_TMO));
          if (lane == 0) *lds_bad = (t == call) ? 1u : 0u;
          asm volatile("s_waitcnt lgkmcnt(0)" ::: "memory");
        }
        BAR();
      }
      BAR(); BAR(); BAR(); BAR();
    } else if (wv >= 4) {
      const int lw = wv - 4;
      const char* stage = myws + OFF_STAGE + (size_t)(call & 1u) * PAR_BYTES;
      const char* abase = stage + (size_t)rt * PIECE + lw * 4096 + lane * 16;
      const char* bbase = stage + A_BYTES + (size_t)ct * PIECE + lw * 4096 + lane * 16;
      BAR();
      for (int c = 0; c < 67; ++c) {
        if (c < 64) {
          const int cs = ((a.rank + (c & 7)) & 7) * 8 + (c >> 3);
          const char* ga = abase + (size_t)cs * (8 * PIECE);
          const char* gb = bbase + (size_t)cs * (16 * PIECE);
          LAS char* la = (LAS char*)lds + (c & 3) * 32768 + lw * 4096;
#pragma unroll
          for (int f = 0; f < 4; ++f)
            __builtin_amdgcn_global_load_lds((const __attribute__((address_space(1))) void*)(ga + f * 1024), (LAS void*)(la + f * 1024), 16, 0, 17);
#pragma unroll
          for (int f = 0; f < 4; ++f)
            __builtin_amdgcn_global_load_lds((const __attribute__((address_space(1))) void*)(gb + f * 1024), (LAS void*)(la + 16384 + f * 1024), 16, 0, 17);
          asm volatile("s_waitcnt vmcnt(16)" ::: "memory");
        } else if (c == 64) {
          asm volatile("s_waitcnt vmcnt(8)" ::: "memory");
        } else {
          asm volatile("s_waitcnt vmcnt(0)" ::: "memory");
        }
        BAR();
      }
    } else {
      f32x16 acc[4];
#pragma unroll
      for (int c = 0; c < 4; ++c)
#pragma unroll
        for (int e = 0; e < 16; ++e) acc[c][e] = 0.f;
      BAR(); BAR(); BAR(); BAR();
      for (int t = 0; t < 64; ++t) {
        const LAS char* sa = (const LAS char*)lds + (t & 3) * 32768 + wv * 4096 + lane * 16;
        const LAS char* sb = (const LAS char*)lds + (t & 3) * 32768 + 16384 + lane * 16;
#pragma unroll
        for (int s = 0; s < 4; ++s) {
          const h8 af = *(const LAS h8*)(sa + s * 1024);
#pragma unroll
          for (int c = 0; c < 4; ++c) {
            const h8 bf = *(const LAS h8*)(sb + s * 4096 + c * 1024);
            acc[c] = __builtin_amdgcn_mfma_f32_32x32x16_f16(af, bf, acc[c], 0, 0, 0);
          }
        }
        asm volatile("s_waitcnt lgkmcnt(0)" ::: "memory");
        BAR();
      }
      const bool bad = *lds_bad != 0u;
      const int g = a.rank >> 2;
      const int col = g * 2048 + ct * 128 + 4 * (lane & 31);
      const float qnan = __builtin_nanf("");
      if (rt < 4) {
        float* o = a.out[a.rank] + (size_t)(rt * 128 + wv * 32 + 4 * (lane >> 5)) * NCOL + col;
#pragma unroll
        for (int e = 0; e < 16; ++e) {
          f32x4 v;
#pragma unroll
          for (int c = 0; c < 4; ++c) { const float y = acc[c][e]; v[c] = bad ? qnan : (y < 0.f ? 0.f : y); }
          *(f32x4*)(o + (size_t)((e & 3) + 8 * (e >> 2)) * NCOL) = v;
        }
      } else {
        const int q = ((g ^ 1) << 2) + (rt - 4);
        float* o = a.out[q & 7] + (size_t)((a.rank & 3) * 128 + wv * 32 + 4 * (lane >> 5)) * NCOL + col;
#pragma unroll
        for (int e = 0; e < 16; ++e) {
          f32x4 v;
#pragma unroll
          for (int c = 0; c < 4; ++c) { const float y = acc[c][e]; v[c] = bad ? qnan : (y < 0.f ? 0.f : y); }
          asm volatile("global_store_dwordx4 %0, %1, off sc0 sc1\n\ts_nop 1"
                       :: "v"(o + (size_t)((e & 3) + 8 * (e >> 2)) * NCOL), "v"(v) : "memory");
        }
        asm volatile("s_waitcnt vmcnt(0)" ::: "memory");
      }
    }
    BAR();
    if (rt >= 4 && tid == 0) {
      const int q = (((a.rank >> 2) ^ 1) << 2) + (rt - 4);
      st_sys(a.ws[q & 7] + OFF_OFLAG + (size_t)((a.rank & 3) * 16 + ct) * 4, call);
    }
  } else {
    call = __builtin_amdgcn_readfirstlane(ld_agent(myws + OFF_EPOCH)) + 1u;
    if (wv == 0) {
      const char* f = myws + OFF_OFLAG + lane * 4;
      const unsigned long long t0 = __builtin_amdgcn_s_memrealtime();
      bool bad = false;
      for (;;) {
        if (__all(ld_sys(f) >= call)) break;
        if (__builtin_amdgcn_readfirstlane(ld_agent(myws + OFF_TMO)) == call ||
            __builtin_amdgcn_s_memrealtime() - t0 > DEADLINE) {
          if (lane == 0) st_agent(myws + OFF_TMO, call);
          bad = true;
          break;
        }
        __builtin_amdgcn_s_sleep(1);
      }
      if (bad || __builtin_amdgcn_readfirstlane(ld_agent(myws + OFF_TMO)) == call) {
        if (lane == 0) {
          const unsigned long long t1 = __builtin_amdgcn_s_memrealtime();
          while (__builtin_amdgcn_s_memrealtime() - t1 < 100000ull) __builtin_amdgcn_s_sleep(8);
          __hip_atomic_store((unsigned*)(a.out[a.rank] + (a.rank >> 2) * 2048), 0x7fc00000u, __ATOMIC_RELAXED, __HIP_MEMORY_SCOPE_SYSTEM);
        }
      }
    }
  }
  if (tid == 0) {
    const unsigned t = __hip_atomic_fetch_add((unsigned*)(myws + OFF_ENDCNT), 1u, __ATOMIC_RELAXED, __HIP_MEMORY_SCOPE_AGENT);
    if (t % GRID == GRID - 1) st_agent(myws + OFF_EPOCH, call);
  }
}

}

extern "C" void kernel_launch(int rank, int nranks, ncclComm_t comm,
                              hipStream_t stream,
                              void* const* d_in, const int* in_sizes, int n_in,
                              void* d_out, int out_size,
                              const int* in_dtypes, int out_dtype,
                              void* const* d_out_all,
                              void* const* d_ws_all, size_t ws_bytes) {
  (void)comm; (void)in_dtypes; (void)out_dtype; (void)d_out;
  if (nranks != NR || (unsigned)rank >= (unsigned)NR || n_in < 2 || ws_bytes < WS_NEED ||
      in_sizes[0] != 4096 * KSH || in_sizes[1] != KSH * NCOL || out_size != 512 * NCOL) {
    fprintf(stderr, "gemm_rs: unexpected problem shape\n");
    return;
  }
  Args a;
  for (int p = 0; p < NR; ++p) { a.ws[p] = (char*)d_ws_all[p]; a.out[p] = (float*)d_out_all[p]; }
  a.x = (const float*)d_in[0];
  a.w = (const float*)d_in[1];
  a.rank = rank;
  a.pad = 0;
  gemm_rs_kernel<<<GRID, THREADS, 0, stream>>>(a);
  hipError_t e = hipGetLastError();
  if (e != hipSuccess) fprintf(stderr, "gemm_rs: launch failed: %s\n", hipGetErrorString(e));
}
```

```cpp
#include <hip/hip_runtime.h>
#include <rccl/rccl.h>
#include <cstdio>

namespace {

constexpr int NR = 8;
constexpr int NCOL = 4096;
constexpr int KSH = 512;
#define LATE_V 6
constexpr int LATE = LATE_V;

constexpr size_t OFF_EPOCH = 0;
constexpr size_t OFF_ENDCNT = 128;
constexpr size_t OFF_TMO = 256;
constexpr size_t OFF_OFLAG = 1024;
constexpr size_t OFF_FLAGS = 4096;
constexpr size_t OFF_STAGE = 1u << 20;
constexpr size_t PIECE = 16384;
constexpr size_t A_BYTES = 64 * 8 * PIECE;
constexpr size_t B_BYTES = 64 * 16 * PIECE;
constexpr size_t PAR_BYTES = A_BYTES + B_BYTES;
constexpr size_t CSLOT = 17408;
constexpr size_t OFF_CSLOT = OFF_STAGE + 2 * PAR_BYTES;
constexpr size_t WS_NEED = OFF_CSLOT + 256 * CSLOT;

constexpr int NPROD = 64, NCONS = 128, GRID = NPROD + NCONS;
#define NLW_V 4
constexpr int NLW = NLW_V;
constexpr int THREADS = (4 + NLW + 1) * 64;
constexpr unsigned long long DEADLINE = 30000000ull;

typedef unsigned u32x4 __attribute__((ext_vector_type(4)));
typedef unsigned u32x2 __attribute__((ext_vector_type(2)));
typedef float f32x4 __attribute__((ext_vector_type(4)));
typedef float f32x16 __attribute__((ext_vector_type(16)));
typedef _Float16 h2 __attribute__((ext_vector_type(2)));
typedef _Float16 h8 __attribute__((ext_vector_type(8)));
#define LAS __attribute__((address_space(3)))
#define GAS __attribute__((address_space(1)))

struct Args {
  char* ws[NR];
  float* out[NR];
  const float* x;
  const float* w;
  int rank;
  int pad;
};

__device__ __forceinline__ unsigned ld_sys(const void* p) {
  return __hip_atomic_load((const unsigned*)p, __ATOMIC_RELAXED, __HIP_MEMORY_SCOPE_SYSTEM);
}
__device__ __forceinline__ unsigned ld_agent(const void* p) {
  return __hip_atomic_load((const unsigned*)p, __ATOMIC_RELAXED, __HIP_MEMORY_SCOPE_AGENT);
}
__device__ __forceinline__ void st_sys(void* p, unsigned v) {
  __hip_atomic_store((unsigned*)p, v, __ATOMIC_RELAXED, __HIP_MEMORY_SCOPE_SYSTEM);
}
__device__ __forceinline__ void st_agent(void* p, unsigned v) {
  __hip_atomic_store((unsigned*)p, v, __ATOMIC_RELAXED, __HIP_MEMORY_SCOPE_AGENT);
}
__device__ __forceinline__ unsigned pk(float a, float b) {
  h2 h; h.x = (_Float16)a; h.y = (_Float16)b; return __builtin_bit_cast(unsigned, h);
}
#define STAMP(i) do {} while (0)
#define BAR() do { asm volatile("" ::: "memory"); __builtin_amdgcn_s_barrier(); asm volatile("" ::: "memory"); } while (0)

struct Piece { int kc, tile; };
__device__ __forceinline__ Piece decode(int n) {
  Piece p;
  constexpr int nf = LATE * 24, n2 = nf + (8 - LATE) * 20;
  int i;
  if (n < nf) { p.kc = n / 24; i = n % 24; }
  else if (n < n2) { const int v = n - nf; p.kc = LATE + v / 20; i = v % 20; }
  else { const int v = n - n2; p.kc = LATE + v / 4; i = 20 + v % 4; }
  p.tile = i < 16 ? 8 + i : (i < 20 ? i - 12 : i - 20);
  return p;
}
struct Src { const char* base; unsigned ustep, loff, js; };
__device__ __forceinline__ Src src_of(const Args& a, int d, const Piece& p, int lane) {
  Src s;
  const unsigned i = lane & 31, h = lane >> 5;
  if (p.tile >= 8) {
    s.base = (const char*)a.w + ((size_t)(p.kc * 64) * NCOL + (d >> 2) * 2048 + (p.tile - 8) * 128) * 4;
    s.ustep = NCOL * 4; s.loff = h * (32 * NCOL * 4) + i * 16; s.js = 4 * NCOL * 4;
  } else {
    const int row0 = p.tile < 4 ? d * 512 + p.tile * 128 : (((d >> 2) ^ 1) * 4 + (p.tile - 4)) * 512 + (d & 3) * 128;
    s.base = (const char*)a.x + ((size_t)row0 * KSH + p.kc * 64) * 4;
    s.ustep = 32 * KSH * 4; s.loff = i * (KSH * 4) + h * 128; s.js = 16;
  }
  return s;
}
__device__ __forceinline__ void load8(f32x4 (&r)[8], const char* base, unsigned loff, unsigned js) {
  unsigned o[8];
#pragma unroll
  for (int j = 0; j < 8; ++j) o[j] = loff + j * js;
  asm volatile(
      "s_nop 4\n\t"
      "global_load_dwordx4 %0, %8, %16\n\t"
      "global_load_dwordx4 %1, %9, %16\n\t"
      "global_load_dwordx4 %2, %10, %16\n\t"
      "global_load_dwordx4 %3, %11, %16\n\t"
      "global_load_dwordx4 %4, %12, %16\n\t"
      "global_load_dwordx4 %5, %13, %16\n\t"
      "global_load_dwordx4 %6, %14, %16\n\t"
      "global_load_dwordx4 %7, %15, %16"
      : "=&v"(r[0]), "=&v"(r[1]), "=&v"(r[2]), "=&v"(r[3]), "=&v"(r[4]), "=&v"(r[5]), "=&v"(r[6]), "=&v"(r[7])
      : "v"(o[0]), "v"(o[1]), "v"(o[2]), "v"(o[3]), "v"(o[4]), "v"(o[5]), "v"(o[6]), "v"(o[7]), "s"(base)
      : "memory");
}
#define WAIT_R(N, r) asm volatile("s_waitcnt vmcnt(" #N ")" : "+v"(r[0]), "+v"(r[1]), "+v"(r[2]), "+v"(r[3]), \
    "+v"(r[4]), "+v"(r[5]), "+v"(r[6]), "+v"(r[7]) :: "memory")
__device__ __forceinline__ void cvt4(const f32x4 (&r)[8], u32x4 (&fr)[4]) {
#pragma unroll
  for (int c = 0; c < 4; ++c)
    fr[c] = (u32x4){pk(r[0][c], r[1][c]), pk(r[2][c], r[3][c]), pk(r[4][c], r[5][c]), pk(r[6][c], r[7][c])};
}
__device__ __forceinline__ void store4(char* dbase, unsigned voff, const u32x4 (&fr)[4]) {
  asm volatile(
      "s_nop 4\n\t"
      "global_store_dwordx4 %0, %1, %5 sc0 sc1\n\t"
      "global_store_dwordx4 %0, %2, %5 offset:1024 sc0 sc1\n\t"
      "global_store_dwordx4 %0, %3, %5 offset:2048 sc0 sc1\n\t"
      "global_store_dwordx4 %0, %4, %5 offset:3072 sc0 sc1\n\t"
      "s_nop 1"
      :: "v"(voff), "v"(fr[0]), "v"(fr[1]), "v"(fr[2]), "v"(fr[3]), "s"(dbase) : "memory");
}

__device__ __forceinline__ void producer(const Args& a, int d, int slot, int lane) {
  constexpr int NP = 192, NSLOT = NPROD / NR;
  char* dws = a.ws[d];
  unsigned call_v;
  asm volatile("s_nop 4\n\tglobal_load_dword %0, %1, %2 sc1" : "=&v"(call_v) : "v"(0u), "s"(a.ws[a.rank] + OFF_EPOCH) : "memory");
  const unsigned voff = lane * 16;
  f32x4 r0[8], r1[8];
  u32x4 fr[4];
  STAMP(0);
  Piece p = decode(slot);
  Src s = src_of(a, d, p, lane);
  load8(r0, s.base, s.loff, s.js);
  load8(r1, s.base + s.ustep, s.loff, s.js);
  WAIT_R(0, r0); WAIT_R(0, r1);
  asm volatile("s_nop 0" : "+v"(call_v));
  const unsigned call = __builtin_amdgcn_readfirstlane(call_v) + 1u;
  char* stage = dws + OFF_STAGE + (size_t)(call & 1u) * PAR_BYTES;
  char* prevflag = nullptr;
  STAMP(1);
  for (int n = slot; n < NP; n += NSLOT) {
    const int nn = n + NSLOT < NP ? n + NSLOT : n;
    const Piece pn = decode(nn);
    const Src sn = src_of(a, d, pn, lane);
    const int cs = a.rank * 8 + p.kc;
    char* dst = p.tile >= 8 ? stage + A_BYTES + ((size_t)cs * 16 + (p.tile - 8)) * PIECE : stage + ((size_t)cs * 8 + p.tile) * PIECE;
    WAIT_R(16, r0); cvt4(r0, fr); load8(r0, s.base + 2 * s.ustep, s.loff, s.js); store4(dst, voff, fr);
    WAIT_R(16, r1); cvt4(r1, fr); load8(r1, s.base + 3 * s.ustep, s.loff, s.js); store4(dst + 4096, voff, fr);
    WAIT_R(16, r0);
    if (prevflag != nullptr && lane == 0) st_sys(prevflag, call);
    cvt4(r0, fr); load8(r0, sn.base, sn.loff, sn.js); store4(dst + 8192, voff, fr);
    WAIT_R(16, r1); cvt4(r1, fr); load8(r1, sn.base + sn.ustep, sn.loff, sn.js); store4(dst + 12288, voff, fr);
    prevflag = dws + OFF_FLAGS + (size_t)(p.tile * 64 + cs) * 4;
    p = pn; s = sn;
  }
  asm volatile("s_waitcnt vmcnt(0)" : "+v"(r0[0]), "+v"(r1[0]) :: "memory");
  if (lane == 0) st_sys(prevflag, call);
  STAMP(2);
}

__device__ __forceinline__ void expander(const Args& a, unsigned call, int ti, int part, int half, int lane,
                                         LAS char* buf, LAS unsigned* sync) {
  char* myws = a.ws[a.rank];
  const int slot = ti * 4 + part;
  const char* flag = myws + OFF_OFLAG + (size_t)slot * 4;
  bool bad = false;
  {
    const unsigned long long t0 = __builtin_amdgcn_s_memrealtime();
    while (__builtin_amdgcn_readfirstlane(ld_sys(flag)) < call) {
      if (__builtin_amdgcn_readfirstlane(ld_agent(myws + OFF_TMO)) == call ||
          __builtin_amdgcn_s_memrealtime() - t0 > DEADLINE) {
        if (lane == 0) st_agent(myws + OFF_TMO, call);
        bad = true;
        break;
      }
      __builtin_amdgcn_s_sleep(1);
    }
  }
  if (half == 0) STAMP(3 + part * 2);
  const char* src = myws + OFF_CSLOT + (size_t)slot * CSLOT + lane * 16;
  if (!bad) {
#pragma unroll
    for (int i = 0; i < 5; ++i)
      __builtin_amdgcn_global_load_lds((const GAS void*)(src + (half * 5 + i) * 1024), (LAS void*)(buf + (half * 5 + i) * 1024), 16, 0, 17);
    if (half == 0) __builtin_amdgcn_global_load_lds((const GAS void*)(src + 16384), (LAS void*)(buf + 16384), 16, 0, 17);
  }
  const unsigned tm = __builtin_amdgcn_readfirstlane(ld_agent(myws + OFF_TMO));
  asm volatile("s_waitcnt vmcnt(0)" ::: "memory");
  bad = bad || tm == call;
  if (lane == 0) __hip_atomic_store(sync + half, bad ? 2u : 1u, __ATOMIC_RELEASE, __HIP_MEMORY_SCOPE_WORKGROUP);
  {
    unsigned o;
    const unsigned long long t0 = __builtin_amdgcn_s_memrealtime();
    while ((o = __builtin_amdgcn_readfirstlane(__hip_atomic_load(sync + (half ^ 1), __ATOMIC_ACQUIRE, __HIP_MEMORY_SCOPE_WORKGROUP))) == 0u) {
      if (__builtin_amdgcn_s_memrealtime() - t0 > 2 * DEADLINE) { o = 2u; break; }
    }
    bad = bad || o == 2u;
  }
  float* o = a.out[a.rank] + (size_t)((ti >> 4) * 128 + part * 32 + half * 16 + 4 * (lane >> 5)) * NCOL +
             ((a.rank >> 2) ^ 1) * 2048 + (ti & 15) * 128 + 4 * (lane & 31);
  if (bad) {
    const float qnan = __builtin_nanf("");
#pragma unroll
    for (int e = 0; e < 8; ++e) *(f32x4*)(o + (size_t)((e & 3) + 8 * (e >> 2)) * NCOL) = (f32x4){qnan, qnan, qnan, qnan};
    return;
  }
  const u32x2 m = *(const LAS u32x2*)(buf + 16384 + lane * 8);
  unsigned s0 = (unsigned)__builtin_popcount(m[1]), s1 = (unsigned)__builtin_popcount(m[0]);
  unsigned ssum = s0 | (s1 << 16);
#pragma unroll
  for (int d = 1; d < 64; d <<= 1) ssum += __shfl_xor(ssum, d, 64);
  ssum = __builtin_amdgcn_readfirstlane(ssum);
  const unsigned n0 = ssum & 0xffffu, total = n0 + (ssum >> 16);
  if (total > 2560u) {
#pragma unroll
    for (int i = 10; i < 16; ++i)
      __builtin_amdgcn_global_load_lds((const GAS void*)(src + i * 1024), (LAS void*)(buf + i * 1024), 16, 0, 17);
    asm volatile("s_waitcnt vmcnt(0)" ::: "memory");
  }
  unsigned base = half ? n0 : 0u;
  unsigned bits = half ? m[0] : m[1];
#pragma unroll
  for (int e = 0; e < 8; ++e) {
    f32x4 v;
#pragma unroll
    for (int c = 0; c < 4; ++c) {
      const bool nz = (int)bits < 0;
      bits <<= 1;
      const unsigned long long mm = __ballot(nz);
      unsigned pos = __builtin_amdgcn_mbcnt_hi((unsigned)(mm >> 32), __builtin_amdgcn_mbcnt_lo((unsigned)mm, base)) & 4095u;
      asm volatile("" : "+v"(pos));
      const float x = *(const LAS float*)(buf + pos * 4u);
      v[c] = nz ? x : 0.f;
      base += (unsigned)__builtin_popcountll(mm);
    }
    *(f32x4*)(o + (size_t)((e & 3) + 8 * (e >> 2)) * NCOL) = v;
  }
  if (half == 0) STAMP(4 + part * 2);
}

__global__ void __launch_bounds__(THREADS) gemm_rs_kernel(Args a) {
  __shared__ __attribute__((aligned(16))) char lds[4 * 32768 + 512];
  const int tid = threadIdx.x, lane = tid & 63;
  const int wv = __builtin_amdgcn_readfirstlane(tid >> 6);
  const int b = blockIdx.x, xcd = b & 7, jb = b >> 3;
  char* myws = a.ws[a.rank];
  unsigned call = 0;

  if (jb < 8) {
    call = __builtin_amdgcn_readfirstlane(ld_agent(myws + OFF_EPOCH)) + 1u;
    LAS unsigned* sync = (LAS unsigned*)((LAS char*)lds + 4 * CSLOT);
    if (wv == 0) {
      producer(a, xcd, jb, lane);
      BAR();
    } else {
      if (wv == 1 && lane < 8) sync[lane] = 0u;
      asm volatile("s_waitcnt lgkmcnt(0)" ::: "memory");
      BAR();
      if (wv <= 8) expander(a, call, jb * 8 + xcd, (wv - 1) >> 1, (wv - 1) & 1, lane,
                            (LAS char*)lds + ((wv - 1) >> 1) * CSLOT, sync + ((wv - 1) >> 1) * 2);
    }
  } else {
    const int jc = jb - 8;
    const int rt = (xcd & 3) + (jc >> 3) * 4;
    const int ct = (xcd >> 2) * 8 + (jc & 7);
    call = __builtin_amdgcn_readfirstlane(ld_agent(myws + OFF_EPOCH)) + 1u;
    LAS unsigned* lds_bad = (LAS unsigned*)((LAS char*)lds + 4 * 32768);
    LAS unsigned* lds_sched = (LAS unsigned*)((LAS char*)lds + 4 * 32768 + 64);
    if (wv == 4 + NLW) {
      const char* fa = myws + OFF_FLAGS + (size_t)(rt * 64 + lane) * 4;
      const char* fb = myws + OFF_FLAGS + (size_t)((8 + ct) * 64 + lane) * 4;
      const char* stage = myws + OFF_STAGE + (size_t)(call & 1u) * PAR_BYTES;
      const char* pa = stage + (size_t)rt * PIECE + lane * 128;
      const char* pb = stage + A_BYTES + (size_t)ct * PIECE + lane * 128;
      LAS char* sink = (LAS char*)lds + 4 * 32768 + 256;
      unsigned long long ready = 0, done = 0, seen = 0;
      STAMP(0);
      for (int c = 0; c < 64; ++c) {
        if ((ready & ~done) == 0ull) {
          const unsigned long long t0 = __builtin_amdgcn_s_memrealtime();
          for (;;) {
            const unsigned va = ld_sys(fa), vb = ld_sys(fb);
            ready = __ballot(va >= call && vb >= call);
            if ((ready & ~done) != 0ull) break;
            if (__builtin_amdgcn_readfirstlane(ld_agent(myws + OFF_TMO)) == call ||
                __builtin_amdgcn_s_memrealtime() - t0 > DEADLINE) {
              if (lane == 0) st_agent(myws + OFF_TMO, call);
              ready = ~0ull; seen = ~0ull;
              break;
            }
            __builtin_amdgcn_s_sleep(1);
          }
        }
        const int cs = __builtin_ctzll(ready & ~done);
        done |= 1ull << cs;
        seen |= 1ull << cs;
        if (lane == 0) lds_sched[c & 7] = (unsigned)cs;
        if (c == 0) STAMP(1);
        if (c == 8) STAMP(2);
        if (c == 32) STAMP(3);
        if (c == 56) STAMP(4);
        if (c == 63) STAMP(5);
        if (c == 63) {
          const unsigned t = __builtin_amdgcn_readfirstlane(ld_agent(myws + OFF_TMO));
          if (lane == 0) *lds_bad = (t == call) ? 1u : 0u;
        }
        asm volatile("s_waitcnt lgkmcnt(0)" ::: "memory");
        BAR();
        unsigned long long fresh = ready & ~seen;
        seen |= fresh;
        while (fresh) {
          const int k = __builtin_ctzll(fresh);
          fresh &= fresh - 1;
          const char* ga = pa + (size_t)k * (8 * PIECE);
          const char* gb = pb + (size_t)k * (16 * PIECE);
          __builtin_amdgcn_global_load_lds((const GAS void*)ga, (LAS void*)sink, 4, 0, 16);
          __builtin_amdgcn_global_load_lds((const GAS void*)(ga + 8192), (LAS void*)sink, 4, 0, 16);
          __builtin_amdgcn_global_load_lds((const GAS void*)gb, (LAS void*)sink, 4, 0, 16);
          __builtin_amdgcn_global_load_lds((const GAS void*)(gb + 8192), (LAS void*)sink, 4, 0, 16);
        }
      }
      BAR(); BAR(); BAR(); BAR();
      asm volatile("s_waitcnt vmcnt(0)" ::: "memory");
    } else if (wv >= 4) {
      constexpr int PER = 32 / NLW;
      const int lw = wv - 4;
      const char* stage = myws + OFF_STAGE + (size_t)(call & 1u) * PAR_BYTES;
      const bool isb = lw >= NLW / 2;
      const int part = (lw & (NLW / 2 - 1)) * (PER * 1024);
      const char* gbase = (isb ? stage + A_BYTES + (size_t)ct * PIECE : stage + (size_t)rt * PIECE) + part + lane * 16;
      const size_t cstride = isb ? 16 * PIECE : 8 * PIECE;
      const int lbase = (isb ? 16384 : 0) + part;
      BAR();
      for (int c = 0; c < 67; ++c) {
        if (c < 64) {
          const int cs = __builtin_amdgcn_readfirstlane(*(volatile LAS unsigned*)(lds_sched + (c & 7))) & 63;
          const char* ga = gbase + (size_t)cs * cstride;
          LAS char* la = (LAS char*)lds + (c & 3) * 32768 + lbase;
#pragma unroll
          for (int f = 0; f < PER; ++f)
            __builtin_amdgcn_global_load_lds((const GAS void*)(ga + f * 1024), (LAS void*)(la + f * 1024), 16, 0, 17);
          if constexpr (PER == 8) asm volatile("s_waitcnt vmcnt(16)" ::: "memory");
          else asm volatile("s_waitcnt vmcnt(8)" ::: "memory");
        } else if (c == 64) {
          if constexpr (PER == 8) asm volatile("s_waitcnt vmcnt(8)" ::: "memory");
          else asm volatile("s_waitcnt vmcnt(4)" ::: "memory");
        } else {
          asm volatile("s_waitcnt vmcnt(0)" ::: "memory");
        }
        BAR();
      }
    } else {
      f32x16 acc[4];
#pragma unroll
      for (int c = 0; c < 4; ++c)
#pragma unroll
        for (int e = 0; e < 16; ++e) acc[c][e] = 0.f;
      BAR(); BAR(); BAR(); BAR();
      for (int t = 0; t < 64; ++t) {
        const LAS char* sa = (const LAS char*)lds + (t & 3) * 32768 + wv * 4096 + lane * 16;
        const LAS char* sb = (const LAS char*)lds + (t & 3) * 32768 + 16384 + lane * 16;
        h8 af[2], bf[2][4];
        af[0] = *(const LAS h8*)(sa);
#pragma unroll
        for (int c = 0; c < 4; ++c) bf[0][c] = *(const LAS h8*)(sb + c * 1024);
#pragma unroll
        for (int s = 0; s < 4; ++s) {
          if (s < 3) {
            af[(s + 1) & 1] = *(const LAS h8*)(sa + (s + 1) * 1024);
#pragma unroll
            for (int c = 0; c < 4; ++c) bf[(s + 1) & 1][c] = *(const LAS h8*)(sb + (s + 1) * 4096 + c * 1024);
          }
          __builtin_amdgcn_sched_barrier(0);
#pragma unroll
          for (int c = 0; c < 4; ++c) acc[c] = __builtin_amdgcn_mfma_f32_32x32x16_f16(af[s & 1], bf[s & 1][c], acc[c], 0, 0, 0);
          __builtin_amdgcn_sched_barrier(0);
        }
        asm volatile("s_waitcnt lgkmcnt(0)" ::: "memory");
        BAR();
      }
      if (wv == 0) STAMP(6);
      const bool bad = *lds_bad != 0u;
      const int g = a.rank >> 2;
      const int col = g * 2048 + ct * 128 + 4 * (lane & 31);
      const float qnan = __builtin_nanf("");
      if (rt < 4) {
        float* o = a.out[a.rank] + (size_t)(rt * 128 + wv * 32 + 4 * (lane >> 5)) * NCOL + col;
#pragma unroll
        for (int e = 0; e < 16; ++e) {
          f32x4 v;
#pragma unroll
          for (int c = 0; c < 4; ++c) { const float y = acc[c][e]; v[c] = bad ? qnan : (y < 0.f ? 0.f : y); }
          *(f32x4*)(o + (size_t)((e & 3) + 8 * (e >> 2)) * NCOL) = v;
        }
      } else {
        LAS char* buf = (LAS char*)lds + wv * CSLOT;
        const int q = ((g ^ 1) << 2) + (rt - 4);
        const int slot = ((a.rank & 3) * 16 + ct) * 4 + wv;
        char* dst = a.ws[q & 7] + OFF_CSLOT + (size_t)slot * CSLOT + lane * 16;
        if (bad) {
          const unsigned nb = 0x7fc00000u;
          for (int i = 0; i < 16; ++i)
            asm volatile("global_store_dwordx4 %0, %1, off sc0 sc1\n\ts_nop 1" :: "v"(dst + i * 1024), "v"((u32x4){nb, nb, nb, nb}) : "memory");
          if (lane < 32)
            asm volatile("global_store_dwordx4 %0, %1, off sc0 sc1\n\ts_nop 1" :: "v"(dst + 16384), "v"((u32x4){~0u, ~0u, ~0u, ~0u}) : "memory");
        } else {
        unsigned base = 0, sent = 0, mhi = 0, mlo = 0;
#pragma unroll
        for (int e4 = 0; e4 < 4; ++e4) {
#pragma unroll
          for (int ee = 0; ee < 4; ++ee) {
#pragma unroll
            for (int c = 0; c < 4; ++c) {
              const float y = acc[c][e4 * 4 + ee];
              const bool nz = !(y <= 0.f);
              const unsigned long long m = __ballot(nz);
              unsigned pos = __builtin_amdgcn_mbcnt_hi((unsigned)(m >> 32), __builtin_amdgcn_mbcnt_lo((unsigned)m, base));
              asm volatile("" : "+v"(pos));
              *(LAS float*)(buf + 4u * (nz ? pos : 4288u + lane)) = y;
              if (e4 < 2) mhi = mhi * 2u + (nz ? 1u : 0u); else mlo = mlo * 2u + (nz ? 1u : 0u);
              base += (unsigned)__builtin_popcountll(m);
            }
          }
          if (e4 == 3) {
            *(LAS u32x2*)(buf + 16384 + lane * 8) = (u32x2){mlo, mhi};
            base = (base + 255u) & ~255u;
          }
          for (; sent + 256u <= base; sent += 256u) {
            const u32x4 v = *(const LAS u32x4*)(buf + sent * 4u + lane * 16);
            asm volatile("global_store_dwordx4 %0, %1, off sc0 sc1\n\ts_nop 1" :: "v"(dst + sent * 4u), "v"(v) : "memory");
          }
        }
        if (lane < 32) {
          const u32x4 v = *(const LAS u32x4*)(buf + 16384 + lane * 16);
          asm volatile("global_store_dwordx4 %0, %1, off sc0 sc1\n\ts_nop 1" :: "v"(dst + 16384), "v"(v) : "memory");
        }
        }
        if (wv == 0) STAMP(10);
        asm volatile("s_waitcnt vmcnt(0)" ::: "memory");
        if (lane == 0) st_sys(a.ws[q & 7] + OFF_OFLAG + (size_t)slot * 4, call);
      }
    }
    if (wv == 0) STAMP(7);
  }
  if (tid == 0) {
    const unsigned t = __hip_atomic_fetch_add((unsigned*)(myws + OFF_ENDCNT), 1u, __ATOMIC_RELAXED, __HIP_MEMORY_SCOPE_AGENT);
    if (t % GRID == GRID - 1) st_agent(myws + OFF_EPOCH, call);
  }
}

}

extern "C" void kernel_launch(int rank, int nranks, ncclComm_t comm,
                              hipStream_t stream,
                              void* const* d_in, const int* in_sizes, int n_in,
                              void* d_out, int out_size,
                              const int* in_dtypes, int out_dtype,
                              void* const* d_out_all,
                              void* const* d_ws_all, size_t ws_bytes) {
  (void)comm; (void)in_dtypes; (void)out_dtype; (void)d_out;
  if (nranks != NR || (unsigned)rank >= (unsigned)NR || n_in < 2 || ws_bytes < WS_NEED ||
      in_sizes[0] != 4096 * KSH || in_sizes[1] != KSH * NCOL || out_size != 512 * NCOL) {
    fprintf(stderr, "gemm_rs: unexpected problem shape\n");
    return;
  }
  Args a;
  for (int p = 0; p < NR; ++p) { a.ws[p] = (char*)d_ws_all[p]; a.out[p] = (float*)d_out_all[p]; }
  a.x = (const float*)d_in[0];
  a.w = (const float*)d_in[1];
  a.rank = rank;
  a.pad = 0;
  gemm_rs_kernel<<<GRID, THREADS, 0, stream>>>(a);
  hipError_t e = hipGetLastError();
  if (e != hipSuccess) fprintf(stderr, "gemm_rs: launch failed: %s\n", hipGetErrorString(e));
}
```

```cpp
#include <hip/hip_runtime.h>
#include <rccl/rccl.h>
#include <cstdio>

namespace {

constexpr int NR = 8;
constexpr int NCOL = 4096;
constexpr int KSH = 512;
#define LATE_V 5
constexpr int LATE = LATE_V;

constexpr size_t OFF_EPOCH = 0;
constexpr size_t OFF_ENDCNT = 128;
constexpr size_t OFF_TMO = 256;
constexpr size_t OFF_OFLAG = 1024;
constexpr size_t OFF_FLAGS = 4096;
constexpr size_t OFF_STAGE = 1u << 20;
constexpr size_t PIECE = 16384;
constexpr size_t A_BYTES = 64 * 8 * PIECE;
constexpr size_t B_BYTES = 64 * 16 * PIECE;
constexpr size_t PAR_BYTES = A_BYTES + B_BYTES;
constexpr size_t CSLOT = 17408;
constexpr size_t OFF_CSLOT = OFF_STAGE + 2 * PAR_BYTES;
constexpr size_t WS_NEED = OFF_CSLOT + 256 * CSLOT;

constexpr int NPROD = 64, NCONS = 128, GRID = NPROD + NCONS;
constexpr int THREADS = 512;
constexpr int NSTAGE = 6;
constexpr int DIST = 4;
constexpr int LDS_CTRL = 6 * 16384;
constexpr unsigned long long DEADLINE = 30000000ull;

typedef unsigned u32x4 __attribute__((ext_vector_type(4)));
typedef unsigned u32x2 __attribute__((ext_vector_type(2)));
typedef float f32x4 __attribute__((ext_vector_type(4)));
typedef float f32x16 __attribute__((ext_vector_type(16)));
typedef _Float16 h2 __attribute__((ext_vector_type(2)));
typedef _Float16 h8 __attribute__((ext_vector_type(8)));
#define LAS __attribute__((address_space(3)))
#define GAS __attribute__((address_space(1)))

struct Args {
  char* ws[NR];
  float* out[NR];
  const float* x;
  const float* w;
  int rank;
  int pad;
};

__device__ __forceinline__ unsigned ld_sys(const void* p) {
  return __hip_atomic_load((const unsigned*)p, __ATOMIC_RELAXED, __HIP_MEMORY_SCOPE_SYSTEM);
}
__device__ __forceinline__ unsigned ld_agent(const void* p) {
  return __hip_atomic_load((const unsigned*)p, __ATOMIC_RELAXED, __HIP_MEMORY_SCOPE_AGENT);
}
__device__ __forceinline__ void st_sys(void* p, unsigned v) {
  __hip_atomic_store((unsigned*)p, v, __ATOMIC_RELAXED, __HIP_MEMORY_SCOPE_SYSTEM);
}
__device__ __forceinline__ void st_agent(void* p, unsigned v) {
  __hip_atomic_store((unsigned*)p, v, __ATOMIC_RELAXED, __HIP_MEMORY_SCOPE_AGENT);
}
__device__ __forceinline__ void st16_sys(void* p, u32x4 v) {
  asm volatile("global_store_dwordx4 %0, %1, off sc0 sc1\n\ts_nop 1" :: "v"(p), "v"(v) : "memory");
}
__device__ __forceinline__ unsigned pk(float a, float b) {
  h2 h; h.x = (_Float16)a; h.y = (_Float16)b; return __builtin_bit_cast(unsigned, h);
}
#define STAMP(i) do {} while (0)
#define BAR() do { asm volatile("" ::: "memory"); __builtin_amdgcn_s_barrier(); asm volatile("" ::: "memory"); } while (0)

struct Piece { int kc, tile; };
__device__ __forceinline__ Piece decode(int n) {
  Piece p;
  constexpr int nf = LATE * 24, n2 = nf + (8 - LATE) * 20;
  int i;
  if (n < nf) { p.kc = n / 24; i = n % 24; }
  else if (n < n2) { const int v = n - nf; p.kc = LATE + v / 20; i = v % 20; }
  else { const int v = n - n2; p.kc = LATE + v / 4; i = 20 + v % 4; }
  p.tile = i < 16 ? 8 + i : (i < 20 ? i - 12 : i - 20);
  return p;
}
struct Src { const char* base; unsigned ustep, loff, js; };
__device__ __forceinline__ Src src_of(const Args& a, int d, const Piece& p, int lane) {
  Src s;
  const unsigned i = lane & 31, h = lane >> 5;
  if (p.tile >= 8) {
    s.base = (const char*)a.w + ((size_t)(p.kc * 64) * NCOL + (d >> 2) * 2048 + (p.tile - 8) * 128) * 4;
    s.ustep = NCOL * 4; s.loff = h * (32 * NCOL * 4) + i * 16; s.js = 4 * NCOL * 4;
  } else {
    const int row0 = p.tile < 4 ? d * 512 + p.tile * 128 : (((d >> 2) ^ 1) * 4 + (p.tile - 4)) * 512 + (d & 3) * 128;
    s.base = (const char*)a.x + ((size_t)row0 * KSH + p.kc * 64) * 4;
    s.ustep = 32 * KSH * 4; s.loff = i * (KSH * 4) + h * 128; s.js = 16;
  }
  return s;
}
__device__ __forceinline__ void load8(f32x4 (&r)[8], const char* base, unsigned loff, unsigned js) {
  unsigned o[8];
#pragma unroll
  for (int j = 0; j < 8; ++j) o[j] = loff + j * js;
  asm volatile(
      "s_nop 4\n\t"
      "global_load_dwordx4 %0, %8, %16\n\t"
      "global_load_dwordx4 %1, %9, %16\n\t"
      "global_load_dwordx4 %2, %10, %16\n\t"
      "global_load_dwordx4 %3, %11, %16\n\t"
      "global_load_dwordx4 %4, %12, %16\n\t"
      "global_load_dwordx4 %5, %13, %16\n\t"
      "global_load_dwordx4 %6, %14, %16\n\t"
      "global_load_dwordx4 %7, %15, %16"
      : "=&v"(r[0]), "=&v"(r[1]), "=&v"(r[2]), "=&v"(r[3]), "=&v"(r[4]), "=&v"(r[5]), "=&v"(r[6]), "=&v"(r[7])
      : "v"(o[0]), "v"(o[1]), "v"(o[2]), "v"(o[3]), "v"(o[4]), "v"(o[5]), "v"(o[6]), "v"(o[7]), "s"(base)
      : "memory");
}
#define WAIT_R(N, r) asm volatile("s_waitcnt vmcnt(" #N ")" : "+v"(r[0]), "+v"(r[1]), "+v"(r[2]), "+v"(r[3]), \
    "+v"(r[4]), "+v"(r[5]), "+v"(r[6]), "+v"(r[7]) :: "memory")
__device__ __forceinline__ void cvt4(const f32x4 (&r)[8], u32x4 (&fr)[4]) {
#pragma unroll
  for (int c = 0; c < 4; ++c)
    fr[c] = (u32x4){pk(r[0][c], r[1][c]), pk(r[2][c], r[3][c]), pk(r[4][c], r[5][c]), pk(r[6][c], r[7][c])};
}
__device__ __forceinline__ void store4(char* dbase, unsigned voff, const u32x4 (&fr)[4]) {
  asm volatile(
      "s_nop 4\n\t"
      "global_store_dwordx4 %0, %1, %5 sc0 sc1\n\t"
      "global_store_dwordx4 %0, %2, %5 offset:1024 sc0 sc1\n\t"
      "global_store_dwordx4 %0, %3, %5 offset:2048 sc0 sc1\n\t"
      "global_store_dwordx4 %0, %4, %5 offset:3072 sc0 sc1\n\t"
      "s_nop 1"
      :: "v"(voff), "v"(fr[0]), "v"(fr[1]), "v"(fr[2]), "v"(fr[3]), "s"(dbase) : "memory");
}

__device__ __forceinline__ void producer(const Args& a, int d, int slot, int lane) {
  constexpr int NP = 192, NSLOT = NPROD / NR;
  char* dws = a.ws[d];
  unsigned call_v;
  asm volatile("s_nop 4\n\tglobal_load_dword %0, %1, %2 sc1" : "=&v"(call_v) : "v"(0u), "s"(a.ws[a.rank] + OFF_EPOCH) : "memory");
  const unsigned voff = lane * 16;
  f32x4 r0[8], r1[8];
  u32x4 fr[4];
  STAMP(0);
  Piece p = decode(slot);
  Src s = src_of(a, d, p, lane);
  load8(r0, s.base, s.loff, s.js);
  load8(r1, s.base + s.ustep, s.loff, s.js);
  WAIT_R(0, r0); WAIT_R(0, r1);
  asm volatile("s_nop 0" : "+v"(call_v));
  const unsigned call = __builtin_amdgcn_readfirstlane(call_v) + 1u;
  char* stage = dws + OFF_STAGE + (size_t)(call & 1u) * PAR_BYTES;
  char* prevflag = nullptr;
  STAMP(1);
  for (int n = slot; n < NP; n += NSLOT) {
    const int nn = n + NSLOT < NP ? n + NSLOT : n;
    const Piece pn = decode(nn);
    const Src sn = src_of(a, d, pn, lane);
    const int cs = a.rank * 8 + p.kc;
    char* dst = p.tile >= 8 ? stage + A_BYTES + ((size_t)cs * 16 + (p.tile - 8)) * PIECE : stage + ((size_t)cs * 8 + p.tile) * PIECE;
    WAIT_R(16, r0); cvt4(r0, fr); load8(r0, s.base + 2 * s.ustep, s.loff, s.js); store4(dst, voff, fr);
    WAIT_R(16, r1); cvt4(r1, fr); load8(r1, s.base + 3 * s.ustep, s.loff, s.js); store4(dst + 4096, voff, fr);
    asm volatile("s_waitcnt vmcnt(24)" ::: "memory");
    if (prevflag != nullptr && lane == 0) st_sys(prevflag, call);
    WAIT_R(16, r0); cvt4(r0, fr); load8(r0, sn.base, sn.loff, sn.js); store4(dst + 8192, voff, fr);
    WAIT_R(16, r1); cvt4(r1, fr); load8(r1, sn.base + sn.ustep, sn.loff, sn.js); store4(dst + 12288, voff, fr);
    prevflag = dws + OFF_FLAGS + (size_t)(p.tile * 64 + cs) * 4;
    p = pn; s = sn;
  }
  asm volatile("s_waitcnt vmcnt(0)" : "+v"(r0[0]), "+v"(r1[0]) :: "memory");
  if (lane == 0) st_sys(prevflag, call);
  STAMP(2);
}

__device__ __forceinline__ void expander(const Args& a, unsigned call, int ti, int part, int half, int lane,
                                         LAS char* buf, LAS unsigned* sync) {
  char* myws = a.ws[a.rank];
  const int slot = ti * 4 + part;
  const char* flag = myws + OFF_OFLAG + (size_t)slot * 4;
  bool bad = false;
  {
    const unsigned long long t0 = __builtin_amdgcn_s_memrealtime();
    for (;;) {
      const unsigned fv = ld_sys(flag), tm = ld_agent(myws + OFF_TMO);
      if (__builtin_amdgcn_readfirstlane(fv) >= call) break;
      if (__builtin_amdgcn_readfirstlane(tm) == call || __builtin_amdgcn_s_memrealtime() - t0 > DEADLINE) {
        if (lane == 0) st_agent(myws + OFF_TMO, call);
        bad = true;
        break;
      }
      __builtin_amdgcn_s_sleep(1);
    }
  }
  if (half == 0) STAMP(3 + part * 2);
  const char* src = myws + OFF_CSLOT + (size_t)slot * CSLOT + lane * 16;
  if (!bad) {
#pragma unroll
    for (int i = 0; i < 5; ++i)
      __builtin_amdgcn_global_load_lds((const GAS void*)(src + (half * 5 + i) * 1024), (LAS void*)(buf + (half * 5 + i) * 1024), 16, 0, 17);
    if (half == 0) __builtin_amdgcn_global_load_lds((const GAS void*)(src + 16384), (LAS void*)(buf + 16384), 16, 0, 17);
  }
  const unsigned tm = __builtin_amdgcn_readfirstlane(ld_agent(myws + OFF_TMO));
  asm volatile("s_waitcnt vmcnt(0)" ::: "memory");
  bad = bad || tm == call;
  if (lane == 0) __hip_atomic_store(sync + half, bad ? 2u : 1u, __ATOMIC_RELEASE, __HIP_MEMORY_SCOPE_WORKGROUP);
  {
    unsigned o;
    const unsigned long long t0 = __builtin_amdgcn_s_memrealtime();
    while ((o = __builtin_amdgcn_readfirstlane(__hip_atomic_load(sync + (half ^ 1), __ATOMIC_ACQUIRE, __HIP_MEMORY_SCOPE_WORKGROUP))) == 0u) {
      if (__builtin_amdgcn_s_memrealtime() - t0 > 2 * DEADLINE) { o = 2u; break; }
    }
    bad = bad || o == 2u;
  }
  float* o = a.out[a.rank] + (size_t)((ti >> 4) * 128 + part * 32 + half * 16 + 4 * (lane >> 5)) * NCOL +
             ((a.rank >> 2) ^ 1) * 2048 + (ti & 15) * 128 + 4 * (lane & 31);
  if (bad) {
    for (int e = 0; e < 8; ++e)
      st16_sys(o + (size_t)((e & 3) + 8 * (e >> 2)) * NCOL, (u32x4){0x7fc00000u, 0x7fc00000u, 0x7fc00000u, 0x7fc00000u});
    return;
  }
  const u32x2 m = *(const LAS u32x2*)(buf + 16384 + lane * 8);
  const u32x2 cnt = *(const LAS u32x2*)(buf + 16896);
  const unsigned n0 = __builtin_amdgcn_readfirstlane(cnt[0]), total = __builtin_amdgcn_readfirstlane(cnt[1]);
  if (total > 2560u) {
#pragma unroll
    for (int i = 10; i < 16; ++i)
      __builtin_amdgcn_global_load_lds((const GAS void*)(src + i * 1024), (LAS void*)(buf + i * 1024), 16, 0, 17);
    asm volatile("s_waitcnt vmcnt(0)" ::: "memory");
  }
  unsigned base = half ? n0 : 0u;
  unsigned bits = half ? m[0] : m[1];
#pragma unroll
  for (int e = 0; e < 8; ++e) {
    f32x4 v;
#pragma unroll
    for (int c = 0; c < 4; ++c) {
      const bool nz = (int)bits < 0;
      bits <<= 1;
      const unsigned long long mm = __ballot(nz);
      unsigned pos = __builtin_amdgcn_mbcnt_hi((unsigned)(mm >> 32), __builtin_amdgcn_mbcnt_lo((unsigned)mm, base)) & 4095u;
      asm volatile("" : "+v"(pos));
      const float x = *(const LAS float*)(buf + pos * 4u);
      v[c] = nz ? x : 0.f;
      base += (unsigned)__builtin_popcountll(mm);
    }
    st16_sys(o + (size_t)((e & 3) + 8 * (e >> 2)) * NCOL, __builtin_bit_cast(u32x4, v));
  }
  if (half == 0) STAMP(4 + part * 2);
}

__global__ void __launch_bounds__(THREADS) gemm_rs_kernel(Args a) {
  __shared__ __attribute__((aligned(16))) char lds[LDS_CTRL + 512];
  const int tid = threadIdx.x, lane = tid & 63;
  const int wv = __builtin_amdgcn_readfirstlane(tid >> 6);
  const int b = blockIdx.x, xcd = b & 7, jb = b >> 3;
  char* myws = a.ws[a.rank];
  unsigned call = 0;

  if (jb < 8) {
    call = __builtin_amdgcn_readfirstlane(ld_agent(myws + OFF_EPOCH)) + 1u;
    LAS unsigned* sync = (LAS unsigned*)((LAS char*)lds + LDS_CTRL);
    if (wv == 0) {
      producer(a, xcd, jb, lane);
    } else if (wv == 1 && lane < 8) {
      sync[lane] = 0u;
    }
    asm volatile("s_waitcnt lgkmcnt(0)" ::: "memory");
    BAR();
    expander(a, call, jb * 8 + xcd, wv >> 1, wv & 1, lane, (LAS char*)lds + (wv >> 1) * CSLOT, sync + (wv >> 1) * 2);
  } else {
    const int jc = jb - 8;
    const int rt = (xcd & 3) + (jc >> 3) * 4;
    const int ct = (xcd >> 2) * 8 + (jc & 7);
    call = __builtin_amdgcn_readfirstlane(ld_agent(myws + OFF_EPOCH)) + 1u;
    LAS unsigned* lds_bad = (LAS unsigned*)((LAS char*)lds + LDS_CTRL);
    LAS unsigned* lds_sched = (LAS unsigned*)((LAS char*)lds + LDS_CTRL + 64);
    const char* stage = myws + OFF_STAGE + (size_t)(call & 1u) * PAR_BYTES;
    if (wv == 6) {
      const char* fa = myws + OFF_FLAGS + (size_t)(rt * 64 + lane) * 4;
      const char* fb = myws + OFF_FLAGS + (size_t)((8 + ct) * 64 + lane) * 4;
      const char* ptm = myws + OFF_TMO;
      const char* pa = stage + (size_t)rt * PIECE + lane * 128;
      const char* pb = stage + A_BYTES + (size_t)ct * PIECE + lane * 128;
      LAS char* sink = (LAS char*)lds + LDS_CTRL + 256;
      unsigned long long ready = 0, done = 0, seen = 0;
      STAMP(0);
      for (int c = 0; c < 64; ++c) {
        if ((ready & ~done) == 0ull) {
          const unsigned long long t0 = __builtin_amdgcn_s_memrealtime();
          for (;;) {
            const unsigned va = ld_sys(fa), vb = ld_sys(fb), tm = ld_agent(ptm);
            const unsigned lo = min(va, vb);
            ready = __ballot(lo >= call);
            if ((ready & ~done) != 0ull) break;
            if (__builtin_amdgcn_readfirstlane(tm) == call || __builtin_amdgcn_s_memrealtime() - t0 > DEADLINE) {
              if (lane == 0) st_agent(myws + OFF_TMO, call);
              ready = ~0ull; seen = ~0ull;
              break;
            }
            __builtin_amdgcn_s_sleep(1);
          }
        }
        const int cs = __builtin_ctzll(ready & ~done);
        done |= 1ull << cs;
        seen |= 1ull << cs;
        if (lane == 0) lds_sched[c & 7] = (unsigned)cs;
        if (c == 0) STAMP(1);
        if (c == 8) STAMP(2);
        if (c == 32) STAMP(3);
        if (c == 56) STAMP(4);
        if (c == 63) STAMP(5);
        if (c == 63) {
          const unsigned t = __builtin_amdgcn_readfirstlane(ld_agent(myws + OFF_TMO));
          if (lane == 0) *lds_bad = (t == call) ? 1u : 0u;
        }
        asm volatile("s_waitcnt lgkmcnt(0)" ::: "memory");
        BAR();
        unsigned long long fresh = ready & ~seen;
        seen |= fresh;
        while (fresh) {
          const int k = __builtin_ctzll(fresh);
          fresh &= fresh - 1;
          const char* ga = pa + (size_t)k * (8 * PIECE);
          const char* gb = pb + (size_t)k * (16 * PIECE);
          __builtin_amdgcn_global_load_lds((const GAS void*)ga, (LAS void*)sink, 4, 0, 16);
          __builtin_amdgcn_global_load_lds((const GAS void*)(ga + 8192), (LAS void*)sink, 4, 0, 16);
          __builtin_amdgcn_global_load_lds((const GAS void*)gb, (LAS void*)sink, 4, 0, 16);
          __builtin_amdgcn_global_load_lds((const GAS void*)(gb + 8192), (LAS void*)sink, 4, 0, 16);
        }
      }
      for (int i = 0; i <= DIST; ++i) BAR();
      asm volatile("s_waitcnt vmcnt(0)" ::: "memory");
    } else if (wv == 7) {
    } else if (wv >= 4) {
      const int lw = wv - 4;
      const char* gbase = stage + A_BYTES + (size_t)ct * PIECE + lw * 8192 + lane * 16;
      BAR();
      int st = 0;
      for (int c = 0; c < 64 + DIST; ++c) {
        if (c < 64) {
          const int cs = __builtin_amdgcn_readfirstlane(*(volatile LAS unsigned*)(lds_sched + (c & 7))) & 63;
          const char* ga = gbase + (size_t)cs * (16 * PIECE);
          LAS char* la = (LAS char*)lds + st * 16384 + lw * 8192;
#pragma unroll
          for (int f = 0; f < 8; ++f)
            __builtin_amdgcn_global_load_lds((const GAS void*)(ga + f * 1024), (LAS void*)(la + f * 1024), 16, 0, 17);
          asm volatile("s_waitcnt vmcnt(16)" ::: "memory");
        } else if (c == 64) {
          asm volatile("s_waitcnt vmcnt(8)" ::: "memory");
        } else {
          asm volatile("s_waitcnt vmcnt(0)" ::: "memory");
        }
        st = st == NSTAGE - 1 ? 0 : st + 1;
        BAR();
      }
    } else {
      f32x16 acc[4];
#pragma unroll
      for (int c = 0; c < 4; ++c)
#pragma unroll
        for (int e = 0; e < 16; ++e) acc[c][e] = 0.f;
      const __amdgpu_buffer_rsrc_t rsa = __builtin_amdgcn_make_buffer_rsrc((void*)(stage + (size_t)rt * PIECE), 0, (int)A_BYTES, 0x00020000);
      const unsigned aoff = wv * 4096 + lane * 16;
      h8 ar[NSTAGE][4];
      h8 bf[2][4];
#define SCHED_READ(C) (*(volatile LAS unsigned*)(lds_sched + ((C) & 7)))
#define LOAD_A(SLOT, SV) do {                                                                            \
        const unsigned cs_ = __builtin_amdgcn_readfirstlane(SV) & 63u;                                   \
        _Pragma("unroll") for (int s_ = 0; s_ < 4; ++s_)                                                 \
          ar[SLOT][s_] = __builtin_bit_cast(h8, __builtin_amdgcn_raw_buffer_load_b128(rsa, aoff + s_ * 1024, cs_ * (unsigned)(8 * PIECE), 17)); \
      } while (0)
#define READ_B(BUF, ST, S) do {                                                                          \
        _Pragma("unroll") for (int c_ = 0; c_ < 4; ++c_)                                                 \
          bf[BUF][c_] = *(const LAS h8*)((const LAS char*)lds + (ST) * 16384 + (S) * 4096 + c_ * 1024 + lane * 16); \
      } while (0)
#define MMA4(SLOT, S) do {                                                                               \
        __builtin_amdgcn_sched_barrier(0);                                                               \
        _Pragma("unroll") for (int c_ = 0; c_ < 4; ++c_)                                                 \
          acc[c_] = __builtin_amdgcn_mfma_f32_32x32x16_f16(ar[SLOT][S], bf[(S) & 1][c_], acc[c_], 0, 0, 0); \
        __builtin_amdgcn_sched_barrier(0);                                                               \
      } while (0)
#define STEP(K, T, MORE) do {                                                                            \
        READ_B(1, K, 1);                                                                                 \
        unsigned sv_ = 0; if (MORE) sv_ = SCHED_READ((T) + DIST);                                        \
        MMA4(K, 0);                                                                                      \
        READ_B(0, K, 2);                                                                                 \
        if (MORE) LOAD_A(((K) + DIST) % NSTAGE, sv_);                                                    \
        MMA4(K, 1);                                                                                      \
        READ_B(1, K, 3);                                                                                 \
        MMA4(K, 2);                                                                                      \
        READ_B(0, ((K) + 1) % NSTAGE, 0);                                                                \
        MMA4(K, 3);                                                                                      \
        asm volatile("s_waitcnt lgkmcnt(4)" ::: "memory");        \
        BAR();                                                                                           \
      } while (0)
      BAR(); LOAD_A(0, SCHED_READ(0)); BAR(); LOAD_A(1, SCHED_READ(1)); BAR(); LOAD_A(2, SCHED_READ(2));
      BAR(); LOAD_A(3, SCHED_READ(3)); BAR();
      READ_B(0, 0, 0);
      for (int t = 0; t < 60; t += 6) {
        STEP(0, t, true); STEP(1, t + 1, true); STEP(2, t + 2, true);
        STEP(3, t + 3, true); STEP(4, t + 4, true); STEP(5, t + 5, true);
      }
      STEP(0, 60, false); STEP(1, 61, false); STEP(2, 62, false); STEP(3, 63, false);
#undef SCHED_READ
#undef LOAD_A
#undef READ_B
#undef MMA4
#undef STEP
      if (wv == 0) STAMP(6);
      const bool bad = *lds_bad != 0u;
      const int g = a.rank >> 2;
      if (rt < 4) {
        const float qnan = __builtin_nanf("");
        float* o = a.out[a.rank] + (size_t)(rt * 128 + wv * 32 + 4 * (lane >> 5)) * NCOL + g * 2048 + ct * 128 + 4 * (lane & 31);
#pragma unroll
        for (int e = 0; e < 16; ++e) {
          f32x4 v;
#pragma unroll
          for (int c = 0; c < 4; ++c) { const float y = acc[c][e]; v[c] = bad ? qnan : (y < 0.f ? 0.f : y); }
          st16_sys(o + (size_t)((e & 3) + 8 * (e >> 2)) * NCOL, __builtin_bit_cast(u32x4, v));
        }
      } else {
        LAS char* buf = (LAS char*)lds + wv * CSLOT;
        const int q = ((g ^ 1) << 2) + (rt - 4);
        const int slot = ((a.rank & 3) * 16 + ct) * 4 + wv;
        char* dst = a.ws[q & 7] + OFF_CSLOT + (size_t)slot * CSLOT + lane * 16;
        if (bad) {
          const unsigned nb = 0x7fc00000u;
          for (int i = 0; i < 16; ++i) st16_sys(dst + i * 1024, (u32x4){nb, nb, nb, nb});
          if (lane < 32) st16_sys(dst + 16384, (u32x4){~0u, ~0u, ~0u, ~0u});
          if (lane == 32) st16_sys(dst + 16384, (u32x4){2048u, 4096u, 0u, 0u});
        } else {
          unsigned base = 0, sent = 0, n0 = 0, mhi = 0, mlo = 0;
#pragma unroll
          for (int e4 = 0; e4 < 4; ++e4) {
#pragma unroll
            for (int ee = 0; ee < 4; ++ee) {
#pragma unroll
              for (int c = 0; c < 4; ++c) {
                const float y = acc[c][e4 * 4 + ee];
                const bool nz = !(y <= 0.f);
                const unsigned long long m = __ballot(nz);
                unsigned pos = __builtin_amdgcn_mbcnt_hi((unsigned)(m >> 32), __builtin_amdgcn_mbcnt_lo((unsigned)m, base));
                asm volatile("" : "+v"(pos));
                *(LAS float*)(buf + 4u * (nz ? pos : 4288u + lane)) = y;
                if (e4 < 2) mhi = mhi * 2u + (nz ? 1u : 0u); else mlo = mlo * 2u + (nz ? 1u : 0u);
                base += (unsigned)__builtin_popcountll(m);
              }
            }
            if (e4 == 1) n0 = base;
            asm volatile("s_waitcnt vmcnt(0)" ::: "memory");
            for (; sent + 256u <= base; sent += 256u)
              st16_sys(dst + sent * 4u, *(const LAS u32x4*)(buf + sent * 4u + lane * 16));
          }
          if (sent + lane * 4u < base) st16_sys(dst + sent * 4u, *(const LAS u32x4*)(buf + sent * 4u + lane * 16));
          *(LAS u32x2*)(buf + 16384 + lane * 8) = (u32x2){mlo, mhi};
          if (lane == 0) *(LAS u32x2*)(buf + 16896) = (u32x2){n0, base};
          if (lane < 33) st16_sys(dst + 16384, *(const LAS u32x4*)(buf + 16384 + lane * 16));
        }
        if (wv == 0) STAMP(10);
        asm volatile("s_waitcnt vmcnt(0)" ::: "memory");
        if (lane == 0) st_sys(a.ws[q & 7] + OFF_OFLAG + (size_t)slot * 4, call);
      }
      if (wv == 0) STAMP(7);
    }
  }
  if (tid == 0) {
    const unsigned t = __hip_atomic_fetch_add((unsigned*)(myws + OFF_ENDCNT), 1u, __ATOMIC_RELAXED, __HIP_MEMORY_SCOPE_AGENT);
    if (t % GRID == GRID - 1) st_agent(myws + OFF_EPOCH, call);
  }
}

}

extern "C" void kernel_launch(int rank, int nranks, ncclComm_t comm,
                              hipStream_t stream,
                              void* const* d_in, const int* in_sizes, int n_in,
                              void* d_out, int out_size,
                              const int* in_dtypes, int out_dtype,
                              void* const* d_out_all,
                              void* const* d_ws_all, size_t ws_bytes) {
  (void)comm; (void)in_dtypes; (void)out_dtype; (void)d_out;
  if (nranks != NR || (unsigned)rank >= (unsigned)NR || n_in < 2 || ws_bytes < WS_NEED ||
      in_sizes[0] != 4096 * KSH || in_sizes[1] != KSH * NCOL || out_size != 512 * NCOL) {
    fprintf(stderr, "gemm_rs: unexpected problem shape\n");
    return;
  }
  Args a;
  for (int p = 0; p < NR; ++p) { a.ws[p] = (char*)d_ws_all[p]; a.out[p] = (float*)d_out_all[p]; }
  a.x = (const float*)d_in[0];
  a.w = (const float*)d_in[1];
  a.rank = rank;
  a.pad = 0;
  gemm_rs_kernel<<<GRID, THREADS, 0, stream>>>(a);
  hipError_t e = hipGetLastError();
  if (e != hipSuccess) fprintf(stderr, "gemm_rs: launch failed: %s\n", hipGetErrorString(e));
}
```

```cpp
#include <hip/hip_runtime.h>
#include <rccl/rccl.h>
#include <cstdio>

namespace {

constexpr int NR = 8;
constexpr int NCOL = 4096;
constexpr int KSH = 512;
#define LATE_V 4
constexpr int LATE = LATE_V;
#define KD_V 7
#define NSLOT_V 7
constexpr int NSLOT = NSLOT_V;
#define PACE_V 195
constexpr unsigned PACE_P = PACE_V * NSLOT / 8, PACE_D = PACE_P * 4096 / 3712;
constexpr int KD = KD_V;

constexpr size_t OFF_EPOCH = 0;
constexpr size_t OFF_ENDCNT = 128;
constexpr size_t OFF_TMO = 256;
constexpr size_t OFF_OFLAG = 1024;
constexpr size_t OFF_PFLAG = 4096;
constexpr size_t OFF_LFLAG = 16384;
constexpr size_t OFF_STAGE = 1u << 20;
constexpr size_t PIECE = 16384;
constexpr size_t A_BYTES = 64 * 8 * PIECE;
constexpr size_t B_BYTES = 64 * 16 * PIECE;
constexpr size_t PUNIT = 4224;
constexpr size_t PPAR = 64 * 24 * 4 * PUNIT;
constexpr size_t FPAR = A_BYTES + B_BYTES;
constexpr size_t OFF_PSTAGE = OFF_STAGE + 2 * FPAR;
constexpr size_t CSLOT = 17408;
constexpr size_t OFF_CSLOT = OFF_PSTAGE + 2 * PPAR;
constexpr size_t WS_NEED = OFF_CSLOT + 256 * CSLOT;

constexpr int NPROD = 64, NCONS = 128, NDEC = 56, GRID = NPROD + NCONS + NDEC;
constexpr int THREADS = 512;
constexpr int NSTAGE = 6;
constexpr int DIST = 4;
constexpr int LDS_CTRL = 6 * 16384;
constexpr unsigned long long DEADLINE = 30000000ull;

typedef unsigned u32x4 __attribute__((ext_vector_type(4)));
typedef unsigned u32x2 __attribute__((ext_vector_type(2)));
typedef float f32x4 __attribute__((ext_vector_type(4)));
typedef float f32x16 __attribute__((ext_vector_type(16)));
typedef _Float16 h2 __attribute__((ext_vector_type(2)));
typedef _Float16 h8 __attribute__((ext_vector_type(8)));
#define LAS __attribute__((address_space(3)))
#define GAS __attribute__((address_space(1)))

struct Args {
  char* ws[NR];
  float* out[NR];
  const float* x;
  const float* w;
  int rank;
  int pad;
};

__device__ __forceinline__ unsigned ld_sys(const void* p) {
  return __hip_atomic_load((const unsigned*)p, __ATOMIC_RELAXED, __HIP_MEMORY_SCOPE_SYSTEM);
}
__device__ __forceinline__ unsigned ld_agent(const void* p) {
  return __hip_atomic_load((const unsigned*)p, __ATOMIC_RELAXED, __HIP_MEMORY_SCOPE_AGENT);
}
__device__ __forceinline__ void st_sys(void* p, unsigned v) {
  __hip_atomic_store((unsigned*)p, v, __ATOMIC_RELAXED, __HIP_MEMORY_SCOPE_SYSTEM);
}
__device__ __forceinline__ void st_agent(void* p, unsigned v) {
  __hip_atomic_store((unsigned*)p, v, __ATOMIC_RELAXED, __HIP_MEMORY_SCOPE_AGENT);
}
__device__ __forceinline__ void st16_sys(void* p, u32x4 v) {
  asm volatile("global_store_dwordx4 %0, %1, off sc0 sc1\n\ts_nop 1" :: "v"(p), "v"(v) : "memory");
}
__device__ __forceinline__ unsigned pk(float a, float b) {
  h2 h; h.x = (_Float16)a; h.y = (_Float16)b; return __builtin_bit_cast(unsigned, h);
}
#define STAMP(i) do {} while (0)
#define BAR() do { asm volatile("" ::: "memory"); __builtin_amdgcn_s_barrier(); asm volatile("" ::: "memory"); } while (0)

typedef unsigned short us2 __attribute__((ext_vector_type(2)));
constexpr int PK_UNIT = (int)PUNIT;
constexpr int PK_LDS = 4096 + 128;
struct Packed { u32x4 l0, l1, c0; u32x2 c1; u32x4 hdr; };

__device__ __forceinline__ unsigned pk_sub(unsigned a, unsigned b) { return __builtin_bit_cast(unsigned, __builtin_bit_cast(us2, a) - __builtin_bit_cast(us2, b)); }
__device__ __forceinline__ unsigned pk_add(unsigned a, unsigned b) { return __builtin_bit_cast(unsigned, __builtin_bit_cast(us2, a) + __builtin_bit_cast(us2, b)); }
__device__ __forceinline__ unsigned pk_min(unsigned a, unsigned b) { return __builtin_bit_cast(unsigned, __builtin_elementwise_min(__builtin_bit_cast(us2, a), __builtin_bit_cast(us2, b))); }
template <int CTRL, int RM> __device__ __forceinline__ unsigned dpp_add(unsigned x) {
  return x + (unsigned)__builtin_amdgcn_update_dpp(0, (int)x, CTRL, RM, 0xf, false);
}
__device__ __forceinline__ unsigned wave_scan(unsigned x) {
  x = dpp_add<0x111, 0xf>(x); x = dpp_add<0x112, 0xf>(x); x = dpp_add<0x114, 0xf>(x); x = dpp_add<0x118, 0xf>(x);
  x = dpp_add<0x142, 0xa>(x); x = dpp_add<0x143, 0xc>(x);
  return x;
}
__device__ __forceinline__ unsigned esc_addr(unsigned b, int lane) {
  const unsigned j = 15u - (b & 15u);
  return (j >> 2) * 1024u + lane * 16u + (j & 3u) * 4u + (b >> 4) * 2u + 1u;
}

__device__ __forceinline__ bool pk_encode(const u32x4 (&fr)[4], Packed& p, int lane, LAS char* lds) {
  unsigned d[16];
#pragma unroll
  for (int j = 0; j < 16; ++j) d[j] = fr[j >> 2][j & 3];
  unsigned s = 0;
#pragma unroll
  for (int j = 0; j < 16; j += 5) s += ((d[j] >> 10) & 31u) + ((d[j] >> 26) & 31u);
  s = __builtin_amdgcn_readlane(wave_scan(s), 63);
  int top = (int)((s + 1388u) >> 9);
  int base = top - 6; base = base < 0 ? 0 : (base > 24 ? 24 : base);
  const unsigned base2 = (unsigned)base * 0x00010001u;
  unsigned q[16], acc = 0;
#pragma unroll
  for (int j = 0; j < 16; ++j) {
    const unsigned x = d[j] >> 10;
    const unsigned c = pk_min(pk_sub(x & 0x001f001fu, base2), 0x00070007u);
    const unsigned c6 = (x & 0x00200020u) | ((d[j] >> 5) & 0x00180018u) | c;
    q[j] = (c6 & 0x3fu) | (c6 >> 10);
    acc = pk_add(pk_add(acc, acc), (pk_add(c, 0x00010001u) >> 3) & 0x00010001u);
  }
#pragma unroll
  for (int i = 0; i < 4; ++i) {
    p.l0[i] = __builtin_amdgcn_perm(d[2 * i + 1], d[2 * i], 0x06040200u);
    p.l1[i] = __builtin_amdgcn_perm(d[2 * i + 9], d[2 * i + 8], 0x06040200u);
  }
  p.c0[0] = q[0] | (q[1] << 12) | (q[2] << 24);
  p.c0[1] = (q[2] >> 8) | (q[3] << 4) | (q[4] << 16) | (q[5] << 28);
  p.c0[2] = (q[5] >> 4) | (q[6] << 8) | (q[7] << 20);
  p.c0[3] = q[8] | (q[9] << 12) | (q[10] << 24);
  p.c1[0] = (q[10] >> 8) | (q[11] << 4) | (q[12] << 16) | (q[13] << 28);
  p.c1[1] = (q[13] >> 4) | (q[14] << 8) | (q[15] << 20);
  const unsigned cnt = (unsigned)__builtin_popcount(acc);
  const unsigned incl = wave_scan(cnt);
  const unsigned total = __builtin_amdgcn_readlane(incl, 63);
  if (total > 127u) { p.hdr = (u32x4){0x80u, 0u, 0u, 0u}; return true; }
  LAS char* list = lds + 4096;
  if (total != 0u) {
#pragma unroll
    for (int f = 0; f < 4; ++f) *(LAS u32x4*)(lds + f * 1024 + lane * 16) = fr[f];
    unsigned off = 1u + incl - cnt;
    while (acc != 0u) {
      const unsigned b = (unsigned)__builtin_ctz(acc);
      acc &= acc - 1u;
      const unsigned hb = *(const LAS unsigned char*)(lds + esc_addr(b, lane));
      *(LAS unsigned char*)(list + off) = (unsigned char)((hb >> 2) & 31u);
      ++off;
    }
  }
  if (lane == 0) *(LAS unsigned char*)list = (unsigned char)base;
  p.hdr = *(const LAS u32x4*)(list + (lane & 7) * 16);
  return false;
}

__device__ __forceinline__ void pk_decode(const Packed& p, unsigned h, u32x4 (&fr)[4], int lane, LAS char* lds) {
  const unsigned base2 = (__builtin_amdgcn_readfirstlane(h) & 31u) * 0x00010001u;
  unsigned q[16];
  q[0] = p.c0[0] & 0xfffu; q[1] = (p.c0[0] >> 12) & 0xfffu; q[2] = __builtin_amdgcn_alignbit(p.c0[1], p.c0[0], 24) & 0xfffu;
  q[3] = (p.c0[1] >> 4) & 0xfffu; q[4] = (p.c0[1] >> 16) & 0xfffu; q[5] = __builtin_amdgcn_alignbit(p.c0[2], p.c0[1], 28) & 0xfffu;
  q[6] = (p.c0[2] >> 8) & 0xfffu; q[7] = p.c0[2] >> 20;
  q[8] = p.c0[3] & 0xfffu; q[9] = (p.c0[3] >> 12) & 0xfffu; q[10] = __builtin_amdgcn_alignbit(p.c1[0], p.c0[3], 24) & 0xfffu;
  q[11] = (p.c1[0] >> 4) & 0xfffu; q[12] = (p.c1[0] >> 16) & 0xfffu; q[13] = __builtin_amdgcn_alignbit(p.c1[1], p.c1[0], 28) & 0xfffu;
  q[14] = (p.c1[1] >> 8) & 0xfffu; q[15] = p.c1[1] >> 20;
  unsigned acc = 0;
#pragma unroll
  for (int j = 0; j < 16; ++j) {
    const unsigned c6 = (q[j] | (q[j] << 10)) & 0x003f003fu;
    const unsigned c = c6 & 0x00070007u;
    const unsigned hi8 = ((c6 << 2) & 0x00800080u) | ((c6 >> 3) & 0x00030003u) | (pk_add(c, base2) << 2);
    const unsigned l = (j < 8 ? p.l0 : p.l1)[(j & 7) >> 1];
    fr[j >> 2][j & 3] = __builtin_amdgcn_perm(hi8, l, (j & 1) ? 0x06030402u : 0x06010400u);
    acc = pk_add(pk_add(acc, acc), (pk_add(c, 0x00010001u) >> 3) & 0x00010001u);
  }
  const unsigned cnt = (unsigned)__builtin_popcount(acc);
  const unsigned incl = wave_scan(cnt);
  if (__builtin_amdgcn_readlane(incl, 63) != 0u) {
    LAS char* list = lds + 4096;
    if (lane < 32) *(LAS unsigned*)(list + lane * 4) = h;
#pragma unroll
    for (int f = 0; f < 4; ++f) *(LAS u32x4*)(lds + f * 1024 + lane * 16) = fr[f];
    unsigned off = 1u + incl - cnt;
    while (acc != 0u) {
      const unsigned b = (unsigned)__builtin_ctz(acc);
      acc &= acc - 1u;
      LAS unsigned char* t = (LAS unsigned char*)(lds + esc_addr(b, lane));
      const unsigned e = *(const LAS unsigned char*)(list + off);
      *t = (unsigned char)((*t & 0x83u) | (e << 2));
      ++off;
    }
#pragma unroll
    for (int f = 0; f < 4; ++f) fr[f] = *(const LAS u32x4*)(lds + f * 1024 + lane * 16);
  }
}

struct Piece { int kc, tile; };
__device__ __forceinline__ Piece decode(int n) {
  Piece p;
  constexpr int nf = LATE * 24, n2 = nf + (8 - LATE) * 20;
  int i;
  if (n < nf) { p.kc = n / 24; i = n % 24; }
  else if (n < n2) { const int v = n - nf; p.kc = LATE + v / 20; i = v % 20; }
  else { const int v = n - n2; p.kc = LATE + v / 4; i = 20 + v % 4; }
  p.tile = i < 16 ? 8 + i : (i < 20 ? i - 12 : i - 20);
  return p;
}
__device__ __forceinline__ bool is_direct(const Piece& p) { return p.kc >= KD || (p.tile < 4 && p.kc >= LATE); }
__device__ __forceinline__ size_t stage_off(const Piece& p, int cs) {
  return p.tile >= 8 ? A_BYTES + ((size_t)cs * 16 + (p.tile - 8)) * PIECE : ((size_t)cs * 8 + p.tile) * PIECE;
}
struct Src { const char* base; unsigned ustep, loff, js; };
__device__ __forceinline__ Src src_of(const Args& a, int d, const Piece& p, int lane) {
  Src s;
  const unsigned i = lane & 31, h = lane >> 5;
  if (p.tile >= 8) {
    s.base = (const char*)a.w + ((size_t)(p.kc * 64) * NCOL + (d >> 2) * 2048 + (p.tile - 8) * 128) * 4;
    s.ustep = NCOL * 4; s.loff = h * (4 * NCOL * 4) + i * 16; s.js = 8 * NCOL * 4;
  } else {
    const int row0 = p.tile < 4 ? d * 512 + p.tile * 128 : (((d >> 2) ^ 1) * 4 + (p.tile - 4)) * 512 + (d & 3) * 128;
    s.base = (const char*)a.x + ((size_t)row0 * KSH + p.kc * 64) * 4;
    s.ustep = 32 * KSH * 4; s.loff = i * (KSH * 4) + h * 16; s.js = 32;
  }
  return s;
}
__device__ __forceinline__ void load8(f32x4 (&r)[8], const char* base, unsigned loff, unsigned js) {
  unsigned o[8];
#pragma unroll
  for (int j = 0; j < 8; ++j) o[j] = loff + j * js;
  asm volatile(
      "s_nop 4\n\t"
      "global_load_dwordx4 %0, %8, %16\n\t"
      "global_load_dwordx4 %1, %9, %16\n\t"
      "global_load_dwordx4 %2, %10, %16\n\t"
      "global_load_dwordx4 %3, %11, %16\n\t"
      "global_load_dwordx4 %4, %12, %16\n\t"
      "global_load_dwordx4 %5, %13, %16\n\t"
      "global_load_dwordx4 %6, %14, %16\n\t"
      "global_load_dwordx4 %7, %15, %16"
      : "=&v"(r[0]), "=&v"(r[1]), "=&v"(r[2]), "=&v"(r[3]), "=&v"(r[4]), "=&v"(r[5]), "=&v"(r[6]), "=&v"(r[7])
      : "v"(o[0]), "v"(o[1]), "v"(o[2]), "v"(o[3]), "v"(o[4]), "v"(o[5]), "v"(o[6]), "v"(o[7]), "s"(base)
      : "memory");
}
#define WAIT_R(N, r) asm volatile("s_waitcnt vmcnt(" #N ")" : "+v"(r[0]), "+v"(r[1]), "+v"(r[2]), "+v"(r[3]), \
    "+v"(r[4]), "+v"(r[5]), "+v"(r[6]), "+v"(r[7]) :: "memory")
__device__ __forceinline__ void cvt4(const f32x4 (&r)[8], u32x4 (&fr)[4]) {
#pragma unroll
  for (int c = 0; c < 4; ++c)
    fr[c] = (u32x4){pk(r[0][c], r[1][c]), pk(r[2][c], r[3][c]), pk(r[4][c], r[5][c]), pk(r[6][c], r[7][c])};
}
__device__ __forceinline__ void store4(char* dbase, unsigned voff, const u32x4 (&fr)[4]) {
  asm volatile(
      "s_nop 4\n\t"
      "global_store_dwordx4 %0, %1, %5 sc0 sc1\n\t"
      "global_store_dwordx4 %0, %2, %5 offset:1024 sc0 sc1\n\t"
      "global_store_dwordx4 %0, %3, %5 offset:2048 sc0 sc1\n\t"
      "global_store_dwordx4 %0, %4, %5 offset:3072 sc0 sc1\n\t"
      "s_nop 1"
      :: "v"(voff), "v"(fr[0]), "v"(fr[1]), "v"(fr[2]), "v"(fr[3]), "s"(dbase) : "memory");
}
__device__ __forceinline__ void store_packed(char* dbase, int lane, const Packed& p) {
  asm volatile(
      "s_nop 4\n\t"
      "global_store_dwordx4 %0, %2, %6 sc0 sc1\n\t"
      "global_store_dwordx4 %0, %3, %6 offset:1024 sc0 sc1\n\t"
      "global_store_dwordx4 %0, %4, %6 offset:2048 sc0 sc1\n\t"
      "global_store_dwordx2 %1, %5, %6 offset:3072 sc0 sc1\n\t"
      "s_nop 1"
      :: "v"(lane * 16), "v"(lane * 8), "v"(p.l0), "v"(p.l1), "v"(p.c0), "v"(p.c1), "s"(dbase) : "memory");
}

__device__ __forceinline__ void producer(const Args& a, unsigned call, int d, int slot, int u, int lane,
                                         LAS char* scratch, LAS unsigned* cnt) {
  constexpr int NP = 192;
  char* dws = a.ws[d];
  const bool self = d == a.rank;
  const unsigned voff = lane * 16;
  f32x4 r[8];
  u32x4 fr[4];
  if (u == 0) STAMP(0);
  Piece p = decode(slot);
  Src s = src_of(a, d, p, lane);
  load8(r, s.base + u * s.ustep, s.loff, s.js);
  char* pstage = dws + OFF_PSTAGE + (size_t)(call & 1u) * PPAR;
  char* fstage = dws + OFF_STAGE + (size_t)(call & 1u) * FPAR;
  char* prevflag = nullptr;
  bool prevdirect = false;
  unsigned long long tnext = 0;
  int i = 0;
  auto publish = [&](int idx) -> int {
    unsigned old = 0;
    if (lane == 0) old = __hip_atomic_fetch_add(cnt + idx, 1u, __ATOMIC_RELAXED, __HIP_MEMORY_SCOPE_WORKGROUP);
    if (__builtin_amdgcn_readfirstlane(old) != 3u) return 0;
    if (lane == 0) { if (prevdirect) st16_sys(prevflag, (u32x4){call, call, call, call}); else st_sys(prevflag, call); }
    return 1;
  };
  int after = 4;
  WAIT_R(0, r);
  if (u == 0) STAMP(1);
  for (int n = slot; n < NP; n += NSLOT, ++i) {
    const int nn = n + NSLOT < NP ? n + NSLOT : n;
    const Piece pn = decode(nn);
    const Src sn = src_of(a, d, pn, lane);
    const int cs = a.rank * 8 + p.kc;
    const bool direct = self || is_direct(p);
    asm volatile("s_waitcnt vmcnt(6)\n\ts_cmp_gt_u32 %8, 5\n\ts_cbranch_scc1 1f\n\ts_waitcnt vmcnt(5)\n\t"
                 "s_cmp_gt_u32 %8, 4\n\ts_cbranch_scc1 1f\n\ts_waitcnt vmcnt(4)\n1:"
                 : "+v"(r[0]), "+v"(r[1]), "+v"(r[2]), "+v"(r[3]), "+v"(r[4]), "+v"(r[5]), "+v"(r[6]), "+v"(r[7])
                 : "s"(__builtin_amdgcn_readfirstlane(after)) : "scc", "memory");
    cvt4(r, fr);
    load8(r, sn.base + u * sn.ustep, sn.loff, sn.js);
    Packed pk;
    bool raw = true;
    if (!direct) raw = pk_encode(fr, pk, lane, scratch);
    asm volatile("s_waitcnt vmcnt(8)" ::: "memory");
    after = (direct ? 4 : 5) + (prevflag != nullptr ? publish(i - 1) : 0);
    if (!self) {
      unsigned long long now = __builtin_amdgcn_s_memrealtime();
      while (now < tnext) { __builtin_amdgcn_s_sleep(1); now = __builtin_amdgcn_s_memrealtime(); }
      tnext = now + (direct ? PACE_D : PACE_P);
    }
    if (direct) {
      store4(fstage + stage_off(p, cs) + u * 4096, voff, fr);
      prevflag = dws + OFF_LFLAG + (size_t)(p.tile * 64 + cs) * 16;
    } else {
      char* dst = pstage + (size_t)((cs * 24 + p.tile) * 4 + u) * PUNIT;
      if (raw) store4(dst, voff, fr); else store_packed(dst, lane, pk);
      if (lane < 8) st16_sys(dst + 4096 + lane * 16, pk.hdr);
      prevflag = dws + OFF_PFLAG + (size_t)(p.tile * 64 + cs) * 4;
    }
    prevdirect = direct;
    p = pn; s = sn;
  }
  asm volatile("s_waitcnt vmcnt(0)" : "+v"(r[0]) :: "memory");
  publish(i - 1);
  if (u == 0) STAMP(2);
}

__device__ __forceinline__ void decoder(const Args& a, unsigned call, int p, int j, int lane, LAS char* scratch) {
  char* myws = a.ws[a.rank];
  const char* pstage = myws + OFF_PSTAGE + (size_t)(call & 1u) * PPAR;
  const int u = j & 3;
  for (int k = 0; k < 12; ++k) {
    const Piece pc = decode((j >> 2) + 16 * k);
    if (is_direct(pc)) continue;
    const int cs = p * 8 + pc.kc;
    const char* flag = myws + OFF_PFLAG + (size_t)(pc.tile * 64 + cs) * 4;
    {
      const unsigned long long t0 = __builtin_amdgcn_s_memrealtime();
      bool bad = false;
      for (;;) {
        const unsigned fv = ld_sys(flag), tm = ld_agent(myws + OFF_TMO);
        if (__builtin_amdgcn_readfirstlane(fv) >= call) break;
        if (__builtin_amdgcn_readfirstlane(tm) == call || __builtin_amdgcn_s_memrealtime() - t0 > DEADLINE) {
          if (lane == 0) st_agent(myws + OFF_TMO, call);
          bad = true;
          break;
        }
        __builtin_amdgcn_s_sleep(2);
      }
      if (bad) return;
    }
    const char* src = pstage + (size_t)((cs * 24 + pc.tile) * 4 + u) * PUNIT;
    Packed pk; unsigned h;
    asm volatile("s_nop 4\n\t"
                 "global_load_dword %4, %7, %8 offset:0 sc0 sc1\n\t"
                 "global_load_dwordx4 %2, %5, %8 offset:2048 sc0 sc1\n\t"
                 "global_load_dwordx2 %3, %6, %8 offset:3072 sc0 sc1\n\t"
                 "global_load_dwordx4 %0, %5, %8 sc0 sc1\n\t"
                 "global_load_dwordx4 %1, %5, %8 offset:1024 sc0 sc1\n\t"
                 "s_waitcnt vmcnt(0)"
                 : "=&v"(pk.l0), "=&v"(pk.l1), "=&v"(pk.c0), "=&v"(pk.c1), "=&v"(h)
                 : "v"(lane * 16), "v"(lane * 8), "v"(4096 + (lane & 31) * 4), "s"(src) : "memory");
    u32x4 fr[4];
    if (__builtin_amdgcn_readfirstlane(h) & 0x80u) {
      asm volatile("global_load_dwordx4 %0, %4, %5 sc0 sc1\n\t"
                   "global_load_dwordx4 %1, %4, %5 offset:1024 sc0 sc1\n\t"
                   "global_load_dwordx4 %2, %4, %5 offset:2048 sc0 sc1\n\t"
                   "global_load_dwordx4 %3, %4, %5 offset:3072 sc0 sc1\n\t"
                   "s_waitcnt vmcnt(0)"
                   : "=&v"(fr[0]), "=&v"(fr[1]), "=&v"(fr[2]), "=&v"(fr[3]) : "v"(lane * 16), "s"(src) : "memory");
    } else {
      pk_decode(pk, h, fr, lane, scratch);
    }
    char* dst = myws + OFF_STAGE + (size_t)(call & 1u) * FPAR + stage_off(pc, cs);
    store4(dst + u * 4096, lane * 16, fr);
    asm volatile("s_waitcnt vmcnt(0)" ::: "memory");
    if (lane == 0) st_agent(myws + OFF_LFLAG + (size_t)((pc.tile * 64 + cs) * 4 + u) * 4, call);
  }
}

__device__ __forceinline__ void expander(const Args& a, unsigned call, int ti, int part, int half, int lane,
                                         LAS char* buf, LAS unsigned* sync) {
  char* myws = a.ws[a.rank];
  const int slot = ti * 4 + part;
  const char* flag = myws + OFF_OFLAG + (size_t)slot * 4;
  bool bad = false;
  {
    const unsigned long long t0 = __builtin_amdgcn_s_memrealtime();
    for (;;) {
      const unsigned fv = ld_sys(flag), tm = ld_agent(myws + OFF_TMO);
      if (__builtin_amdgcn_readfirstlane(fv) >= call) break;
      if (__builtin_amdgcn_readfirstlane(tm) == call || __builtin_amdgcn_s_memrealtime() - t0 > DEADLINE) {
        if (lane == 0) st_agent(myws + OFF_TMO, call);
        bad = true;
        break;
      }
      __builtin_amdgcn_s_sleep(1);
    }
  }
  if (half == 0) STAMP(3 + part * 2);
  const char* src = myws + OFF_CSLOT + (size_t)slot * CSLOT + lane * 16;
  if (!bad) {
#pragma unroll
    for (int i = 0; i < 5; ++i)
      __builtin_amdgcn_global_load_lds((const GAS void*)(src + (half * 5 + i) * 1024), (LAS void*)(buf + (half * 5 + i) * 1024), 16, 0, 17);
    if (half == 0) __builtin_amdgcn_global_load_lds((const GAS void*)(src + 16384), (LAS void*)(buf + 16384), 16, 0, 17);
  }
  const unsigned tm = __builtin_amdgcn_readfirstlane(ld_agent(myws + OFF_TMO));
  asm volatile("s_waitcnt vmcnt(0)" ::: "memory");
  bad = bad || tm == call;
  if (lane == 0) __hip_atomic_store(sync + half, bad ? 2u : 1u, __ATOMIC_RELEASE, __HIP_MEMORY_SCOPE_WORKGROUP);
  {
    unsigned o;
    const unsigned long long t0 = __builtin_amdgcn_s_memrealtime();
    while ((o = __builtin_amdgcn_readfirstlane(__hip_atomic_load(sync + (half ^ 1), __ATOMIC_ACQUIRE, __HIP_MEMORY_SCOPE_WORKGROUP))) == 0u) {
      if (__builtin_amdgcn_s_memrealtime() - t0 > 2 * DEADLINE) { o = 2u; break; }
    }
    bad = bad || o == 2u;
  }
  float* o = a.out[a.rank] + (size_t)((ti >> 4) * 128 + part * 32 + half * 16 + 4 * (lane >> 5)) * NCOL +
             ((a.rank >> 2) ^ 1) * 2048 + (ti & 15) * 128 + 4 * (lane & 31);
  if (bad) {
    for (int e = 0; e < 8; ++e)
      st16_sys(o + (size_t)((e & 3) + 8 * (e >> 2)) * NCOL, (u32x4){0x7fc00000u, 0x7fc00000u, 0x7fc00000u, 0x7fc00000u});
    return;
  }
  const u32x2 m = *(const LAS u32x2*)(buf + 16384 + lane * 8);
  const u32x2 cnt = *(const LAS u32x2*)(buf + 16896);
  const unsigned n0 = __builtin_amdgcn_readfirstlane(cnt[0]), total = __builtin_amdgcn_readfirstlane(cnt[1]);
  if (total > 2560u) {
#pragma unroll
    for (int i = 10; i < 16; ++i)
      __builtin_amdgcn_global_load_lds((const GAS void*)(src + i * 1024), (LAS void*)(buf + i * 1024), 16, 0, 17);
    asm volatile("s_waitcnt vmcnt(0)" ::: "memory");
  }
  unsigned base = half ? n0 : 0u;
  unsigned bits = half ? m[0] : m[1];
#pragma unroll
  for (int e = 0; e < 8; ++e) {
    f32x4 v;
#pragma unroll
    for (int c = 0; c < 4; ++c) {
      const bool nz = (int)bits < 0;
      bits <<= 1;
      const unsigned long long mm = __ballot(nz);
      unsigned pos = __builtin_amdgcn_mbcnt_hi((unsigned)(mm >> 32), __builtin_amdgcn_mbcnt_lo((unsigned)mm, base)) & 4095u;
      asm volatile("" : "+v"(pos));
      const float x = *(const LAS float*)(buf + pos * 4u);
      v[c] = nz ? x : 0.f;
      base += (unsigned)__builtin_popcountll(mm);
    }
    st16_sys(o + (size_t)((e & 3) + 8 * (e >> 2)) * NCOL, __builtin_bit_cast(u32x4, v));
  }
  if (half == 0) STAMP(4 + part * 2);
}

__global__ void __launch_bounds__(THREADS) gemm_rs_kernel(Args a) {
  __shared__ __attribute__((aligned(16))) char lds[LDS_CTRL + 512];
  const int tid = threadIdx.x, lane = tid & 63;
  const int wv = __builtin_amdgcn_readfirstlane(tid >> 6);
  const int b = blockIdx.x, xcd = b & 7, jb = b >> 3;
  char* myws = a.ws[a.rank];
  unsigned call = 0;

  if (jb < 8) {
    call = __builtin_amdgcn_readfirstlane(ld_agent(myws + OFF_EPOCH)) + 1u;
    LAS unsigned* sync = (LAS unsigned*)((LAS char*)lds + LDS_CTRL);
    if (wv == 7 && lane < 56) sync[lane] = 0u;
    asm volatile("s_waitcnt lgkmcnt(0)" ::: "memory");
    BAR();
    if (wv < 4 && jb < NSLOT) producer(a, call, xcd, jb, wv, lane, (LAS char*)lds + wv * PK_LDS, sync + 8);
    BAR();
    expander(a, call, jb * 8 + xcd, wv >> 1, wv & 1, lane, (LAS char*)lds + (wv >> 1) * CSLOT, sync + (wv >> 1) * 2);
  } else if (jb >= 24) {
    call = __builtin_amdgcn_readfirstlane(ld_agent(myws + OFF_EPOCH)) + 1u;
    decoder(a, call, (a.rank + 1 + (jb - 24)) & 7, xcd * 8 + wv, lane, (LAS char*)lds + wv * PK_LDS);
    BAR();
  } else {
    const int jc = jb - 8;
    const int rt = (xcd & 3) + (jc >> 3) * 4;
    const int ct = (xcd >> 2) * 8 + (jc & 7);
    call = __builtin_amdgcn_readfirstlane(ld_agent(myws + OFF_EPOCH)) + 1u;
    LAS unsigned* lds_bad = (LAS unsigned*)((LAS char*)lds + LDS_CTRL);
    LAS unsigned* lds_sched = (LAS unsigned*)((LAS char*)lds + LDS_CTRL + 64);
    const char* stage = myws + OFF_STAGE + (size_t)(call & 1u) * FPAR;
    if (wv == 6) {
      const char* fa = myws + OFF_LFLAG + (size_t)(rt * 64 + lane) * 16;
      const char* fb = myws + OFF_LFLAG + (size_t)((8 + ct) * 64 + lane) * 16;
      const char* ptm = myws + OFF_TMO;
      const char* pa = stage + (size_t)rt * PIECE + lane * 128;
      const char* pb = stage + A_BYTES + (size_t)ct * PIECE + lane * 128;
      LAS char* sink = (LAS char*)lds + LDS_CTRL + 256;
      unsigned long long ready = 0, done = 0, seen = 0;
      STAMP(0);
      for (int c = 0; c < 64; ++c) {
        if ((ready & ~done) == 0ull) {
          const unsigned long long t0 = __builtin_amdgcn_s_memrealtime();
          for (;;) {
            u32x4 va, vb; unsigned tm;
            asm volatile("global_load_dwordx4 %0, %3, off sc0 sc1\n\t"
                         "global_load_dwordx4 %1, %4, off sc0 sc1\n\t"
                         "global_load_dword %2, %5, off sc1\n\t"
                         "s_waitcnt vmcnt(0)"
                         : "=&v"(va), "=&v"(vb), "=&v"(tm) : "v"(fa), "v"(fb), "v"(ptm) : "memory");
            const unsigned lo = min(min(min(va[0], va[1]), min(va[2], va[3])), min(min(vb[0], vb[1]), min(vb[2], vb[3])));
            ready = __ballot(lo >= call);
            if ((ready & ~done) != 0ull) break;
            if (__builtin_amdgcn_readfirstlane(tm) == call || __builtin_amdgcn_s_memrealtime() - t0 > DEADLINE) {
              if (lane == 0) st_agent(myws + OFF_TMO, call);
              ready = ~0ull; seen = ~0ull;
              break;
            }
            __builtin_amdgcn_s_sleep(1);
          }
        }
        const int cs = __builtin_ctzll(ready & ~done);
        done |= 1ull << cs;
        seen |= 1ull << cs;
        if (lane == 0) lds_sched[c & 7] = (unsigned)cs;
        if (c == 0) STAMP(1);
        if (c == 8) STAMP(2);
        if (c == 32) STAMP(3);
        if (c == 56) STAMP(4);
        if (c == 63) STAMP(5);
        if (c == 63) {
          const unsigned t = __builtin_amdgcn_readfirstlane(ld_agent(myws + OFF_TMO));
          if (lane == 0) *lds_bad = (t == call) ? 1u : 0u;
        }
        asm volatile("s_waitcnt lgkmcnt(0)" ::: "memory");
        BAR();
        unsigned long long fresh = ready & ~seen;
        seen |= fresh;
        while (fresh) {
          const int k = __builtin_ctzll(fresh);
          fresh &= fresh - 1;
          const char* ga = pa + (size_t)k * (8 * PIECE);
          const char* gb = pb + (size_t)k * (16 * PIECE);
          __builtin_amdgcn_global_load_lds((const GAS void*)ga, (LAS void*)sink, 4, 0, 16);
          __builtin_amdgcn_global_load_lds((const GAS void*)(ga + 8192), (LAS void*)sink, 4, 0, 16);
          __builtin_amdgcn_global_load_lds((const GAS void*)gb, (LAS void*)sink, 4, 0, 16);
          __builtin_amdgcn_global_load_lds((const GAS void*)(gb + 8192), (LAS void*)sink, 4, 0, 16);
        }
      }
      for (int i = 0; i <= DIST; ++i) BAR();
      asm volatile("s_waitcnt vmcnt(0)" ::: "memory");
    } else if (wv == 7) {
    } else if (wv >= 4) {
      const int lw = wv - 4;
      const char* gbase = stage + A_BYTES + (size_t)ct * PIECE + lw * 8192 + lane * 16;
      BAR();
      int st = 0;
      for (int c = 0; c < 64 + DIST; ++c) {
        if (c < 64) {
          const int cs = __builtin_amdgcn_readfirstlane(*(volatile LAS unsigned*)(lds_sched + (c & 7))) & 63;
          const char* ga = gbase + (size_t)cs * (16 * PIECE);
          LAS char* la = (LAS char*)lds + st * 16384 + lw * 8192;
#pragma unroll
          for (int f = 0; f < 8; ++f)
            __builtin_amdgcn_global_load_lds((const GAS void*)(ga + f * 1024), (LAS void*)(la + f * 1024), 16, 0, 17);
          asm volatile("s_waitcnt vmcnt(16)" ::: "memory");
        } else if (c == 64) {
          asm volatile("s_waitcnt vmcnt(8)" ::: "memory");
        } else {
          asm volatile("s_waitcnt vmcnt(0)" ::: "memory");
        }
        st = st == NSTAGE - 1 ? 0 : st + 1;
        BAR();
      }
    } else {
      f32x16 acc[4];
#pragma unroll
      for (int c = 0; c < 4; ++c)
#pragma unroll
        for (int e = 0; e < 16; ++e) acc[c][e] = 0.f;
      const __amdgpu_buffer_rsrc_t rsa = __builtin_amdgcn_make_buffer_rsrc((void*)(stage + (size_t)rt * PIECE), 0, (int)A_BYTES, 0x00020000);
      const unsigned aoff = wv * 4096 + lane * 16;
      h8 ar[NSTAGE][4];
      h8 bf[2][4];
#define SCHED_READ(C) (*(volatile LAS unsigned*)(lds_sched + ((C) & 7)))
#define LOAD_A(SLOT, SV) do {                                                                            \
        const unsigned cs_ = __builtin_amdgcn_readfirstlane(SV) & 63u;                                   \
        _Pragma("unroll") for (int s_ = 0; s_ < 4; ++s_)                                                 \
          ar[SLOT][s_] = __builtin_bit_cast(h8, __builtin_amdgcn_raw_buffer_load_b128(rsa, aoff + s_ * 1024, cs_ * (unsigned)(8 * PIECE), 17)); \
      } while (0)
#define READ_B(BUF, ST, S) do {                                                                          \
        _Pragma("unroll") for (int c_ = 0; c_ < 4; ++c_)                                                 \
          bf[BUF][c_] = *(const LAS h8*)((const LAS char*)lds + (ST) * 16384 + (S) * 4096 + c_ * 1024 + lane * 16); \
      } while (0)
#define MMA4(SLOT, S) do {                                                                               \
        __builtin_amdgcn_sched_barrier(0);                                                               \
        _Pragma("unroll") for (int c_ = 0; c_ < 4; ++c_)                                                 \
          acc[c_] = __builtin_amdgcn_mfma_f32_32x32x16_f16(ar[SLOT][S], bf[(S) & 1][c_], acc[c_], 0, 0, 0); \
        __builtin_amdgcn_sched_barrier(0);                                                               \
      } while (0)
#define STEP(K, T, MORE) do {                                                                            \
        READ_B(1, K, 1);                                                                                 \
        unsigned sv_ = 0; if (MORE) sv_ = SCHED_READ((T) + DIST);                                        \
        MMA4(K, 0);                                                                                      \
        READ_B(0, K, 2);                                                                                 \
        if (MORE) LOAD_A(((K) + DIST) % NSTAGE, sv_);                                                    \
        MMA4(K, 1);                                                                                      \
        READ_B(1, K, 3);                                                                                 \
        MMA4(K, 2);                                                                                      \
        READ_B(0, ((K) + 1) % NSTAGE, 0);                                                                \
        MMA4(K, 3);                                                                                      \
        asm volatile("s_waitcnt lgkmcnt(4)" ::: "memory");        \
        BAR();                                                                                           \
      } while (0)
      BAR(); LOAD_A(0, SCHED_READ(0)); BAR(); LOAD_A(1, SCHED_READ(1)); BAR(); LOAD_A(2, SCHED_READ(2));
      BAR(); LOAD_A(3, SCHED_READ(3)); BAR();
      READ_B(0, 0, 0);
      for (int t = 0; t < 60; t += 6) {
        STEP(0, t, true); STEP(1, t + 1, true); STEP(2, t + 2, true);
        STEP(3, t + 3, true); STEP(4, t + 4, true); STEP(5, t + 5, true);
      }
      STEP(0, 60, false); STEP(1, 61, false); STEP(2, 62, false); STEP(3, 63, false);
#undef SCHED_READ
#undef LOAD_A
#undef READ_B
#undef MMA4
#undef STEP
      if (wv == 0) STAMP(6);
      const bool bad = *lds_bad != 0u;
      const int g = a.rank >> 2;
      if (rt < 4) {
        const float qnan = __builtin_nanf("");
        float* o = a.out[a.rank] + (size_t)(rt * 128 + wv * 32 + 4 * (lane >> 5)) * NCOL + g * 2048 + ct * 128 + 4 * (lane & 31);
#pragma unroll
        for (int e = 0; e < 16; ++e) {
          f32x4 v;
#pragma unroll
          for (int c = 0; c < 4; ++c) { const float y = acc[c][e]; v[c] = bad ? qnan : (y < 0.f ? 0.f : y); }
          st16_sys(o + (size_t)((e & 3) + 8 * (e >> 2)) * NCOL, __builtin_bit_cast(u32x4, v));
        }
      } else {
        LAS char* buf = (LAS char*)lds + wv * CSLOT;
        const int q = ((g ^ 1) << 2) + (rt - 4);
        const int slot = ((a.rank & 3) * 16 + ct) * 4 + wv;
        char* dst = a.ws[q & 7] + OFF_CSLOT + (size_t)slot * CSLOT + lane * 16;
        if (bad) {
          const unsigned nb = 0x7fc00000u;
          for (int i = 0; i < 16; ++i) st16_sys(dst + i * 1024, (u32x4){nb, nb, nb, nb});
          if (lane < 32) st16_sys(dst + 16384, (u32x4){~0u, ~0u, ~0u, ~0u});
          if (lane == 32) st16_sys(dst + 16384, (u32x4){2048u, 4096u, 0u, 0u});
        } else {
          unsigned base = 0, sent = 0, n0 = 0, mhi = 0, mlo = 0;
#pragma unroll
          for (int e4 = 0; e4 < 4; ++e4) {
#pragma unroll
            for (int ee = 0; ee < 4; ++ee) {
#pragma unroll
              for (int c = 0; c < 4; ++c) {
                const float y = acc[c][e4 * 4 + ee];
                const bool nz = !(y <= 0.f);
                const unsigned long long m = __ballot(nz);
                unsigned pos = __builtin_amdgcn_mbcnt_hi((unsigned)(m >> 32), __builtin_amdgcn_mbcnt_lo((unsigned)m, base));
                asm volatile("" : "+v"(pos));
                *(LAS float*)(buf + 4u * (nz ? pos : 4288u + lane)) = y;
                if (e4 < 2) mhi = mhi * 2u + (nz ? 1u : 0u); else mlo = mlo * 2u + (nz ? 1u : 0u);
                base += (unsigned)__builtin_popcountll(m);
              }
            }
            if (e4 == 1) n0 = base;
            asm volatile("s_waitcnt vmcnt(0)" ::: "memory");
            for (; sent + 256u <= base; sent += 256u)
              st16_sys(dst + sent * 4u, *(const LAS u32x4*)(buf + sent * 4u + lane * 16));
          }
          if (sent + lane * 4u < base) st16_sys(dst + sent * 4u, *(const LAS u32x4*)(buf + sent * 4u + lane * 16));
          *(LAS u32x2*)(buf + 16384 + lane * 8) = (u32x2){mlo, mhi};
          if (lane == 0) *(LAS u32x2*)(buf + 16896) = (u32x2){n0, base};
          if (lane < 33) st16_sys(dst + 16384, *(const LAS u32x4*)(buf + 16384 + lane * 16));
        }
        if (wv == 0) STAMP(10);
        asm volatile("s_waitcnt vmcnt(0)" ::: "memory");
        if (lane == 0) st_sys(a.ws[q & 7] + OFF_OFLAG + (size_t)slot * 4, call);
      }
      if (wv == 0) STAMP(7);
    }
  }
  if (tid == 0) {
    const unsigned t = __hip_atomic_fetch_add((unsigned*)(myws + OFF_ENDCNT), 1u, __ATOMIC_RELAXED, __HIP_MEMORY_SCOPE_AGENT);
    if (t % GRID == GRID - 1) st_agent(myws + OFF_EPOCH, call);
  }
}

}

extern "C" void kernel_launch(int rank, int nranks, ncclComm_t comm,
                              hipStream_t stream,
                              void* const* d_in, const int* in_sizes, int n_in,
                              void* d_out, int out_size,
                              const int* in_dtypes, int out_dtype,
                              void* const* d_out_all,
                              void* const* d_ws_all, size_t ws_bytes) {
  (void)comm; (void)in_dtypes; (void)out_dtype; (void)d_out;
  if (nranks != NR || (unsigned)rank >= (unsigned)NR || n_in < 2 || ws_bytes < WS_NEED ||
      in_sizes[0] != 4096 * KSH || in_sizes[1] != KSH * NCOL || out_size != 512 * NCOL) {
    fprintf(stderr, "gemm_rs: unexpected problem shape\n");
    return;
  }
  Args a;
  for (int p = 0; p < NR; ++p) { a.ws[p] = (char*)d_ws_all[p]; a.out[p] = (float*)d_out_all[p]; }
  a.x = (const float*)d_in[0];
  a.w = (const float*)d_in[1];
  a.rank = rank;
  a.pad = 0;
  gemm_rs_kernel<<<GRID, THREADS, 0, stream>>>(a);
  hipError_t e = hipGetLastError();
  if (e != hipSuccess) fprintf(stderr, "gemm_rs: launch failed: %s\n", hipGetErrorString(e));
}
```

```cpp
#include <hip/hip_runtime.h>
#include <rccl/rccl.h>
#include <cstdio>

namespace {

constexpr int NR = 8;
constexpr int NCOL = 4096;
constexpr int KSH = 512;
#define LATE_V 3
constexpr int LATE = LATE_V;
#define KD_V 7
#define NSLOT_V 7
constexpr int NSLOT = NSLOT_V;
#define PACE_V 200
constexpr unsigned PACE_P = PACE_V * NSLOT / 8, PACE_D = PACE_P * 4096 / 3712;
constexpr int KD = KD_V;

constexpr size_t OFF_EPOCH = 0;
constexpr size_t OFF_ENDCNT = 128;
constexpr size_t OFF_TMO = 256;
constexpr size_t OFF_OFLAG = 1024;
constexpr size_t OFF_QCNT = 2048;
constexpr size_t OFF_PFLAG = 4096;
constexpr size_t OFF_LFLAG = 16384;
constexpr size_t OFF_STAGE = 1u << 20;
constexpr size_t PIECE = 16384;
constexpr size_t A_BYTES = 64 * 8 * PIECE;
constexpr size_t B_BYTES = 64 * 16 * PIECE;
constexpr size_t PUNIT = 4224;
constexpr size_t PPAR = 64 * 24 * 4 * PUNIT;
constexpr size_t FPAR = A_BYTES + B_BYTES;
constexpr size_t OFF_PSTAGE = OFF_STAGE + 2 * FPAR;
constexpr size_t CSLOT = 17408;
constexpr size_t OFF_CSLOT = OFF_PSTAGE + 2 * PPAR;
constexpr size_t WS_NEED = OFF_CSLOT + 256 * CSLOT;

constexpr int NPROD = 64, NCONS = 128, NDEC = 56, GRID = NPROD + NCONS + NDEC;
constexpr int THREADS = 512;
constexpr int NSTAGE = 6;
constexpr int DIST = 4;
constexpr int LDS_CTRL = 6 * 16384;
constexpr unsigned long long DEADLINE = 30000000ull;

typedef unsigned u32x4 __attribute__((ext_vector_type(4)));
typedef unsigned u32x2 __attribute__((ext_vector_type(2)));
typedef float f32x4 __attribute__((ext_vector_type(4)));
typedef float f32x16 __attribute__((ext_vector_type(16)));
typedef _Float16 h2 __attribute__((ext_vector_type(2)));
typedef _Float16 h8 __attribute__((ext_vector_type(8)));
#define LAS __attribute__((address_space(3)))
#define GAS __attribute__((address_space(1)))

struct Args {
  char* ws[NR];
  float* out[NR];
  const float* x;
  const float* w;
  int rank;
  int pad;
};

__device__ __forceinline__ unsigned ld_sys(const void* p) {
  return __hip_atomic_load((const unsigned*)p, __ATOMIC_RELAXED, __HIP_MEMORY_SCOPE_SYSTEM);
}
__device__ __forceinline__ unsigned ld_agent(const void* p) {
  return __hip_atomic_load((const unsigned*)p, __ATOMIC_RELAXED, __HIP_MEMORY_SCOPE_AGENT);
}
__device__ __forceinline__ void st_sys(void* p, unsigned v) {
  __hip_atomic_store((unsigned*)p, v, __ATOMIC_RELAXED, __HIP_MEMORY_SCOPE_SYSTEM);
}
__device__ __forceinline__ void st_agent(void* p, unsigned v) {
  __hip_atomic_store((unsigned*)p, v, __ATOMIC_RELAXED, __HIP_MEMORY_SCOPE_AGENT);
}
__device__ __forceinline__ void st16_sys(void* p, u32x4 v) {
  asm volatile("global_store_dwordx4 %0, %1, off sc0 sc1\n\ts_nop 1" :: "v"(p), "v"(v) : "memory");
}
__device__ __forceinline__ unsigned pk(float a, float b) {
  h2 h; h.x = (_Float16)a; h.y = (_Float16)b; return __builtin_bit_cast(unsigned, h);
}
#define STAMP(i) do {} while (0)
#define BAR() do { asm volatile("" ::: "memory"); __builtin_amdgcn_s_barrier(); asm volatile("" ::: "memory"); } while (0)

typedef unsigned short us2 __attribute__((ext_vector_type(2)));
constexpr int PK_UNIT = (int)PUNIT;
constexpr int PK_LDS = 4096 + 128;
struct Packed { u32x4 l0, l1, c0; u32x2 c1; u32x4 hdr; };

__device__ __forceinline__ unsigned pk_sub(unsigned a, unsigned b) { return __builtin_bit_cast(unsigned, __builtin_bit_cast(us2, a) - __builtin_bit_cast(us2, b)); }
__device__ __forceinline__ unsigned pk_add(unsigned a, unsigned b) { return __builtin_bit_cast(unsigned, __builtin_bit_cast(us2, a) + __builtin_bit_cast(us2, b)); }
__device__ __forceinline__ unsigned pk_min(unsigned a, unsigned b) { return __builtin_bit_cast(unsigned, __builtin_elementwise_min(__builtin_bit_cast(us2, a), __builtin_bit_cast(us2, b))); }
template <int CTRL, int RM> __device__ __forceinline__ unsigned dpp_add(unsigned x) {
  return x + (unsigned)__builtin_amdgcn_update_dpp(0, (int)x, CTRL, RM, 0xf, false);
}
__device__ __forceinline__ unsigned wave_scan(unsigned x) {
  x = dpp_add<0x111, 0xf>(x); x = dpp_add<0x112, 0xf>(x); x = dpp_add<0x114, 0xf>(x); x = dpp_add<0x118, 0xf>(x);
  x = dpp_add<0x142, 0xa>(x); x = dpp_add<0x143, 0xc>(x);
  return x;
}
__device__ __forceinline__ unsigned esc_addr(unsigned b, int lane) {
  const unsigned j = 15u - (b & 15u);
  return (j >> 2) * 1024u + lane * 16u + (j & 3u) * 4u + (b >> 4) * 2u + 1u;
}

__device__ __forceinline__ bool pk_encode(const u32x4 (&fr)[4], Packed& p, int lane, LAS char* lds) {
  unsigned d[16];
#pragma unroll
  for (int j = 0; j < 16; ++j) d[j] = fr[j >> 2][j & 3];
  unsigned s = 0;
#pragma unroll
  for (int j = 0; j < 16; j += 5) s += ((d[j] >> 10) & 31u) + ((d[j] >> 26) & 31u);
  s = __builtin_amdgcn_readlane(wave_scan(s), 63);
  int top = (int)((s + 1388u) >> 9);
  int base = top - 6; base = base < 0 ? 0 : (base > 24 ? 24 : base);
  const unsigned base2 = (unsigned)base * 0x00010001u;
  unsigned q[16], acc = 0;
#pragma unroll
  for (int j = 0; j < 16; ++j) {
    const unsigned x = d[j] >> 10;
    const unsigned c = pk_min(pk_sub(x & 0x001f001fu, base2), 0x00070007u);
    const unsigned c6 = (x & 0x00200020u) | ((d[j] >> 5) & 0x00180018u) | c;
    q[j] = (c6 & 0x3fu) | (c6 >> 10);
    acc = pk_add(pk_add(acc, acc), (pk_add(c, 0x00010001u) >> 3) & 0x00010001u);
  }
#pragma unroll
  for (int i = 0; i < 4; ++i) {
    p.l0[i] = __builtin_amdgcn_perm(d[2 * i + 1], d[2 * i], 0x06040200u);
    p.l1[i] = __builtin_amdgcn_perm(d[2 * i + 9], d[2 * i + 8], 0x06040200u);
  }
  p.c0[0] = q[0] | (q[1] << 12) | (q[2] << 24);
  p.c0[1] = (q[2] >> 8) | (q[3] << 4) | (q[4] << 16) | (q[5] << 28);
  p.c0[2] = (q[5] >> 4) | (q[6] << 8) | (q[7] << 20);
  p.c0[3] = q[8] | (q[9] << 12) | (q[10] << 24);
  p.c1[0] = (q[10] >> 8) | (q[11] << 4) | (q[12] << 16) | (q[13] << 28);
  p.c1[1] = (q[13] >> 4) | (q[14] << 8) | (q[15] << 20);
  const unsigned cnt = (unsigned)__builtin_popcount(acc);
  const unsigned incl = wave_scan(cnt);
  const unsigned total = __builtin_amdgcn_readlane(incl, 63);
  if (total > 127u) { p.hdr = (u32x4){0x80u, 0u, 0u, 0u}; return true; }
  LAS char* list = lds + 4096;
  if (total != 0u) {
#pragma unroll
    for (int f = 0; f < 4; ++f) *(LAS u32x4*)(lds + f * 1024 + lane * 16) = fr[f];
    unsigned off = 1u + incl - cnt;
    while (acc != 0u) {
      const unsigned b = (unsigned)__builtin_ctz(acc);
      acc &= acc - 1u;
      const unsigned hb = *(const LAS unsigned char*)(lds + esc_addr(b, lane));
      *(LAS unsigned char*)(list + off) = (unsigned char)((hb >> 2) & 31u);
      ++off;
    }
  }
  if (lane == 0) *(LAS unsigned char*)list = (unsigned char)base;
  p.hdr = *(const LAS u32x4*)(list + (lane & 7) * 16);
  return false;
}

__device__ __forceinline__ void pk_decode(const Packed& p, unsigned h, u32x4 (&fr)[4], int lane, LAS char* lds) {
  const unsigned base2 = (__builtin_amdgcn_readfirstlane(h) & 31u) * 0x00010001u;
  unsigned q[16];
  q[0] = p.c0[0] & 0xfffu; q[1] = (p.c0[0] >> 12) & 0xfffu; q[2] = __builtin_amdgcn_alignbit(p.c0[1], p.c0[0], 24) & 0xfffu;
  q[3] = (p.c0[1] >> 4) & 0xfffu; q[4] = (p.c0[1] >> 16) & 0xfffu; q[5] = __builtin_amdgcn_alignbit(p.c0[2], p.c0[1], 28) & 0xfffu;
  q[6] = (p.c0[2] >> 8) & 0xfffu; q[7] = p.c0[2] >> 20;
  q[8] = p.c0[3] & 0xfffu; q[9] = (p.c0[3] >> 12) & 0xfffu; q[10] = __builtin_amdgcn_alignbit(p.c1[0], p.c0[3], 24) & 0xfffu;
  q[11] = (p.c1[0] >> 4) & 0xfffu; q[12] = (p.c1[0] >> 16) & 0xfffu; q[13] = __builtin_amdgcn_alignbit(p.c1[1], p.c1[0], 28) & 0xfffu;
  q[14] = (p.c1[1] >> 8) & 0xfffu; q[15] = p.c1[1] >> 20;
  unsigned acc = 0;
#pragma unroll
  for (int j = 0; j < 16; ++j) {
    const unsigned c6 = (q[j] | (q[j] << 10)) & 0x003f003fu;
    const unsigned c = c6 & 0x00070007u;
    const unsigned hi8 = ((c6 << 2) & 0x00800080u) | ((c6 >> 3) & 0x00030003u) | (pk_add(c, base2) << 2);
    const unsigned l = (j < 8 ? p.l0 : p.l1)[(j & 7) >> 1];
    fr[j >> 2][j & 3] = __builtin_amdgcn_perm(hi8, l, (j & 1) ? 0x06030402u : 0x06010400u);
    acc = pk_add(pk_add(acc, acc), (pk_add(c, 0x00010001u) >> 3) & 0x00010001u);
  }
  const unsigned cnt = (unsigned)__builtin_popcount(acc);
  const unsigned incl = wave_scan(cnt);
  if (__builtin_amdgcn_readlane(incl, 63) != 0u) {
    LAS char* list = lds + 4096;
    if (lane < 32) *(LAS unsigned*)(list + lane * 4) = h;
#pragma unroll
    for (int f = 0; f < 4; ++f) *(LAS u32x4*)(lds + f * 1024 + lane * 16) = fr[f];
    unsigned off = 1u + incl - cnt;
    while (acc != 0u) {
      const unsigned b = (unsigned)__builtin_ctz(acc);
      acc &= acc - 1u;
      LAS unsigned char* t = (LAS unsigned char*)(lds + esc_addr(b, lane));
      const unsigned e = *(const LAS unsigned char*)(list + off);
      *t = (unsigned char)((*t & 0x83u) | (e << 2));
      ++off;
    }
#pragma unroll
    for (int f = 0; f < 4; ++f) fr[f] = *(const LAS u32x4*)(lds + f * 1024 + lane * 16);
  }
}

struct Piece { int kc, tile; };
__device__ __forceinline__ Piece decode(int n) {
  Piece p;
  constexpr int nf = LATE * 24, n2 = nf + (8 - LATE) * 20;
  int i;
  if (n < nf) { p.kc = n / 24; i = n % 24; }
  else if (n < n2) { const int v = n - nf; p.kc = LATE + v / 20; i = v % 20; }
  else { const int v = n - n2; p.kc = LATE + v / 4; i = 20 + v % 4; }
  p.tile = i < 16 ? 8 + i : (i < 20 ? i - 12 : i - 20);
  return p;
}
#define FIRSTD_V 7
__device__ __forceinline__ bool is_direct(const Piece& p, int n) { return n < FIRSTD_V || p.kc >= KD || (p.tile < 4 && p.kc >= LATE); }
__device__ __forceinline__ size_t stage_off(const Piece& p, int cs) {
  return p.tile >= 8 ? A_BYTES + ((size_t)cs * 16 + (p.tile - 8)) * PIECE : ((size_t)cs * 8 + p.tile) * PIECE;
}
struct Src { const char* base; unsigned ustep, loff, js; };
__device__ __forceinline__ Src src_of(const Args& a, int d, const Piece& p, int lane) {
  Src s;
  const unsigned i = lane & 31, h = lane >> 5;
  if (p.tile >= 8) {
    s.base = (const char*)a.w + ((size_t)(p.kc * 64) * NCOL + (d >> 2) * 2048 + (p.tile - 8) * 128) * 4;
    s.ustep = NCOL * 4; s.loff = h * (4 * NCOL * 4) + i * 16; s.js = 8 * NCOL * 4;
  } else {
    const int row0 = p.tile < 4 ? d * 512 + p.tile * 128 : (((d >> 2) ^ 1) * 4 + (p.tile - 4)) * 512 + (d & 3) * 128;
    s.base = (const char*)a.x + ((size_t)row0 * KSH + p.kc * 64) * 4;
    s.ustep = 32 * KSH * 4; s.loff = i * (KSH * 4) + h * 16; s.js = 32;
  }
  return s;
}
__device__ __forceinline__ void load8(f32x4 (&r)[8], const char* base, unsigned loff, unsigned js) {
  unsigned o[8];
#pragma unroll
  for (int j = 0; j < 8; ++j) o[j] = loff + j * js;
  asm volatile(
      "s_nop 4\n\t"
      "global_load_dwordx4 %0, %8, %16\n\t"
      "global_load_dwordx4 %1, %9, %16\n\t"
      "global_load_dwordx4 %2, %10, %16\n\t"
      "global_load_dwordx4 %3, %11, %16\n\t"
      "global_load_dwordx4 %4, %12, %16\n\t"
      "global_load_dwordx4 %5, %13, %16\n\t"
      "global_load_dwordx4 %6, %14, %16\n\t"
      "global_load_dwordx4 %7, %15, %16"
      : "=&v"(r[0]), "=&v"(r[1]), "=&v"(r[2]), "=&v"(r[3]), "=&v"(r[4]), "=&v"(r[5]), "=&v"(r[6]), "=&v"(r[7])
      : "v"(o[0]), "v"(o[1]), "v"(o[2]), "v"(o[3]), "v"(o[4]), "v"(o[5]), "v"(o[6]), "v"(o[7]), "s"(base)
      : "memory");
}
#define WAIT_R(N, r) asm volatile("s_waitcnt vmcnt(" #N ")" : "+v"(r[0]), "+v"(r[1]), "+v"(r[2]), "+v"(r[3]), \
    "+v"(r[4]), "+v"(r[5]), "+v"(r[6]), "+v"(r[7]) :: "memory")
__device__ __forceinline__ void cvt4(const f32x4 (&r)[8], u32x4 (&fr)[4]) {
#pragma unroll
  for (int c = 0; c < 4; ++c)
    fr[c] = (u32x4){pk(r[0][c], r[1][c]), pk(r[2][c], r[3][c]), pk(r[4][c], r[5][c]), pk(r[6][c], r[7][c])};
}
__device__ __forceinline__ void store4(char* dbase, unsigned voff, const u32x4 (&fr)[4]) {
  asm volatile(
      "s_nop 4\n\t"
      "global_store_dwordx4 %0, %1, %5 sc0 sc1\n\t"
      "global_store_dwordx4 %0, %2, %5 offset:1024 sc0 sc1\n\t"
      "global_store_dwordx4 %0, %3, %5 offset:2048 sc0 sc1\n\t"
      "global_store_dwordx4 %0, %4, %5 offset:3072 sc0 sc1\n\t"
      "s_nop 1"
      :: "v"(voff), "v"(fr[0]), "v"(fr[1]), "v"(fr[2]), "v"(fr[3]), "s"(dbase) : "memory");
}
__device__ __forceinline__ void store_packed(char* dbase, int lane, const Packed& p) {
  asm volatile(
      "s_nop 4\n\t"
      "global_store_dwordx4 %0, %2, %6 sc0 sc1\n\t"
      "global_store_dwordx4 %0, %3, %6 offset:1024 sc0 sc1\n\t"
      "global_store_dwordx4 %0, %4, %6 offset:2048 sc0 sc1\n\t"
      "global_store_dwordx2 %1, %5, %6 offset:3072 sc0 sc1\n\t"
      "s_nop 1"
      :: "v"(lane * 16), "v"(lane * 8), "v"(p.l0), "v"(p.l1), "v"(p.c0), "v"(p.c1), "s"(dbase) : "memory");
}

__device__ __forceinline__ void queue_wave(const Args& a, unsigned call, int d, int slot, int lane,
                                           LAS unsigned* seq, LAS unsigned* prog) {
  unsigned* ctr = (unsigned*)(a.ws[a.rank] + OFF_QCNT + (size_t)d * 128);
  for (unsigned i = 0;; ++i) {
    const unsigned cur = i == 0 ? (unsigned)slot : i == 1 ? (unsigned)(slot + NSLOT)
                                : __builtin_amdgcn_readfirstlane(*(volatile LAS unsigned*)(seq + i));
    if (cur >= 192u) break;
    while (__builtin_amdgcn_readfirstlane(*(volatile LAS unsigned*)prog) <= i) __builtin_amdgcn_s_sleep(2);
    if (lane == 0) {
      const unsigned t = __hip_atomic_fetch_add(ctr, 1u, __ATOMIC_RELAXED, __HIP_MEMORY_SCOPE_AGENT) - (call - 1u) * 192u;
      *(volatile LAS unsigned*)(seq + i + 2) = 2u * NSLOT + t;
    }
    asm volatile("s_waitcnt vmcnt(0) lgkmcnt(0)" ::: "memory");
  }
}

__device__ __forceinline__ void producer(const Args& a, unsigned call, int d, int slot, int u, int lane,
                                         LAS char* scratch, LAS unsigned* cnt, LAS unsigned* seq, LAS unsigned* prog) {
  constexpr int NP = 192;
  char* dws = a.ws[d];
  const unsigned voff = lane * 16;
  f32x4 r[8];
  u32x4 fr[4];
  if (u == 0) STAMP(0);
  Piece p = decode(slot);
  Src s = src_of(a, d, p, lane);
  load8(r, s.base + u * s.ustep, s.loff, s.js);
  char* pstage = dws + OFF_PSTAGE + (size_t)(call & 1u) * PPAR;
  char* fstage = dws + OFF_STAGE + (size_t)(call & 1u) * FPAR;
  char* prevflag = nullptr;
  bool prevdirect = false;
  unsigned long long tnext = 0;
  int i = 0;
  auto publish = [&](int idx) -> int {
    unsigned old = 0;
    if (lane == 0) old = __hip_atomic_fetch_add(cnt + idx, 1u, __ATOMIC_RELAXED, __HIP_MEMORY_SCOPE_WORKGROUP);
    if (__builtin_amdgcn_readfirstlane(old) != 3u) return 0;
    if (lane == 0) { if (prevdirect) st16_sys(prevflag, (u32x4){call, call, call, call}); else st_sys(prevflag, call); }
    return 1;
  };
  int after = 4;
  WAIT_R(0, r);
  if (u == 0) STAMP(1);
  int n = slot;
  for (; n < NP; ++i) {
    if (u == 0 && lane == 0) *(volatile LAS unsigned*)prog = (unsigned)i + 1u;
    int nx;
    if (i == 0) nx = slot + NSLOT;
    else {
      unsigned v;
      while ((v = __builtin_amdgcn_readfirstlane(*(volatile LAS unsigned*)(seq + i + 1))) == 0xffffffffu) __builtin_amdgcn_s_sleep(1);
      nx = (int)v;
    }
    const int nn = nx < NP ? nx : n;
    const Piece pn = decode(nn);
    const Src sn = src_of(a, d, pn, lane);
    const int cs = a.rank * 8 + p.kc;
    const bool direct = is_direct(p, n);
    asm volatile("s_waitcnt vmcnt(6)\n\ts_cmp_gt_u32 %8, 5\n\ts_cbranch_scc1 1f\n\ts_waitcnt vmcnt(5)\n\t"
                 "s_cmp_gt_u32 %8, 4\n\ts_cbranch_scc1 1f\n\ts_waitcnt vmcnt(4)\n1:"
                 : "+v"(r[0]), "+v"(r[1]), "+v"(r[2]), "+v"(r[3]), "+v"(r[4]), "+v"(r[5]), "+v"(r[6]), "+v"(r[7])
                 : "s"(__builtin_amdgcn_readfirstlane(after)) : "scc", "memory");
    cvt4(r, fr);
    load8(r, sn.base + u * sn.ustep, sn.loff, sn.js);
    Packed pk;
    bool raw = true;
    if (!direct) raw = pk_encode(fr, pk, lane, scratch);
    asm volatile("s_waitcnt vmcnt(8)" ::: "memory");
    after = (direct ? 4 : 5) + (prevflag != nullptr ? publish(i - 1) : 0);
    {
      unsigned long long now = __builtin_amdgcn_s_memrealtime();
      while (now < tnext) { __builtin_amdgcn_s_sleep(1); now = __builtin_amdgcn_s_memrealtime(); }
      tnext = now + (direct ? PACE_D : PACE_P);
    }
    if (direct) {
      store4(fstage + stage_off(p, cs) + u * 4096, voff, fr);
      prevflag = dws + OFF_LFLAG + (size_t)(p.tile * 64 + cs) * 16;
    } else {
      char* dst = pstage + (size_t)((cs * 24 + p.tile) * 4 + u) * PUNIT;
      if (raw) store4(dst, voff, fr); else store_packed(dst, lane, pk);
      if (lane < 8) st16_sys(dst + 4096 + lane * 16, pk.hdr);
      prevflag = dws + OFF_PFLAG + (size_t)(p.tile * 64 + cs) * 4;
    }
    prevdirect = direct;
    p = pn; s = sn; n = nx;
  }
  asm volatile("s_waitcnt vmcnt(0)" : "+v"(r[0]) :: "memory");
  publish(i - 1);
  if (u == 0) STAMP(2);
}

__device__ __forceinline__ void decoder(const Args& a, unsigned call, int p, int j, int gw, int lane, LAS char* scratch) {
  char* myws = a.ws[a.rank];
  const char* pstage = myws + OFF_PSTAGE + (size_t)(call & 1u) * PPAR;
  const int u = j & 3;
  for (int U = gw; U < 768; U += 448) {
    const Piece pc = decode(U >> 2);
    const Src s = src_of(a, a.rank, pc, lane);
    const int cs = a.rank * 8 + pc.kc;
    f32x4 r[8]; u32x4 fr[4];
    load8(r, s.base + (U & 3) * s.ustep, s.loff, s.js);
    WAIT_R(0, r);
    cvt4(r, fr);
    store4(myws + OFF_STAGE + (size_t)(call & 1u) * FPAR + stage_off(pc, cs) + (U & 3) * 4096, lane * 16, fr);
    asm volatile("s_waitcnt vmcnt(0)" ::: "memory");
    if (lane == 0) st_agent(myws + OFF_LFLAG + (size_t)((pc.tile * 64 + cs) * 4 + (U & 3)) * 4, call);
  }
  for (int k = 0; k < 12; ++k) {
    const Piece pc = decode((j >> 2) + 16 * k);
    if (is_direct(pc, (j >> 2) + 16 * k)) continue;
    const int cs = p * 8 + pc.kc;
    const char* flag = myws + OFF_PFLAG + (size_t)(pc.tile * 64 + cs) * 4;
    {
      const unsigned long long t0 = __builtin_amdgcn_s_memrealtime();
      bool bad = false;
      for (;;) {
        const unsigned fv = ld_sys(flag), tm = ld_agent(myws + OFF_TMO);
        if (__builtin_amdgcn_readfirstlane(fv) >= call) break;
        if (__builtin_amdgcn_readfirstlane(tm) == call || __builtin_amdgcn_s_memrealtime() - t0 > DEADLINE) {
          if (lane == 0) st_agent(myws + OFF_TMO, call);
          bad = true;
          break;
        }
        __builtin_amdgcn_s_sleep(2);
      }
      if (bad) return;
    }
    const char* src = pstage + (size_t)((cs * 24 + pc.tile) * 4 + u) * PUNIT;
    Packed pk; unsigned h;
    asm volatile("s_nop 4\n\t"
                 "global_load_dword %4, %7, %8 offset:0 sc0 sc1\n\t"
                 "global_load_dwordx4 %2, %5, %8 offset:2048 sc0 sc1\n\t"
                 "global_load_dwordx2 %3, %6, %8 offset:3072 sc0 sc1\n\t"
                 "global_load_dwordx4 %0, %5, %8 sc0 sc1\n\t"
                 "global_load_dwordx4 %1, %5, %8 offset:1024 sc0 sc1\n\t"
                 "s_waitcnt vmcnt(0)"
                 : "=&v"(pk.l0), "=&v"(pk.l1), "=&v"(pk.c0), "=&v"(pk.c1), "=&v"(h)
                 : "v"(lane * 16), "v"(lane * 8), "v"(4096 + (lane & 31) * 4), "s"(src) : "memory");
    u32x4 fr[4];
    if (__builtin_amdgcn_readfirstlane(h) & 0x80u) {
      asm volatile("global_load_dwordx4 %0, %4, %5 sc0 sc1\n\t"
                   "global_load_dwordx4 %1, %4, %5 offset:1024 sc0 sc1\n\t"
                   "global_load_dwordx4 %2, %4, %5 offset:2048 sc0 sc1\n\t"
                   "global_load_dwordx4 %3, %4, %5 offset:3072 sc0 sc1\n\t"
                   "s_waitcnt vmcnt(0)"
                   : "=&v"(fr[0]), "=&v"(fr[1]), "=&v"(fr[2]), "=&v"(fr[3]) : "v"(lane * 16), "s"(src) : "memory");
    } else {
      pk_decode(pk, h, fr, lane, scratch);
    }
    char* dst = myws + OFF_STAGE + (size_t)(call & 1u) * FPAR + stage_off(pc, cs);
    store4(dst + u * 4096, lane * 16, fr);
    asm volatile("s_waitcnt vmcnt(0)" ::: "memory");
    if (lane == 0) st_agent(myws + OFF_LFLAG + (size_t)((pc.tile * 64 + cs) * 4 + u) * 4, call);
  }
}

__device__ __forceinline__ void expander(const Args& a, unsigned call, int ti, int part, int half, int lane,
                                         LAS char* buf, LAS unsigned* sync) {
  char* myws = a.ws[a.rank];
  const int slot = ti * 4 + part;
  const char* flag = myws + OFF_OFLAG + (size_t)slot * 4;
  bool bad = false;
  {
    const unsigned long long t0 = __builtin_amdgcn_s_memrealtime();
    for (;;) {
      const unsigned fv = ld_sys(flag), tm = ld_agent(myws + OFF_TMO);
      if (__builtin_amdgcn_readfirstlane(fv) >= call) break;
      if (__builtin_amdgcn_readfirstlane(tm) == call || __builtin_amdgcn_s_memrealtime() - t0 > DEADLINE) {
        if (lane == 0) st_agent(myws + OFF_TMO, call);
        bad = true;
        break;
      }
      __builtin_amdgcn_s_sleep(1);
    }
  }
  if (half == 0) STAMP(3 + part * 2);
  const char* src = myws + OFF_CSLOT + (size_t)slot * CSLOT + lane * 16;
  if (!bad) {
#pragma unroll
    for (int i = 0; i < 5; ++i)
      __builtin_amdgcn_global_load_lds((const GAS void*)(src + (half * 5 + i) * 1024), (LAS void*)(buf + (half * 5 + i) * 1024), 16, 0, 17);
    if (half == 0) __builtin_amdgcn_global_load_lds((const GAS void*)(src + 16384), (LAS void*)(buf + 16384), 16, 0, 17);
  }
  const unsigned tm = __builtin_amdgcn_readfirstlane(ld_agent(myws + OFF_TMO));
  asm volatile("s_waitcnt vmcnt(0)" ::: "memory");
  bad = bad || tm == call;
  if (lane == 0) __hip_atomic_store(sync + half, bad ? 2u : 1u, __ATOMIC_RELEASE, __HIP_MEMORY_SCOPE_WORKGROUP);
  {
    unsigned o;
    const unsigned long long t0 = __builtin_amdgcn_s_memrealtime();
    while ((o = __builtin_amdgcn_readfirstlane(__hip_atomic_load(sync + (half ^ 1), __ATOMIC_ACQUIRE, __HIP_MEMORY_SCOPE_WORKGROUP))) == 0u) {
      if (__builtin_amdgcn_s_memrealtime() - t0 > 2 * DEADLINE) { o = 2u; break; }
    }
    bad = bad || o == 2u;
  }
  float* o = a.out[a.rank] + (size_t)((ti >> 4) * 128 + part * 32 + half * 16 + 4 * (lane >> 5)) * NCOL +
             ((a.rank >> 2) ^ 1) * 2048 + (ti & 15) * 128 + 4 * (lane & 31);
  if (bad) {
    for (int e = 0; e < 8; ++e)
      st16_sys(o + (size_t)((e & 3) + 8 * (e >> 2)) * NCOL, (u32x4){0x7fc00000u, 0x7fc00000u, 0x7fc00000u, 0x7fc00000u});
    return;
  }
  const u32x2 m = *(const LAS u32x2*)(buf + 16384 + lane * 8);
  const u32x2 cnt = *(const LAS u32x2*)(buf + 16896);
  const unsigned n0 = __builtin_amdgcn_readfirstlane(cnt[0]), total = __builtin_amdgcn_readfirstlane(cnt[1]);
  if (total > 2560u) {
#pragma unroll
    for (int i = 10; i < 16; ++i)
      __builtin_amdgcn_global_load_lds((const GAS void*)(src + i * 1024), (LAS void*)(buf + i * 1024), 16, 0, 17);
    asm volatile("s_waitcnt vmcnt(0)" ::: "memory");
  }
  unsigned base = half ? n0 : 0u;
  unsigned bits = half ? m[0] : m[1];
#pragma unroll
  for (int e = 0; e < 8; ++e) {
    f32x4 v;
#pragma unroll
    for (int c = 0; c < 4; ++c) {
      const bool nz = (int)bits < 0;
      bits <<= 1;
      const unsigned long long mm = __ballot(nz);
      unsigned pos = __builtin_amdgcn_mbcnt_hi((unsigned)(mm >> 32), __builtin_amdgcn_mbcnt_lo((unsigned)mm, base)) & 4095u;
      asm volatile("" : "+v"(pos));
      const float x = *(const LAS float*)(buf + pos * 4u);
      v[c] = nz ? x : 0.f;
      base += (unsigned)__builtin_popcountll(mm);
    }
    st16_sys(o + (size_t)((e & 3) + 8 * (e >> 2)) * NCOL, __builtin_bit_cast(u32x4, v));
  }
  if (half == 0) STAMP(4 + part * 2);
}

__global__ void __launch_bounds__(THREADS) gemm_rs_kernel(Args a) {
  __shared__ __attribute__((aligned(16))) char lds[LDS_CTRL + 512];
  const int tid = threadIdx.x, lane = tid & 63;
  const int wv = __builtin_amdgcn_readfirstlane(tid >> 6);
  const int b = blockIdx.x, xcd = b & 7, jb = b >> 3;
  char* myws = a.ws[a.rank];
  unsigned call = 0;

  if (jb < 8) {
    call = __builtin_amdgcn_readfirstlane(ld_agent(myws + OFF_EPOCH)) + 1u;
    LAS unsigned* sync = (LAS unsigned*)((LAS char*)lds + LDS_CTRL);
    LAS unsigned* seq = (LAS unsigned*)((LAS char*)lds + 4 * CSLOT);
    LAS unsigned* cnt = seq + 256;
    if (wv == 7 && lane < 16) sync[lane] = 0u;
    if (wv >= 4) {
#pragma unroll
      for (int q = 0; q < 4; ++q) { seq[(wv - 4) * 64 + lane] = 0xffffffffu; cnt[(wv - 4) * 64 + lane] = 0u; }
    }
    asm volatile("s_waitcnt lgkmcnt(0)" ::: "memory");
    BAR();
    if (jb < NSLOT && xcd != a.rank) {
      if (wv < 4) producer(a, call, xcd, jb, wv, lane, (LAS char*)lds + wv * PK_LDS, cnt, seq, sync + 8);
      else if (wv == 4) queue_wave(a, call, xcd, jb, lane, seq, sync + 8);
    }
    BAR();
    expander(a, call, jb * 8 + xcd, wv >> 1, wv & 1, lane, (LAS char*)lds + (wv >> 1) * CSLOT, sync + (wv >> 1) * 2);
  } else if (jb >= 24) {
    call = __builtin_amdgcn_readfirstlane(ld_agent(myws + OFF_EPOCH)) + 1u;
    decoder(a, call, (a.rank + 1 + (jb - 24)) & 7, xcd * 8 + wv, (jb - 24) * 64 + xcd * 8 + wv, lane, (LAS char*)lds + wv * PK_LDS);
    BAR();
  } else {
    const int jc = jb - 8;
    const int rt = (xcd & 3) + (jc >> 3) * 4;
    const int ct = (xcd >> 2) * 8 + (jc & 7);
    call = __builtin_amdgcn_readfirstlane(ld_agent(myws + OFF_EPOCH)) + 1u;
    LAS unsigned* lds_bad = (LAS unsigned*)((LAS char*)lds + LDS_CTRL);
    LAS unsigned* lds_sched = (LAS unsigned*)((LAS char*)lds + LDS_CTRL + 64);
    const char* stage = myws + OFF_STAGE + (size_t)(call & 1u) * FPAR;
    if (wv == 6) {
      const char* fa = myws + OFF_LFLAG + (size_t)(rt * 64 + lane) * 16;
      const char* fb = myws + OFF_LFLAG + (size_t)((8 + ct) * 64 + lane) * 16;
      const char* ptm = myws + OFF_TMO;
      unsigned long long ready = 0, done = 0;
      STAMP(0);
      for (int c = 0; c < 64; ++c) {
        if ((ready & ~done) == 0ull) {
          const unsigned long long t0 = __builtin_amdgcn_s_memrealtime();
          for (;;) {
            u32x4 va, vb; unsigned tm;
            asm volatile("global_load_dwordx4 %0, %3, off sc0 sc1\n\t"
                         "global_load_dwordx4 %1, %4, off sc0 sc1\n\t"
                         "global_load_dword %2, %5, off sc1\n\t"
                         "s_waitcnt vmcnt(0)"
                         : "=&v"(va), "=&v"(vb), "=&v"(tm) : "v"(fa), "v"(fb), "v"(ptm) : "memory");
            const unsigned lo = min(min(min(va[0], va[1]), min(va[2], va[3])), min(min(vb[0], vb[1]), min(vb[2], vb[3])));
            ready = __ballot(lo >= call);
            if ((ready & ~done) != 0ull) break;
            if (__builtin_amdgcn_readfirstlane(tm) == call || __builtin_amdgcn_s_memrealtime() - t0 > DEADLINE) {
              if (lane == 0) st_agent(myws + OFF_TMO, call);
              ready = ~0ull;
              break;
            }
            __builtin_amdgcn_s_sleep(1);
          }
        }
        const int cs = __builtin_ctzll(ready & ~done);
        done |= 1ull << cs;
        if (lane == 0) lds_sched[c & 7] = (unsigned)cs;
        if (c == 0) STAMP(1);
        if (c == 8) STAMP(2);
        if (c == 32) STAMP(3);
        if (c == 56) STAMP(4);
        if (c == 63) STAMP(5);
        if (c == 63) {
          const unsigned t = __builtin_amdgcn_readfirstlane(ld_agent(myws + OFF_TMO));
          if (lane == 0) *lds_bad = (t == call) ? 1u : 0u;
        }
        asm volatile("s_waitcnt lgkmcnt(0)" ::: "memory");
        BAR();
      }
      for (int i = 0; i <= DIST; ++i) BAR();
      asm volatile("s_waitcnt vmcnt(0)" ::: "memory");
    } else if (wv == 7) {
    } else if (wv >= 4) {
      const int lw = wv - 4;
      const char* gbase = stage + A_BYTES + (size_t)ct * PIECE + lw * 8192 + lane * 16;
      BAR();
      int st = 0;
      for (int c = 0; c < 64 + DIST; ++c) {
        if (c < 64) {
          const int cs = __builtin_amdgcn_readfirstlane(*(volatile LAS unsigned*)(lds_sched + (c & 7))) & 63;
          const char* ga = gbase + (size_t)cs * (16 * PIECE);
          LAS char* la = (LAS char*)lds + st * 16384 + lw * 8192;
#pragma unroll
          for (int f = 0; f < 8; ++f)
            __builtin_amdgcn_global_load_lds((const GAS void*)(ga + f * 1024), (LAS void*)(la + f * 1024), 16, 0, 17);
          asm volatile("s_waitcnt vmcnt(16)" ::: "memory");
        } else if (c == 64) {
          asm volatile("s_waitcnt vmcnt(8)" ::: "memory");
        } else {
          asm volatile("s_waitcnt vmcnt(0)" ::: "memory");
        }
        st = st == NSTAGE - 1 ? 0 : st + 1;
        BAR();
      }
    } else {
      f32x16 acc[4];
#pragma unroll
      for (int c = 0; c < 4; ++c)
#pragma unroll
        for (int e = 0; e < 16; ++e) acc[c][e] = 0.f;
      const __amdgpu_buffer_rsrc_t rsa = __builtin_amdgcn_make_buffer_rsrc((void*)(stage + (size_t)rt * PIECE), 0, (int)A_BYTES, 0x00020000);
      const unsigned aoff = wv * 4096 + lane * 16;
      h8 ar[NSTAGE][4];
      h8 bf[2][4];
#define SCHED_READ(C) (*(volatile LAS unsigned*)(lds_sched + ((C) & 7)))
#define LOAD_A(SLOT, SV) do {                                                                            \
        const unsigned cs_ = __builtin_amdgcn_readfirstlane(SV) & 63u;                                   \
        _Pragma("unroll") for (int s_ = 0; s_ < 4; ++s_)                                                 \
          ar[SLOT][s_] = __builtin_bit_cast(h8, __builtin_amdgcn_raw_buffer_load_b128(rsa, aoff + s_ * 1024, cs_ * (unsigned)(8 * PIECE), 17)); \
      } while (0)
#define READ_B(BUF, ST, S) do {                                                                          \
        _Pragma("unroll") for (int c_ = 0; c_ < 4; ++c_)                                                 \
          bf[BUF][c_] = *(const LAS h8*)((const LAS char*)lds + (ST) * 16384 + (S) * 4096 + c_ * 1024 + lane * 16); \
      } while (0)
#define MMA4(SLOT, S) do {                                                                               \
        __builtin_amdgcn_sched_barrier(0);                                                               \
        _Pragma("unroll") for (int c_ = 0; c_ < 4; ++c_)                                                 \
          acc[c_] = __builtin_amdgcn_mfma_f32_32x32x16_f16(ar[SLOT][S], bf[(S) & 1][c_], acc[c_], 0, 0, 0); \
        __builtin_amdgcn_sched_barrier(0);                                                               \
      } while (0)
#define STEP(K, T, MORE) do {                                                                            \
        READ_B(1, K, 1);                                                                                 \
        unsigned sv_ = 0; if (MORE) sv_ = SCHED_READ((T) + DIST);                                        \
        MMA4(K, 0);                                                                                      \
        READ_B(0, K, 2);                                                                                 \
        if (MORE) LOAD_A(((K) + DIST) % NSTAGE, sv_);                                                    \
        MMA4(K, 1);                                                                                      \
        READ_B(1, K, 3);                                                                                 \
        MMA4(K, 2);                                                                                      \
        READ_B(0, ((K) + 1) % NSTAGE, 0);                                                                \
        MMA4(K, 3);                                                                                      \
        asm volatile("s_waitcnt lgkmcnt(4)" ::: "memory");        \
        BAR();                                                                                           \
      } while (0)
      BAR(); LOAD_A(0, SCHED_READ(0)); BAR(); LOAD_A(1, SCHED_READ(1)); BAR(); LOAD_A(2, SCHED_READ(2));
      BAR(); LOAD_A(3, SCHED_READ(3)); BAR();
      READ_B(0, 0, 0);
      for (int t = 0; t < 60; t += 6) {
        STEP(0, t, true); STEP(1, t + 1, true); STEP(2, t + 2, true);
        STEP(3, t + 3, true); STEP(4, t + 4, true); STEP(5, t + 5, true);
      }
      STEP(0, 60, false); STEP(1, 61, false); STEP(2, 62, false); STEP(3, 63, false);
#undef SCHED_READ
#undef LOAD_A
#undef READ_B
#undef MMA4
#undef STEP
      if (wv == 0) STAMP(6);
      const bool bad = *lds_bad != 0u;
      const int g = a.rank >> 2;
      if (rt < 4) {
        const float qnan = __builtin_nanf("");
        float* o = a.out[a.rank] + (size_t)(rt * 128 + wv * 32 + 4 * (lane >> 5)) * NCOL + g * 2048 + ct * 128 + 4 * (lane & 31);
#pragma unroll
        for (int e = 0; e < 16; ++e) {
          f32x4 v;
#pragma unroll
          for (int c = 0; c < 4; ++c) { const float y = acc[c][e]; v[c] = bad ? qnan : (y < 0.f ? 0.f : y); }
          st16_sys(o + (size_t)((e & 3) + 8 * (e >> 2)) * NCOL, __builtin_bit_cast(u32x4, v));
        }
      } else {
        LAS char* buf = (LAS char*)lds + wv * CSLOT;
        const int q = ((g ^ 1) << 2) + (rt - 4);
        const int slot = ((a.rank & 3) * 16 + ct) * 4 + wv;
        char* dst = a.ws[q & 7] + OFF_CSLOT + (size_t)slot * CSLOT + lane * 16;
        if (bad) {
          const unsigned nb = 0x7fc00000u;
          for (int i = 0; i < 16; ++i) st16_sys(dst + i * 1024, (u32x4){nb, nb, nb, nb});
          if (lane < 32) st16_sys(dst + 16384, (u32x4){~0u, ~0u, ~0u, ~0u});
          if (lane == 32) st16_sys(dst + 16384, (u32x4){2048u, 4096u, 0u, 0u});
        } else {
          unsigned base = 0, sent = 0, n0 = 0, mhi = 0, mlo = 0;
#pragma unroll
          for (int e4 = 0; e4 < 4; ++e4) {
#pragma unroll
            for (int ee = 0; ee < 4; ++ee) {
#pragma unroll
              for (int c = 0; c < 4; ++c) {
                const float y = acc[c][e4 * 4 + ee];
                const bool nz = !(y <= 0.f);
                const unsigned long long m = __ballot(nz);
                unsigned pos = __builtin_amdgcn_mbcnt_hi((unsigned)(m >> 32), __builtin_amdgcn_mbcnt_lo((unsigned)m, base));
                asm volatile("" : "+v"(pos));
                *(LAS float*)(buf + 4u * (nz ? pos : 4288u + lane)) = y;
                if (e4 < 2) mhi = mhi * 2u + (nz ? 1u : 0u); else mlo = mlo * 2u + (nz ? 1u : 0u);
                base += (unsigned)__builtin_popcountll(m);
              }
            }
            if (e4 == 1) n0 = base;
            asm volatile("s_waitcnt vmcnt(0)" ::: "memory");
            for (; sent + 256u <= base; sent += 256u)
              st16_sys(dst + sent * 4u, *(const LAS u32x4*)(buf + sent * 4u + lane * 16));
          }
          if (sent + lane * 4u < base) st16_sys(dst + sent * 4u, *(const LAS u32x4*)(buf + sent * 4u + lane * 16));
          *(LAS u32x2*)(buf + 16384 + lane * 8) = (u32x2){mlo, mhi};
          if (lane == 0) *(LAS u32x2*)(buf + 16896) = (u32x2){n0, base};
          if (lane < 33) st16_sys(dst + 16384, *(const LAS u32x4*)(buf + 16384 + lane * 16));
        }
        if (wv == 0) STAMP(10);
        asm volatile("s_waitcnt vmcnt(0)" ::: "memory");
        if (lane == 0) st_sys(a.ws[q & 7] + OFF_OFLAG + (size_t)slot * 4, call);
      }
      if (wv == 0) STAMP(7);
    }
  }
  if (tid == 0) {
    const unsigned t = __hip_atomic_fetch_add((unsigned*)(myws + OFF_ENDCNT), 1u, __ATOMIC_RELAXED, __HIP_MEMORY_SCOPE_AGENT);
    if (t % GRID == GRID - 1) st_agent(myws + OFF_EPOCH, call);
  }
}

}

extern "C" void kernel_launch(int rank, int nranks, ncclComm_t comm,
                              hipStream_t stream,
                              void* const* d_in, const int* in_sizes, int n_in,
                              void* d_out, int out_size,
                              const int* in_dtypes, int out_dtype,
                              void* const* d_out_all,
                              void* const* d_ws_all, size_t ws_bytes) {
  (void)comm; (void)in_dtypes; (void)out_dtype; (void)d_out;
  if (nranks != NR || (unsigned)rank >= (unsigned)NR || n_in < 2 || ws_bytes < WS_NEED ||
      in_sizes[0] != 4096 * KSH || in_sizes[1] != KSH * NCOL || out_size != 512 * NCOL) {
    fprintf(stderr, "gemm_rs: unexpected problem shape\n");
    return;
  }
  Args a;
  for (int p = 0; p < NR; ++p) { a.ws[p] = (char*)d_ws_all[p]; a.out[p] = (float*)d_out_all[p]; }
  a.x = (const float*)d_in[0];
  a.w = (const float*)d_in[1];
  a.rank = rank;
  a.pad = 0;
  gemm_rs_kernel<<<GRID, THREADS, 0, stream>>>(a);
  hipError_t e = hipGetLastError();
  if (e != hipSuccess) fprintf(stderr, "gemm_rs: launch failed: %s\n", hipGetErrorString(e));
}
```

```cpp
#include <hip/hip_runtime.h>
#include <rccl/rccl.h>
#include <cstdio>

namespace {

constexpr int NR = 8;
constexpr int NCOL = 4096;
constexpr int KSH = 512;
#define LATE_V 3
constexpr int LATE = LATE_V;
#define KD_V 7
#define NSLOT_V 7
constexpr int NSLOT = NSLOT_V;
#define PACE_V 200
constexpr unsigned PACE_P = PACE_V * NSLOT / 8, PACE_D = PACE_P * 4096 / 3712;
constexpr int KD = KD_V;

constexpr size_t OFF_EPOCH = 0;
constexpr size_t OFF_ENDCNT = 128;
constexpr size_t OFF_TMO = 256;
constexpr size_t OFF_OFLAG = 1024;
constexpr size_t OFF_QCNT = 2048;
constexpr size_t OFF_PFLAG = 4096;
constexpr size_t OFF_LFLAG = 16384;
constexpr size_t OFF_STAGE = 1u << 20;
constexpr size_t PIECE = 16384;
constexpr size_t A_BYTES = 64 * 8 * PIECE;
constexpr size_t B_BYTES = 64 * 16 * PIECE;
constexpr size_t PUNIT = 4224;
constexpr size_t PPAR = 64 * 24 * 4 * PUNIT;
constexpr size_t FPAR = A_BYTES + B_BYTES;
constexpr size_t OFF_PSTAGE = OFF_STAGE + 2 * FPAR;
constexpr size_t CSLOT = 17408;
constexpr size_t OFF_CSLOT = OFF_PSTAGE + 2 * PPAR;
constexpr size_t WS_NEED = OFF_CSLOT + 256 * CSLOT;

constexpr int NPROD = 64, NCONS = 128, NDEC = 56, GRID = NPROD + NCONS + NDEC;
constexpr int THREADS = 512;
constexpr int NSTAGE = 6;
#define DIST_V 4
constexpr int DIST = DIST_V;
constexpr int LDS_CTRL = 6 * 16384;
constexpr unsigned long long DEADLINE = 30000000ull;

typedef unsigned u32x4 __attribute__((ext_vector_type(4)));
typedef unsigned u32x2 __attribute__((ext_vector_type(2)));
typedef float f32x4 __attribute__((ext_vector_type(4)));
typedef float f32x16 __attribute__((ext_vector_type(16)));
typedef _Float16 h2 __attribute__((ext_vector_type(2)));
typedef _Float16 h8 __attribute__((ext_vector_type(8)));
#define LAS __attribute__((address_space(3)))
#define GAS __attribute__((address_space(1)))

struct Args {
  char* ws[NR];
  float* out[NR];
  const float* x;
  const float* w;
  int rank;
  int pad;
};

__device__ __forceinline__ unsigned ld_sys(const void* p) {
  return __hip_atomic_load((const unsigned*)p, __ATOMIC_RELAXED, __HIP_MEMORY_SCOPE_SYSTEM);
}
__device__ __forceinline__ unsigned ld_agent(const void* p) {
  return __hip_atomic_load((const unsigned*)p, __ATOMIC_RELAXED, __HIP_MEMORY_SCOPE_AGENT);
}
__device__ __forceinline__ void st_sys(void* p, unsigned v) {
  __hip_atomic_store((unsigned*)p, v, __ATOMIC_RELAXED, __HIP_MEMORY_SCOPE_SYSTEM);
}
__device__ __forceinline__ void st_agent(void* p, unsigned v) {
  __hip_atomic_store((unsigned*)p, v, __ATOMIC_RELAXED, __HIP_MEMORY_SCOPE_AGENT);
}
__device__ __forceinline__ void st16_sys(void* p, u32x4 v) {
  asm volatile("global_store_dwordx4 %0, %1, off sc0 sc1\n\ts_nop 1" :: "v"(p), "v"(v) : "memory");
}
__device__ __forceinline__ unsigned pk(float a, float b) {
  h2 h; h.x = (_Float16)a; h.y = (_Float16)b; return __builtin_bit_cast(unsigned, h);
}
#define BAR() do { asm volatile("" ::: "memory"); __builtin_amdgcn_s_barrier(); asm volatile("" ::: "memory"); } while (0)

typedef unsigned short us2 __attribute__((ext_vector_type(2)));
constexpr int PK_UNIT = (int)PUNIT;
constexpr int PK_LDS = 4096 + 128;
struct Packed { u32x4 l0, l1, c0; u32x2 c1; u32x4 hdr; };

__device__ __forceinline__ unsigned pk_sub(unsigned a, unsigned b) { return __builtin_bit_cast(unsigned, __builtin_bit_cast(us2, a) - __builtin_bit_cast(us2, b)); }
__device__ __forceinline__ unsigned pk_add(unsigned a, unsigned b) { return __builtin_bit_cast(unsigned, __builtin_bit_cast(us2, a) + __builtin_bit_cast(us2, b)); }
__device__ __forceinline__ unsigned pk_min(unsigned a, unsigned b) { return __builtin_bit_cast(unsigned, __builtin_elementwise_min(__builtin_bit_cast(us2, a), __builtin_bit_cast(us2, b))); }
template <int CTRL, int RM> __device__ __forceinline__ unsigned dpp_add(unsigned x) {
  return x + (unsigned)__builtin_amdgcn_update_dpp(0, (int)x, CTRL, RM, 0xf, false);
}
__device__ __forceinline__ unsigned wave_scan(unsigned x) {
  x = dpp_add<0x111, 0xf>(x); x = dpp_add<0x112, 0xf>(x); x = dpp_add<0x114, 0xf>(x); x = dpp_add<0x118, 0xf>(x);
  x = dpp_add<0x142, 0xa>(x); x = dpp_add<0x143, 0xc>(x);
  return x;
}
__device__ __forceinline__ unsigned esc_addr(unsigned b, int lane) {
  const unsigned j = 15u - (b & 15u);
  return (j >> 2) * 1024u + lane * 16u + (j & 3u) * 4u + (b >> 4) * 2u + 1u;
}

__device__ __forceinline__ bool pk_encode(const u32x4 (&fr)[4], Packed& p, int lane, LAS char* lds) {
  unsigned d[16];
#pragma unroll
  for (int j = 0; j < 16; ++j) d[j] = fr[j >> 2][j & 3];
  unsigned s = 0;
#pragma unroll
  for (int j = 0; j < 16; j += 5) s += ((d[j] >> 10) & 31u) + ((d[j] >> 26) & 31u);
  s = __builtin_amdgcn_readlane(wave_scan(s), 63);
  int top = (int)((s + 1388u) >> 9);
  int base = top - 6; base = base < 0 ? 0 : (base > 24 ? 24 : base);
  const unsigned base2 = (unsigned)base * 0x00010001u;
  unsigned q[16], acc = 0;
#pragma unroll
  for (int j = 0; j < 16; ++j) {
    const unsigned x = d[j] >> 10;
    const unsigned c = pk_min(pk_sub(x & 0x001f001fu, base2), 0x00070007u);
    const unsigned c6 = (x & 0x00200020u) | ((d[j] >> 5) & 0x00180018u) | c;
    q[j] = (c6 & 0x3fu) | (c6 >> 10);
    acc = pk_add(pk_add(acc, acc), (pk_add(c, 0x00010001u) >> 3) & 0x00010001u);
  }
#pragma unroll
  for (int i = 0; i < 4; ++i) {
    p.l0[i] = __builtin_amdgcn_perm(d[2 * i + 1], d[2 * i], 0x06040200u);
    p.l1[i] = __builtin_amdgcn_perm(d[2 * i + 9], d[2 * i + 8], 0x06040200u);
  }
  p.c0[0] = q[0] | (q[1] << 12) | (q[2] << 24);
  p.c0[1] = (q[2] >> 8) | (q[3] << 4) | (q[4] << 16) | (q[5] << 28);
  p.c0[2] = (q[5] >> 4) | (q[6] << 8) | (q[7] << 20);
  p.c0[3] = q[8] | (q[9] << 12) | (q[10] << 24);
  p.c1[0] = (q[10] >> 8) | (q[11] << 4) | (q[12] << 16) | (q[13] << 28);
  p.c1[1] = (q[13] >> 4) | (q[14] << 8) | (q[15] << 20);
  const unsigned cnt = (unsigned)__builtin_popcount(acc);
  const unsigned incl = wave_scan(cnt);
  const unsigned total = __builtin_amdgcn_readlane(incl, 63);
  if (total > 127u) { p.hdr = (u32x4){0x80u, 0u, 0u, 0u}; return true; }
  LAS char* list = lds + 4096;
  if (total != 0u) {
#pragma unroll
    for (int f = 0; f < 4; ++f) *(LAS u32x4*)(lds + f * 1024 + lane * 16) = fr[f];
    unsigned off = 1u + incl - cnt;
    while (acc != 0u) {
      const unsigned b = (unsigned)__builtin_ctz(acc);
      acc &= acc - 1u;
      const unsigned hb = *(const LAS unsigned char*)(lds + esc_addr(b, lane));
      *(LAS unsigned char*)(list + off) = (unsigned char)((hb >> 2) & 31u);
      ++off;
    }
  }
  if (lane == 0) *(LAS unsigned char*)list = (unsigned char)base;
  p.hdr = *(const LAS u32x4*)(list + (lane & 7) * 16);
  return false;
}

__device__ __forceinline__ void pk_decode(const Packed& p, unsigned h, u32x4 (&fr)[4], int lane, LAS char* lds) {
  const unsigned base2 = (__builtin_amdgcn_readfirstlane(h) & 31u) * 0x00010001u;
  unsigned q[16];
  q[0] = p.c0[0] & 0xfffu; q[1] = (p.c0[0] >> 12) & 0xfffu; q[2] = __builtin_amdgcn_alignbit(p.c0[1], p.c0[0], 24) & 0xfffu;
  q[3] = (p.c0[1] >> 4) & 0xfffu; q[4] = (p.c0[1] >> 16) & 0xfffu; q[5] = __builtin_amdgcn_alignbit(p.c0[2], p.c0[1], 28) & 0xfffu;
  q[6] = (p.c0[2] >> 8) & 0xfffu; q[7] = p.c0[2] >> 20;
  q[8] = p.c0[3] & 0xfffu; q[9] = (p.c0[3] >> 12) & 0xfffu; q[10] = __builtin_amdgcn_alignbit(p.c1[0], p.c0[3], 24) & 0xfffu;
  q[11] = (p.c1[0] >> 4) & 0xfffu; q[12] = (p.c1[0] >> 16) & 0xfffu; q[13] = __builtin_amdgcn_alignbit(p.c1[1], p.c1[0], 28) & 0xfffu;
  q[14] = (p.c1[1] >> 8) & 0xfffu; q[15] = p.c1[1] >> 20;
  unsigned acc = 0;
#pragma unroll
  for (int j = 0; j < 16; ++j) {
    const unsigned c6 = (q[j] | (q[j] << 10)) & 0x003f003fu;
    const unsigned c = c6 & 0x00070007u;
    const unsigned hi8 = ((c6 << 2) & 0x00800080u) | ((c6 >> 3) & 0x00030003u) | (pk_add(c, base2) << 2);
    const unsigned l = (j < 8 ? p.l0 : p.l1)[(j & 7) >> 1];
    fr[j >> 2][j & 3] = __builtin_amdgcn_perm(hi8, l, (j & 1) ? 0x06030402u : 0x06010400u);
    acc = pk_add(pk_add(acc, acc), (pk_add(c, 0x00010001u) >> 3) & 0x00010001u);
  }
  const unsigned cnt = (unsigned)__builtin_popcount(acc);
  const unsigned incl = wave_scan(cnt);
  if (__builtin_amdgcn_readlane(incl, 63) != 0u) {
    LAS char* list = lds + 4096;
    if (lane < 32) *(LAS unsigned*)(list + lane * 4) = h;
#pragma unroll
    for (int f = 0; f < 4; ++f) *(LAS u32x4*)(lds + f * 1024 + lane * 16) = fr[f];
    unsigned off = 1u + incl - cnt;
    while (acc != 0u) {
      const unsigned b = (unsigned)__builtin_ctz(acc);
      acc &= acc - 1u;
      LAS unsigned char* t = (LAS unsigned char*)(lds + esc_addr(b, lane));
      const unsigned e = *(const LAS unsigned char*)(list + off);
      *t = (unsigned char)((*t & 0x83u) | (e << 2));
      ++off;
    }
#pragma unroll
    for (int f = 0; f < 4; ++f) fr[f] = *(const LAS u32x4*)(lds + f * 1024 + lane * 16);
  }
}

struct Piece { int kc, tile; };
__device__ __forceinline__ Piece decode(int n) {
  Piece p;
  constexpr int nf = LATE * 24, n2 = nf + (8 - LATE) * 20;
  int i;
  if (n < nf) { p.kc = n / 24; i = n % 24; }
  else if (n < n2) { const int v = n - nf; p.kc = LATE + v / 20; i = v % 20; }
  else { const int v = n - n2; p.kc = LATE + v / 4; i = 20 + v % 4; }
  p.tile = i < 16 ? 8 + i : (i < 20 ? i - 12 : i - 20);
  return p;
}
#define FIRSTD_V 7
__device__ __forceinline__ bool is_direct(const Piece& p, int n) { return n < FIRSTD_V || p.kc >= KD || (p.tile < 4 && p.kc >= LATE); }
__device__ __forceinline__ size_t stage_off(const Piece& p, int cs) {
  return p.tile >= 8 ? A_BYTES + ((size_t)cs * 16 + (p.tile - 8)) * PIECE : ((size_t)cs * 8 + p.tile) * PIECE;
}
struct Src { const char* base; unsigned ustep, loff, js; };
__device__ __forceinline__ Src src_of(const Args& a, int d, const Piece& p, int lane) {
  Src s;
  const unsigned i = lane & 31, h = lane >> 5;
  if (p.tile >= 8) {
    s.base = (const char*)a.w + ((size_t)(p.kc * 64) * NCOL + (d >> 2) * 2048 + (p.tile - 8) * 128) * 4;
    s.ustep = NCOL * 4; s.loff = h * (4 * NCOL * 4) + i * 16; s.js = 8 * NCOL * 4;
  } else {
    const int row0 = p.tile < 4 ? d * 512 + p.tile * 128 : (((d >> 2) ^ 1) * 4 + (p.tile - 4)) * 512 + (d & 3) * 128;
    s.base = (const char*)a.x + ((size_t)row0 * KSH + p.kc * 64) * 4;
    s.ustep = 32 * KSH * 4; s.loff = i * (KSH * 4) + h * 16; s.js = 32;
  }
  return s;
}
__device__ __forceinline__ void load8(f32x4 (&r)[8], const char* base, unsigned loff, unsigned js) {
  unsigned o[8];
#pragma unroll
  for (int j = 0; j < 8; ++j) o[j] = loff + j * js;
  asm volatile(
      "s_nop 4\n\t"
      "global_load_dwordx4 %0, %8, %16\n\t"
      "global_load_dwordx4 %1, %9, %16\n\t"
      "global_load_dwordx4 %2, %10, %16\n\t"
      "global_load_dwordx4 %3, %11, %16\n\t"
      "global_load_dwordx4 %4, %12, %16\n\t"
      "global_load_dwordx4 %5, %13, %16\n\t"
      "global_load_dwordx4 %6, %14, %16\n\t"
      "global_load_dwordx4 %7, %15, %16"
      : "=&v"(r[0]), "=&v"(r[1]), "=&v"(r[2]), "=&v"(r[3]), "=&v"(r[4]), "=&v"(r[5]), "=&v"(r[6]), "=&v"(r[7])
      : "v"(o[0]), "v"(o[1]), "v"(o[2]), "v"(o[3]), "v"(o[4]), "v"(o[5]), "v"(o[6]), "v"(o[7]), "s"(base)
      : "memory");
}
#define WAIT_R(N, r) asm volatile("s_waitcnt vmcnt(" #N ")" : "+v"(r[0]), "+v"(r[1]), "+v"(r[2]), "+v"(r[3]), \
    "+v"(r[4]), "+v"(r[5]), "+v"(r[6]), "+v"(r[7]) :: "memory")
__device__ __forceinline__ void cvt4(const f32x4 (&r)[8], u32x4 (&fr)[4]) {
#pragma unroll
  for (int c = 0; c < 4; ++c)
    fr[c] = (u32x4){pk(r[0][c], r[1][c]), pk(r[2][c], r[3][c]), pk(r[4][c], r[5][c]), pk(r[6][c], r[7][c])};
}
__device__ __forceinline__ void store4(char* dbase, unsigned voff, const u32x4 (&fr)[4]) {
  asm volatile(
      "s_nop 4\n\t"
      "global_store_dwordx4 %0, %1, %5 sc0 sc1\n\t"
      "global_store_dwordx4 %0, %2, %5 offset:1024 sc0 sc1\n\t"
      "global_store_dwordx4 %0, %3, %5 offset:2048 sc0 sc1\n\t"
      "global_store_dwordx4 %0, %4, %5 offset:3072 sc0 sc1\n\t"
      "s_nop 1"
      :: "v"(voff), "v"(fr[0]), "v"(fr[1]), "v"(fr[2]), "v"(fr[3]), "s"(dbase) : "memory");
}
__device__ __forceinline__ void store_packed(char* dbase, int lane, const Packed& p) {
  asm volatile(
      "s_nop 4\n\t"
      "global_store_dwordx4 %0, %2, %6 sc0 sc1\n\t"
      "global_store_dwordx4 %0, %3, %6 offset:1024 sc0 sc1\n\t"
      "global_store_dwordx4 %0, %4, %6 offset:2048 sc0 sc1\n\t"
      "global_store_dwordx2 %1, %5, %6 offset:3072 sc0 sc1\n\t"
      "s_nop 1"
      :: "v"(lane * 16), "v"(lane * 8), "v"(p.l0), "v"(p.l1), "v"(p.c0), "v"(p.c1), "s"(dbase) : "memory");
}

__device__ __forceinline__ void queue_wave(const Args& a, unsigned call, int d, int slot, int lane,
                                           LAS unsigned* seq, LAS unsigned* prog) {
  unsigned* ctr = (unsigned*)(a.ws[a.rank] + OFF_QCNT + (size_t)d * 128);
  for (unsigned i = 0;; ++i) {
    const unsigned cur = i == 0 ? (unsigned)slot : i == 1 ? (unsigned)(slot + NSLOT)
                                : __builtin_amdgcn_readfirstlane(*(volatile LAS unsigned*)(seq + i));
    if (cur >= 192u) break;
    while (__builtin_amdgcn_readfirstlane(*(volatile LAS unsigned*)prog) <= i) __builtin_amdgcn_s_sleep(2);
    if (lane == 0) {
      const unsigned t = __hip_atomic_fetch_add(ctr, 1u, __ATOMIC_RELAXED, __HIP_MEMORY_SCOPE_AGENT) - (call - 1u) * 192u;
      *(volatile LAS unsigned*)(seq + i + 2) = 2u * NSLOT + t;
    }
    asm volatile("s_waitcnt vmcnt(0) lgkmcnt(0)" ::: "memory");
  }
}

__device__ __forceinline__ unsigned producer(const Args& a, int d, int slot, int u, int lane,
                                             LAS char* scratch, LAS unsigned* cnt, LAS unsigned* seq, LAS unsigned* prog) {
  constexpr int NP = 192;
  char* dws = a.ws[d];
  const unsigned voff = lane * 16;
  f32x4 r[8];
  u32x4 fr[4];
  Piece p = decode(slot);
  Src s = src_of(a, d, p, lane);
  load8(r, s.base + u * s.ustep, s.loff, s.js);
  unsigned call_v;
  asm volatile("global_load_dword %0, %1, %2 sc1" : "=&v"(call_v) : "v"(0u), "s"(a.ws[a.rank] + OFF_EPOCH) : "memory");
  BAR();
  WAIT_R(0, r);
  asm volatile("s_nop 0" : "+v"(call_v));
  const unsigned call = __builtin_amdgcn_readfirstlane(call_v) + 1u;
  char* pstage = dws + OFF_PSTAGE + (size_t)(call & 1u) * PPAR;
  char* fstage = dws + OFF_STAGE + (size_t)(call & 1u) * FPAR;
  char* prevflag = nullptr;
  bool prevdirect = false;
  unsigned long long tnext = 0;
  int i = 0;
  auto publish = [&](int idx) -> int {
    unsigned old = 0;
    if (lane == 0) old = __hip_atomic_fetch_add(cnt + idx, 1u, __ATOMIC_RELAXED, __HIP_MEMORY_SCOPE_WORKGROUP);
    if (__builtin_amdgcn_readfirstlane(old) != 3u) return 0;
    if (lane == 0) { if (prevdirect) st16_sys(prevflag, (u32x4){call, call, call, call}); else st_sys(prevflag, call); }
    return 1;
  };
  int after = 4;
  int n = slot;
  for (; n < NP; ++i) {
    if (u == 0 && lane == 0) *(volatile LAS unsigned*)prog = (unsigned)i + 1u;
    int nx;
    if (i == 0) nx = slot + NSLOT;
    else {
      unsigned v;
      while ((v = __builtin_amdgcn_readfirstlane(*(volatile LAS unsigned*)(seq + i + 1))) == 0xffffffffu) __builtin_amdgcn_s_sleep(1);
      nx = (int)v;
    }
    const int nn = nx < NP ? nx : n;
    const Piece pn = decode(nn);
    const Src sn = src_of(a, d, pn, lane);
    const int cs = a.rank * 8 + p.kc;
    const bool direct = is_direct(p, n);
    asm volatile("s_waitcnt vmcnt(6)\n\ts_cmp_gt_u32 %8, 5\n\ts_cbranch_scc1 1f\n\ts_waitcnt vmcnt(5)\n\t"
                 "s_cmp_gt_u32 %8, 4\n\ts_cbranch_scc1 1f\n\ts_waitcnt vmcnt(4)\n1:"
                 : "+v"(r[0]), "+v"(r[1]), "+v"(r[2]), "+v"(r[3]), "+v"(r[4]), "+v"(r[5]), "+v"(r[6]), "+v"(r[7])
                 : "s"(__builtin_amdgcn_readfirstlane(after)) : "scc", "memory");
    cvt4(r, fr);
    load8(r, sn.base + u * sn.ustep, sn.loff, sn.js);
    Packed pk;
    bool raw = true;
    if (!direct) raw = pk_encode(fr, pk, lane, scratch);
    asm volatile("s_waitcnt vmcnt(8)" ::: "memory");
    after = (direct ? 4 : 5) + (prevflag != nullptr ? publish(i - 1) : 0);
    {
      unsigned long long now = __builtin_amdgcn_s_memrealtime();
      while (now < tnext) { __builtin_amdgcn_s_sleep(1); now = __builtin_amdgcn_s_memrealtime(); }
      tnext = now + (direct ? PACE_D : PACE_P);
    }
    if (direct) {
      store4(fstage + stage_off(p, cs) + u * 4096, voff, fr);
      prevflag = dws + OFF_LFLAG + (size_t)(p.tile * 64 + cs) * 16;
    } else {
      char* dst = pstage + (size_t)((cs * 24 + p.tile) * 4 + u) * PUNIT;
      if (raw) store4(dst, voff, fr); else store_packed(dst, lane, pk);
      if (lane < 8) st16_sys(dst + 4096 + lane * 16, pk.hdr);
      prevflag = dws + OFF_PFLAG + (size_t)(p.tile * 64 + cs) * 4;
    }
    prevdirect = direct;
    p = pn; s = sn; n = nx;
  }
  asm volatile("s_waitcnt vmcnt(0)" : "+v"(r[0]) :: "memory");
  publish(i - 1);
  return call;
}

__device__ __forceinline__ void decoder(const Args& a, unsigned call, int p, int j, int gw, int lane, LAS char* scratch) {
  char* myws = a.ws[a.rank];
  const char* pstage = myws + OFF_PSTAGE + (size_t)(call & 1u) * PPAR;
  const int u = j & 3;
  for (int U = gw; U < 768; U += 448) {
    const Piece pc = decode(U >> 2);
    const Src s = src_of(a, a.rank, pc, lane);
    const int cs = a.rank * 8 + pc.kc;
    f32x4 r[8]; u32x4 fr[4];
    load8(r, s.base + (U & 3) * s.ustep, s.loff, s.js);
    WAIT_R(0, r);
    cvt4(r, fr);
    store4(myws + OFF_STAGE + (size_t)(call & 1u) * FPAR + stage_off(pc, cs) + (U & 3) * 4096, lane * 16, fr);
    asm volatile("s_waitcnt vmcnt(0)" ::: "memory");
    if (lane == 0) st_agent(myws + OFF_LFLAG + (size_t)((pc.tile * 64 + cs) * 4 + (U & 3)) * 4, call);
  }
  for (int k = 0; k < 12; ++k) {
    const Piece pc = decode((j >> 2) + 16 * k);
    if (is_direct(pc, (j >> 2) + 16 * k)) continue;
    const int cs = p * 8 + pc.kc;
    const char* flag = myws + OFF_PFLAG + (size_t)(pc.tile * 64 + cs) * 4;
    {
      const unsigned long long t0 = __builtin_amdgcn_s_memrealtime();
      bool bad = false;
      for (;;) {
        const unsigned fv = ld_sys(flag), tm = ld_agent(myws + OFF_TMO);
        if (__builtin_amdgcn_readfirstlane(fv) >= call) break;
        if (__builtin_amdgcn_readfirstlane(tm) == call || __builtin_amdgcn_s_memrealtime() - t0 > DEADLINE) {
          if (lane == 0) st_agent(myws + OFF_TMO, call);
          bad = true;
          break;
        }
        __builtin_amdgcn_s_sleep(2);
      }
      if (bad) return;
    }
    const char* src = pstage + (size_t)((cs * 24 + pc.tile) * 4 + u) * PUNIT;
    Packed pk; unsigned h;
    asm volatile("s_nop 4\n\t"
                 "global_load_dword %4, %7, %8 offset:0 sc0 sc1\n\t"
                 "global_load_dwordx4 %2, %5, %8 offset:2048 sc0 sc1\n\t"
                 "global_load_dwordx2 %3, %6, %8 offset:3072 sc0 sc1\n\t"
                 "global_load_dwordx4 %0, %5, %8 sc0 sc1\n\t"
                 "global_load_dwordx4 %1, %5, %8 offset:1024 sc0 sc1\n\t"
                 "s_waitcnt vmcnt(0)"
                 : "=&v"(pk.l0), "=&v"(pk.l1), "=&v"(pk.c0), "=&v"(pk.c1), "=&v"(h)
                 : "v"(lane * 16), "v"(lane * 8), "v"(4096 + (lane & 31) * 4), "s"(src) : "memory");
    u32x4 fr[4];
    if (__builtin_amdgcn_readfirstlane(h) & 0x80u) {
      asm volatile("global_load_dwordx4 %0, %4, %5 sc0 sc1\n\t"
                   "global_load_dwordx4 %1, %4, %5 offset:1024 sc0 sc1\n\t"
                   "global_load_dwordx4 %2, %4, %5 offset:2048 sc0 sc1\n\t"
                   "global_load_dwordx4 %3, %4, %5 offset:3072 sc0 sc1\n\t"
                   "s_waitcnt vmcnt(0)"
                   : "=&v"(fr[0]), "=&v"(fr[1]), "=&v"(fr[2]), "=&v"(fr[3]) : "v"(lane * 16), "s"(src) : "memory");
    } else {
      pk_decode(pk, h, fr, lane, scratch);
    }
    char* dst = myws + OFF_STAGE + (size_t)(call & 1u) * FPAR + stage_off(pc, cs);
    store4(dst + u * 4096, lane * 16, fr);
    asm volatile("s_waitcnt vmcnt(0)" ::: "memory");
    if (lane == 0) st_agent(myws + OFF_LFLAG + (size_t)((pc.tile * 64 + cs) * 4 + u) * 4, call);
  }
}

__device__ __forceinline__ void expander(const Args& a, unsigned call, int ti, int part, int half, int lane,
                                         LAS char* buf, LAS unsigned* sync) {
  char* myws = a.ws[a.rank];
  const int slot = ti * 4 + part;
  const char* flag = myws + OFF_OFLAG + (size_t)slot * 4;
  bool bad = false;
  {
    const unsigned long long t0 = __builtin_amdgcn_s_memrealtime();
    for (;;) {
      const unsigned fv = ld_sys(flag), tm = ld_agent(myws + OFF_TMO);
      if (__builtin_amdgcn_readfirstlane(fv) >= call) break;
      if (__builtin_amdgcn_readfirstlane(tm) == call || __builtin_amdgcn_s_memrealtime() - t0 > DEADLINE) {
        if (lane == 0) st_agent(myws + OFF_TMO, call);
        bad = true;
        break;
      }
      __builtin_amdgcn_s_sleep(1);
    }
  }
  const char* src = myws + OFF_CSLOT + (size_t)slot * CSLOT + lane * 16;
  if (!bad) {
#pragma unroll
    for (int i = 0; i < 5; ++i)
      __builtin_amdgcn_global_load_lds((const GAS void*)(src + (half * 5 + i) * 1024), (LAS void*)(buf + (half * 5 + i) * 1024), 16, 0, 17);
    if (half == 0) __builtin_amdgcn_global_load_lds((const GAS void*)(src + 16384), (LAS void*)(buf + 16384), 16, 0, 17);
  }
  const unsigned tm = __builtin_amdgcn_readfirstlane(ld_agent(myws + OFF_TMO));
  asm volatile("s_waitcnt vmcnt(0)" ::: "memory");
  bad = bad || tm == call;
  if (lane == 0) __hip_atomic_store(sync + half, bad ? 2u : 1u, __ATOMIC_RELEASE, __HIP_MEMORY_SCOPE_WORKGROUP);
  {
    unsigned o;
    const unsigned long long t0 = __builtin_amdgcn_s_memrealtime();
    while ((o = __builtin_amdgcn_readfirstlane(__hip_atomic_load(sync + (half ^ 1), __ATOMIC_ACQUIRE, __HIP_MEMORY_SCOPE_WORKGROUP))) == 0u) {
      if (__builtin_amdgcn_s_memrealtime() - t0 > 2 * DEADLINE) { o = 2u; break; }
    }
    bad = bad || o == 2u;
  }
  float* o = a.out[a.rank] + (size_t)((ti >> 4) * 128 + part * 32 + half * 16 + 4 * (lane >> 5)) * NCOL +
             ((a.rank >> 2) ^ 1) * 2048 + (ti & 15) * 128 + 4 * (lane & 31);
  if (bad) {
    for (int e = 0; e < 8; ++e)
      st16_sys(o + (size_t)((e & 3) + 8 * (e >> 2)) * NCOL, (u32x4){0x7fc00000u, 0x7fc00000u, 0x7fc00000u, 0x7fc00000u});
    return;
  }
  const u32x2 m = *(const LAS u32x2*)(buf + 16384 + lane * 8);
  const u32x2 cnt = *(const LAS u32x2*)(buf + 16896);
  const unsigned n0 = __builtin_amdgcn_readfirstlane(cnt[0]), total = __builtin_amdgcn_readfirstlane(cnt[1]);
  if (total > 2560u) {
#pragma unroll
    for (int i = 10; i < 16; ++i)
      __builtin_amdgcn_global_load_lds((const GAS void*)(src + i * 1024), (LAS void*)(buf + i * 1024), 16, 0, 17);
    asm volatile("s_waitcnt vmcnt(0)" ::: "memory");
  }
  unsigned base = half ? n0 : 0u;
  const unsigned bits = half ? m[0] : m[1];
  unsigned pos[32];
#pragma unroll
  for (int i = 0; i < 32; ++i) {
    const unsigned long long mm = __ballot((int)(bits << i) < 0);
    pos[i] = __builtin_amdgcn_mbcnt_hi((unsigned)(mm >> 32), __builtin_amdgcn_mbcnt_lo((unsigned)mm, base));
    base += (unsigned)__builtin_popcountll(mm);
  }
  float x[32];
#pragma unroll
  for (int i = 0; i < 32; ++i) x[i] = *(const LAS float*)(buf + (pos[i] & 4095u) * 4u);
#pragma unroll
  for (int e = 0; e < 8; ++e) {
    asm volatile("" : "+v"(x[4 * e]), "+v"(x[4 * e + 1]), "+v"(x[4 * e + 2]), "+v"(x[4 * e + 3]));
    f32x4 v;
#pragma unroll
    for (int c = 0; c < 4; ++c) v[c] = (int)(bits << (4 * e + c)) < 0 ? x[4 * e + c] : 0.f;
    st16_sys(o + (size_t)((e & 3) + 8 * (e >> 2)) * NCOL, __builtin_bit_cast(u32x4, v));
  }
}

__global__ void __launch_bounds__(THREADS) gemm_rs_kernel(Args a) {
  __shared__ __attribute__((aligned(16))) char lds[LDS_CTRL + 512];
  const int tid = threadIdx.x, lane = tid & 63;
  const int wv = __builtin_amdgcn_readfirstlane(tid >> 6);
  const int b = blockIdx.x, xcd = b & 7, jb = b >> 3;
  char* myws = a.ws[a.rank];
  unsigned call = 0;

  if (jb < 8) {
    LAS unsigned* sync = (LAS unsigned*)((LAS char*)lds + LDS_CTRL);
    LAS unsigned* seq = (LAS unsigned*)((LAS char*)lds + 4 * CSLOT);
    LAS unsigned* cnt = seq + 256;
    const bool streams = jb < NSLOT && xcd != a.rank;
    if (streams && wv < 4) {
      call = producer(a, xcd, jb, wv, lane, (LAS char*)lds + wv * PK_LDS, cnt, seq, sync + 8);
    } else
    {
      call = __builtin_amdgcn_readfirstlane(ld_agent(myws + OFF_EPOCH)) + 1u;
      if (wv == 7 && lane < 16) sync[lane] = 0u;
      if (wv >= 4) { seq[(wv - 4) * 64 + lane] = 0xffffffffu; cnt[(wv - 4) * 64 + lane] = 0u; }
      asm volatile("s_waitcnt lgkmcnt(0)" ::: "memory");
      BAR();
      if (streams && wv == 4) queue_wave(a, call, xcd, jb, lane, seq, sync + 8);
    }
    BAR();
    expander(a, call, jb * 8 + xcd, wv >> 1, wv & 1, lane, (LAS char*)lds + (wv >> 1) * CSLOT, sync + (wv >> 1) * 2);
  } else if (jb >= 24) {
    call = __builtin_amdgcn_readfirstlane(ld_agent(myws + OFF_EPOCH)) + 1u;
    decoder(a, call, (a.rank + 1 + (jb - 24)) & 7, xcd * 8 + wv, (jb - 24) * 64 + xcd * 8 + wv, lane, (LAS char*)lds + wv * PK_LDS);
    BAR();
  } else {
    const int jc = jb - 8;
    const int rt = (xcd & 3) + (jc >> 3) * 4;
    const int ct = (xcd >> 2) * 8 + (jc & 7);
    call = __builtin_amdgcn_readfirstlane(ld_agent(myws + OFF_EPOCH)) + 1u;
    LAS unsigned* lds_bad = (LAS unsigned*)((LAS char*)lds + LDS_CTRL);
    LAS unsigned* lds_sched = (LAS unsigned*)((LAS char*)lds + LDS_CTRL + 64);
    const char* stage = myws + OFF_STAGE + (size_t)(call & 1u) * FPAR;
    if (wv == 6) {
      const char* fa = myws + OFF_LFLAG + (size_t)(rt * 64 + lane) * 16;
      const char* fb = myws + OFF_LFLAG + (size_t)((8 + ct) * 64 + lane) * 16;
      const char* ptm = myws + OFF_TMO;
      unsigned long long ready = 0, done = 0;
      for (int c = 0; c < 64; ++c) {
        if ((ready & ~done) == 0ull) {
          const unsigned long long t0 = __builtin_amdgcn_s_memrealtime();
          for (;;) {
            u32x4 va, vb; unsigned tm;
            asm volatile("global_load_dwordx4 %0, %3, off sc0 sc1\n\t"
                         "global_load_dwordx4 %1, %4, off sc0 sc1\n\t"
                         "global_load_dword %2, %5, off sc1\n\t"
                         "s_waitcnt vmcnt(0)"
                         : "=&v"(va), "=&v"(vb), "=&v"(tm) : "v"(fa), "v"(fb), "v"(ptm) : "memory");
            const unsigned lo = min(min(min(va[0], va[1]), min(va[2], va[3])), min(min(vb[0], vb[1]), min(vb[2], vb[3])));
            ready = __ballot(lo >= call);
            if ((ready & ~done) != 0ull) break;
            if (__builtin_amdgcn_readfirstlane(tm) == call || __builtin_amdgcn_s_memrealtime() - t0 > DEADLINE) {
              if (lane == 0) st_agent(myws + OFF_TMO, call);
              ready = ~0ull;
              break;
            }
            __builtin_amdgcn_s_sleep(1);
          }
        }
        const int cs = __builtin_ctzll(ready & ~done);
        done |= 1ull << cs;
        if (lane == 0) lds_sched[c & 7] = (unsigned)cs;
        if (c == 63) {
          const unsigned t = __builtin_amdgcn_readfirstlane(ld_agent(myws + OFF_TMO));
          if (lane == 0) *lds_bad = (t == call) ? 1u : 0u;
        }
        asm volatile("s_waitcnt lgkmcnt(0)" ::: "memory");
        BAR();
      }
      for (int i = 0; i <= DIST; ++i) BAR();
      asm volatile("s_waitcnt vmcnt(0)" ::: "memory");
    } else if (wv == 7) {
    } else if (wv >= 4) {
      const int lw = wv - 4;
      const char* gbase = stage + A_BYTES + (size_t)ct * PIECE + lw * 8192 + lane * 16;
      BAR();
      int st = 0;
      for (int c = 0; c < 64 + DIST; ++c) {
        if (c < 64) {
          const int cs = __builtin_amdgcn_readfirstlane(*(volatile LAS unsigned*)(lds_sched + (c & 7))) & 63;
          const char* ga = gbase + (size_t)cs * (16 * PIECE);
          LAS char* la = (LAS char*)lds + st * 16384 + lw * 8192;
#pragma unroll
          for (int f = 0; f < 8; ++f)
            __builtin_amdgcn_global_load_lds((const GAS void*)(ga + f * 1024), (LAS void*)(la + f * 1024), 16, 0, 17);
          if constexpr (DIST == 4) asm volatile("s_waitcnt vmcnt(16)" ::: "memory");
          else asm volatile("s_waitcnt vmcnt(24)" ::: "memory");
        } else if (c + 4 - DIST == 63) {
          asm volatile("s_waitcnt vmcnt(16)" ::: "memory");
        } else if (c + 4 - DIST == 64) {
          asm volatile("s_waitcnt vmcnt(8)" ::: "memory");
        } else {
          asm volatile("s_waitcnt vmcnt(0)" ::: "memory");
        }
        st = st == NSTAGE - 1 ? 0 : st + 1;
        BAR();
      }
    } else {
      f32x16 acc[4];
#pragma unroll
      for (int c = 0; c < 4; ++c)
#pragma unroll
        for (int e = 0; e < 16; ++e) acc[c][e] = 0.f;
      const __amdgpu_buffer_rsrc_t rsa = __builtin_amdgcn_make_buffer_rsrc((void*)(stage + (size_t)rt * PIECE), 0, (int)A_BYTES, 0x00020000);
      const unsigned aoff = wv * 4096 + lane * 16;
      h8 ar[NSTAGE][4];
      h8 bf[2][4];
#define SCHED_READ(C) (*(volatile LAS unsigned*)(lds_sched + ((C) & 7)))
#define LOAD_A(SLOT, SV) do {                                                                            \
        const unsigned cs_ = __builtin_amdgcn_readfirstlane(SV) & 63u;                                   \
        _Pragma("unroll") for (int s_ = 0; s_ < 4; ++s_)                                                 \
          ar[SLOT][s_] = __builtin_bit_cast(h8, __builtin_amdgcn_raw_buffer_load_b128(rsa, aoff + s_ * 1024, cs_ * (unsigned)(8 * PIECE), 17)); \
      } while (0)
#define READ_B(BUF, ST, S) do {                                                                          \
        _Pragma("unroll") for (int c_ = 0; c_ < 4; ++c_)                                                 \
          bf[BUF][c_] = *(const LAS h8*)((const LAS char*)lds + (ST) * 16384 + (S) * 4096 + c_ * 1024 + lane * 16); \
      } while (0)
#define MMA4(SLOT, S) do {                                                                               \
        __builtin_amdgcn_sched_barrier(0);                                                               \
        _Pragma("unroll") for (int c_ = 0; c_ < 4; ++c_)                                                 \
          acc[c_] = __builtin_amdgcn_mfma_f32_32x32x16_f16(ar[SLOT][S], bf[(S) & 1][c_], acc[c_], 0, 0, 0); \
        __builtin_amdgcn_sched_barrier(0);                                                               \
      } while (0)
#define STEP(K, T, MORE) do {                                                                            \
        READ_B(1, K, 1);                                                                                 \
        unsigned sv_ = 0; if (MORE) sv_ = SCHED_READ((T) + DIST);                                        \
        MMA4(K, 0);                                                                                      \
        READ_B(0, K, 2);                                                                                 \
        if (MORE) LOAD_A(((K) + DIST) % NSTAGE, sv_);                                                    \
        MMA4(K, 1);                                                                                      \
        READ_B(1, K, 3);                                                                                 \
        MMA4(K, 2);                                                                                      \
        READ_B(0, ((K) + 1) % NSTAGE, 0);                                                                \
        MMA4(K, 3);                                                                                      \
        asm volatile("s_waitcnt lgkmcnt(4)" ::: "memory");        \
        BAR();                                                                                           \
      } while (0)
      BAR(); LOAD_A(0, SCHED_READ(0)); BAR(); LOAD_A(1, SCHED_READ(1)); BAR(); LOAD_A(2, SCHED_READ(2));
      BAR(); LOAD_A(3, SCHED_READ(3));
      if constexpr (DIST == 5) { BAR(); LOAD_A(4, SCHED_READ(4)); }
      BAR();
      READ_B(0, 0, 0);
      for (int t = 0; t < 54; t += 6) {
        STEP(0, t, true); STEP(1, t + 1, true); STEP(2, t + 2, true);
        STEP(3, t + 3, true); STEP(4, t + 4, true); STEP(5, t + 5, true);
      }
      STEP(0, 54, true); STEP(1, 55, true); STEP(2, 56, true); STEP(3, 57, true); STEP(4, 58, true);
      STEP(5, 59, 59 + DIST <= 63);
      STEP(0, 60, false); STEP(1, 61, false); STEP(2, 62, false); STEP(3, 63, false);
#undef SCHED_READ
#undef LOAD_A
#undef READ_B
#undef MMA4
#undef STEP
      const bool bad = *lds_bad != 0u;
      const int g = a.rank >> 2;
      if (rt < 4) {
        const float qnan = __builtin_nanf("");
        float* o = a.out[a.rank] + (size_t)(rt * 128 + wv * 32 + 4 * (lane >> 5)) * NCOL + g * 2048 + ct * 128 + 4 * (lane & 31);
#pragma unroll
        for (int e = 0; e < 16; ++e) {
          f32x4 v;
#pragma unroll
          for (int c = 0; c < 4; ++c) { const float y = acc[c][e]; v[c] = bad ? qnan : (y < 0.f ? 0.f : y); }
          st16_sys(o + (size_t)((e & 3) + 8 * (e >> 2)) * NCOL, __builtin_bit_cast(u32x4, v));
        }
      } else {
        LAS char* buf = (LAS char*)lds + wv * CSLOT;
        const int q = ((g ^ 1) << 2) + (rt - 4);
        const int slot = ((a.rank & 3) * 16 + ct) * 4 + wv;
        char* dst = a.ws[q & 7] + OFF_CSLOT + (size_t)slot * CSLOT + lane * 16;
        if (bad) {
          const unsigned nb = 0x7fc00000u;
          for (int i = 0; i < 16; ++i) st16_sys(dst + i * 1024, (u32x4){nb, nb, nb, nb});
          if (lane < 32) st16_sys(dst + 16384, (u32x4){~0u, ~0u, ~0u, ~0u});
          if (lane == 32) st16_sys(dst + 16384, (u32x4){2048u, 4096u, 0u, 0u});
        } else {
          unsigned base = 0, sent = 0, n0 = 0, mhi = 0, mlo = 0;
#pragma unroll
          for (int e4 = 0; e4 < 4; ++e4) {
#pragma unroll
            for (int ee = 0; ee < 4; ++ee) {
#pragma unroll
              for (int c = 0; c < 4; ++c) {
                const float y = acc[c][e4 * 4 + ee];
                const bool nz = !(y <= 0.f);
                const unsigned long long m = __ballot(nz);
                unsigned pos = __builtin_amdgcn_mbcnt_hi((unsigned)(m >> 32), __builtin_amdgcn_mbcnt_lo((unsigned)m, base));
                asm volatile("" : "+v"(pos));
                *(LAS float*)(buf + 4u * (nz ? pos : 4288u + lane)) = y;
                if (e4 < 2) mhi = mhi * 2u + (nz ? 1u : 0u); else mlo = mlo * 2u + (nz ? 1u : 0u);
                base += (unsigned)__builtin_popcountll(m);
              }
            }
            if (e4 == 1) n0 = base;
            asm volatile("s_waitcnt vmcnt(0)" ::: "memory");
            for (; sent + 256u <= base; sent += 256u)
              st16_sys(dst + sent * 4u, *(const LAS u32x4*)(buf + sent * 4u + lane * 16));
          }
          if (sent + lane * 4u < base) st16_sys(dst + sent * 4u, *(const LAS u32x4*)(buf + sent * 4u + lane * 16));
          *(LAS u32x2*)(buf + 16384 + lane * 8) = (u32x2){mlo, mhi};
          if (lane == 0) *(LAS u32x2*)(buf + 16896) = (u32x2){n0, base};
          if (lane < 33) st16_sys(dst + 16384, *(const LAS u32x4*)(buf + 16384 + lane * 16));
        }
        asm volatile("s_waitcnt vmcnt(0)" ::: "memory");
        if (lane == 0) st_sys(a.ws[q & 7] + OFF_OFLAG + (size_t)slot * 4, call);
      }
    }
  }
  if (tid == 0) {
    const unsigned t = __hip_atomic_fetch_add((unsigned*)(myws + OFF_ENDCNT), 1u, __ATOMIC_RELAXED, __HIP_MEMORY_SCOPE_AGENT);
    if (t % GRID == GRID - 1) st_agent(myws + OFF_EPOCH, call);
  }
}

}

extern "C" void kernel_launch(int rank, int nranks, ncclComm_t comm,
                              hipStream_t stream,
                              void* const* d_in, const int* in_sizes, int n_in,
                              void* d_out, int out_size,
                              const int* in_dtypes, int out_dtype,
                              void* const* d_out_all,
                              void* const* d_ws_all, size_t ws_bytes) {
  (void)comm; (void)in_dtypes; (void)out_dtype; (void)d_out;
  if (nranks != NR || (unsigned)rank >= (unsigned)NR || n_in < 2 || ws_bytes < WS_NEED ||
      in_sizes[0] != 4096 * KSH || in_sizes[1] != KSH * NCOL || out_size != 512 * NCOL) {
    fprintf(stderr, "gemm_rs: unexpected problem shape\n");
    return;
  }
  Args a;
  for (int p = 0; p < NR; ++p) { a.ws[p] = (char*)d_ws_all[p]; a.out[p] = (float*)d_out_all[p]; }
  a.x = (const float*)d_in[0];
  a.w = (const float*)d_in[1];
  a.rank = rank;
  a.pad = 0;
  gemm_rs_kernel<<<GRID, THREADS, 0, stream>>>(a);
  hipError_t e = hipGetLastError();
  if (e != hipSuccess) fprintf(stderr, "gemm_rs: launch failed: %s\n", hipGetErrorString(e));
}
```

```cpp
#include <hip/hip_runtime.h>
#include <rccl/rccl.h>
#include <cstdio>

namespace {

constexpr int NR = 8;
constexpr int NCOL = 4096;
constexpr int KSH = 512;
#define LATE_V 3
constexpr int LATE = LATE_V;
#define KD_V 7
#define NSLOT_V 7
constexpr int NSLOT = NSLOT_V;
#define PACE_V 200
constexpr unsigned PACE_P = PACE_V * NSLOT / 8, PACE_D = PACE_P * 4096 / 3712;
constexpr int KD = KD_V;

constexpr size_t OFF_EPOCH = 0;
constexpr size_t OFF_ENDCNT = 128;
constexpr size_t OFF_TMO = 256;
constexpr size_t OFF_OFLAG = 1024;
constexpr size_t OFF_QCNT = 2048;
constexpr size_t OFF_PFLAG = 4096;
constexpr size_t OFF_LFLAG = 16384;
constexpr size_t OFF_STAGE = 1u << 20;
constexpr size_t PIECE = 16384;
constexpr size_t A_BYTES = 64 * 8 * PIECE;
constexpr size_t B_BYTES = 64 * 16 * PIECE;
constexpr size_t PUNIT = 4224;
constexpr size_t PPAR = 64 * 24 * 4 * PUNIT;
constexpr size_t FPAR = A_BYTES + B_BYTES;
constexpr size_t OFF_PSTAGE = OFF_STAGE + 2 * FPAR;
constexpr size_t CSLOT = 17408;
constexpr size_t OFF_CSLOT = OFF_PSTAGE + 2 * PPAR;
constexpr size_t WS_NEED = OFF_CSLOT + 256 * CSLOT;

constexpr int NPROD = 64, NCONS = 128, NDEC = 56, GRID = NPROD + NCONS + NDEC;
constexpr int THREADS = 512;
constexpr int NSTAGE = 6;
#define DIST_V 4
constexpr int DIST = DIST_V;
constexpr int LDS_CTRL = 6 * 16384;
constexpr unsigned long long DEADLINE = 30000000ull;

typedef unsigned u32x4 __attribute__((ext_vector_type(4)));
typedef unsigned u32x2 __attribute__((ext_vector_type(2)));
typedef float f32x4 __attribute__((ext_vector_type(4)));
typedef float f32x16 __attribute__((ext_vector_type(16)));
typedef _Float16 h2 __attribute__((ext_vector_type(2)));
typedef _Float16 h8 __attribute__((ext_vector_type(8)));
#define LAS __attribute__((address_space(3)))
#define GAS __attribute__((address_space(1)))

struct Args {
  char* ws[NR];
  float* out[NR];
  const float* x;
  const float* w;
  int rank;
  int pad;
};

__device__ __forceinline__ unsigned ld_sys(const void* p) {
  return __hip_atomic_load((const unsigned*)p, __ATOMIC_RELAXED, __HIP_MEMORY_SCOPE_SYSTEM);
}
__device__ __forceinline__ unsigned ld_agent(const void* p) {
  return __hip_atomic_load((const unsigned*)p, __ATOMIC_RELAXED, __HIP_MEMORY_SCOPE_AGENT);
}
__device__ __forceinline__ void st_sys(void* p, unsigned v) {
  __hip_atomic_store((unsigned*)p, v, __ATOMIC_RELAXED, __HIP_MEMORY_SCOPE_SYSTEM);
}
__device__ __forceinline__ void st_agent(void* p, unsigned v) {
  __hip_atomic_store((unsigned*)p, v, __ATOMIC_RELAXED, __HIP_MEMORY_SCOPE_AGENT);
}
__device__ __forceinline__ void st16_sys(void* p, u32x4 v) {
  asm volatile("global_store_dwordx4 %0, %1, off sc0 sc1\n\ts_nop 1" :: "v"(p), "v"(v) : "memory");
}
__device__ __forceinline__ unsigned pk(float a, float b) {
  h2 h; h.x = (_Float16)a; h.y = (_Float16)b; return __builtin_bit_cast(unsigned, h);
}
#define BAR() do { asm volatile("" ::: "memory"); __builtin_amdgcn_s_barrier(); asm volatile("" ::: "memory"); } while (0)

typedef unsigned short us2 __attribute__((ext_vector_type(2)));
constexpr int PK_UNIT = (int)PUNIT;
constexpr int PK_LDS = 4096 + 128;
#define HDR_LANES 8
constexpr int PK_HDR_LANES = HDR_LANES;
constexpr int PK_MAXESC = 16 * PK_HDR_LANES - 1;
struct Packed { u32x4 l0, l1, c0; u32x2 c1; u32x4 hdr; };

__device__ __forceinline__ unsigned pk_sub(unsigned a, unsigned b) { return __builtin_bit_cast(unsigned, __builtin_bit_cast(us2, a) - __builtin_bit_cast(us2, b)); }
__device__ __forceinline__ unsigned pk_add(unsigned a, unsigned b) { return __builtin_bit_cast(unsigned, __builtin_bit_cast(us2, a) + __builtin_bit_cast(us2, b)); }
__device__ __forceinline__ unsigned pk_min(unsigned a, unsigned b) { return __builtin_bit_cast(unsigned, __builtin_elementwise_min(__builtin_bit_cast(us2, a), __builtin_bit_cast(us2, b))); }
template <int CTRL, int RM> __device__ __forceinline__ unsigned dpp_add(unsigned x) {
  return x + (unsigned)__builtin_amdgcn_update_dpp(0, (int)x, CTRL, RM, 0xf, false);
}
__device__ __forceinline__ unsigned wave_scan(unsigned x) {
  x = dpp_add<0x111, 0xf>(x); x = dpp_add<0x112, 0xf>(x); x = dpp_add<0x114, 0xf>(x); x = dpp_add<0x118, 0xf>(x);
  x = dpp_add<0x142, 0xa>(x); x = dpp_add<0x143, 0xc>(x);
  return x;
}
__device__ __forceinline__ unsigned esc_addr(unsigned b, int lane) {
  const unsigned j = 15u - (b & 15u);
  return (j >> 2) * 1024u + lane * 16u + (j & 3u) * 4u + (b >> 4) * 2u + 1u;
}

__device__ __forceinline__ bool pk_encode(const u32x4 (&fr)[4], Packed& p, int lane, LAS char* lds) {
  unsigned d[16];
#pragma unroll
  for (int j = 0; j < 16; ++j) d[j] = fr[j >> 2][j & 3];
  unsigned s = 0;
#pragma unroll
  for (int j = 0; j < 16; j += 5) s += ((d[j] >> 10) & 31u) + ((d[j] >> 26) & 31u);
  s = __builtin_amdgcn_readlane(wave_scan(s), 63);
  int top = (int)((s + 1388u) >> 9);
  int base = top - 6; base = base < 0 ? 0 : (base > 24 ? 24 : base);
  const unsigned base2 = (unsigned)base * 0x00010001u;
  unsigned q[16], acc = 0;
#pragma unroll
  for (int j = 0; j < 16; ++j) {
    const unsigned x = d[j] >> 10;
    const unsigned c = pk_min(pk_sub(x & 0x001f001fu, base2), 0x00070007u);
    const unsigned c6 = (x & 0x00200020u) | ((d[j] >> 5) & 0x00180018u) | c;
    q[j] = (c6 & 0x3fu) | (c6 >> 10);
    acc = pk_add(pk_add(acc, acc), (pk_add(c, 0x00010001u) >> 3) & 0x00010001u);
  }
#pragma unroll
  for (int i = 0; i < 4; ++i) {
    p.l0[i] = __builtin_amdgcn_perm(d[2 * i + 1], d[2 * i], 0x06040200u);
    p.l1[i] = __builtin_amdgcn_perm(d[2 * i + 9], d[2 * i + 8], 0x06040200u);
  }
  p.c0[0] = q[0] | (q[1] << 12) | (q[2] << 24);
  p.c0[1] = (q[2] >> 8) | (q[3] << 4) | (q[4] << 16) | (q[5] << 28);
  p.c0[2] = (q[5] >> 4) | (q[6] << 8) | (q[7] << 20);
  p.c0[3] = q[8] | (q[9] << 12) | (q[10] << 24);
  p.c1[0] = (q[10] >> 8) | (q[11] << 4) | (q[12] << 16) | (q[13] << 28);
  p.c1[1] = (q[13] >> 4) | (q[14] << 8) | (q[15] << 20);
  const unsigned cnt = (unsigned)__builtin_popcount(acc);
  const unsigned incl = wave_scan(cnt);
  const unsigned total = __builtin_amdgcn_readlane(incl, 63);
  if (total > (unsigned)PK_MAXESC) { p.hdr = (u32x4){0x80u, 0u, 0u, 0u}; return true; }
  LAS char* list = lds + 4096;
  if (total != 0u) {
#pragma unroll
    for (int f = 0; f < 4; ++f) *(LAS u32x4*)(lds + f * 1024 + lane * 16) = fr[f];
    unsigned off = 1u + incl - cnt;
    while (acc != 0u) {
      const unsigned b = (unsigned)__builtin_ctz(acc);
      acc &= acc - 1u;
      const unsigned hb = *(const LAS unsigned char*)(lds + esc_addr(b, lane));
      *(LAS unsigned char*)(list + off) = (unsigned char)((hb >> 2) & 31u);
      ++off;
    }
  }
  if (lane == 0) *(LAS unsigned char*)list = (unsigned char)base;
  p.hdr = *(const LAS u32x4*)(list + (lane & 7) * 16);
  return false;
}

__device__ __forceinline__ void pk_decode(const Packed& p, unsigned h, u32x4 (&fr)[4], int lane, LAS char* lds) {
  const unsigned base2 = (__builtin_amdgcn_readfirstlane(h) & 31u) * 0x00010001u;
  unsigned q[16];
  q[0] = p.c0[0] & 0xfffu; q[1] = (p.c0[0] >> 12) & 0xfffu; q[2] = __builtin_amdgcn_alignbit(p.c0[1], p.c0[0], 24) & 0xfffu;
  q[3] = (p.c0[1] >> 4) & 0xfffu; q[4] = (p.c0[1] >> 16) & 0xfffu; q[5] = __builtin_amdgcn_alignbit(p.c0[2], p.c0[1], 28) & 0xfffu;
  q[6] = (p.c0[2] >> 8) & 0xfffu; q[7] = p.c0[2] >> 20;
  q[8] = p.c0[3] & 0xfffu; q[9] = (p.c0[3] >> 12) & 0xfffu; q[10] = __builtin_amdgcn_alignbit(p.c1[0], p.c0[3], 24) & 0xfffu;
  q[11] = (p.c1[0] >> 4) & 0xfffu; q[12] = (p.c1[0] >> 16) & 0xfffu; q[13] = __builtin_amdgcn_alignbit(p.c1[1], p.c1[0], 28) & 0xfffu;
  q[14] = (p.c1[1] >> 8) & 0xfffu; q[15] = p.c1[1] >> 20;
  unsigned acc = 0;
#pragma unroll
  for (int j = 0; j < 16; ++j) {
    const unsigned c6 = (q[j] | (q[j] << 10)) & 0x003f003fu;
    const unsigned c = c6 & 0x00070007u;
    const unsigned hi8 = ((c6 << 2) & 0x00800080u) | ((c6 >> 3) & 0x00030003u) | (pk_add(c, base2) << 2);
    const unsigned l = (j < 8 ? p.l0 : p.l1)[(j & 7) >> 1];
    fr[j >> 2][j & 3] = __builtin_amdgcn_perm(hi8, l, (j & 1) ? 0x06030402u : 0x06010400u);
    acc = pk_add(pk_add(acc, acc), (pk_add(c, 0x00010001u) >> 3) & 0x00010001u);
  }
  const unsigned cnt = (unsigned)__builtin_popcount(acc);
  const unsigned incl = wave_scan(cnt);
  if (__builtin_amdgcn_readlane(incl, 63) != 0u) {
    LAS char* list = lds + 4096;
    if (lane < 32) *(LAS unsigned*)(list + lane * 4) = h;
#pragma unroll
    for (int f = 0; f < 4; ++f) *(LAS u32x4*)(lds + f * 1024 + lane * 16) = fr[f];
    unsigned off = 1u + incl - cnt;
    while (acc != 0u) {
      const unsigned b = (unsigned)__builtin_ctz(acc);
      acc &= acc - 1u;
      LAS unsigned char* t = (LAS unsigned char*)(lds + esc_addr(b, lane));
      const unsigned e = *(const LAS unsigned char*)(list + off);
      *t = (unsigned char)((*t & 0x83u) | (e << 2));
      ++off;
    }
#pragma unroll
    for (int f = 0; f < 4; ++f) fr[f] = *(const LAS u32x4*)(lds + f * 1024 + lane * 16);
  }
}

struct Piece { int kc, tile; };
__device__ __forceinline__ Piece decode(int n) {
  Piece p;
  constexpr int nf = LATE * 24, n2 = nf + (8 - LATE) * 20;
  int i;
  if (n < nf) { p.kc = n / 24; i = n % 24; }
  else if (n < n2) { const int v = n - nf; p.kc = LATE + v / 20; i = v % 20; }
  else { const int v = n - n2; p.kc = LATE + v / 4; i = 20 + v % 4; }
  p.tile = i < 16 ? 8 + i : (i < 20 ? i - 12 : i - 20);
  return p;
}
#define FIRSTD_V 7
__device__ __forceinline__ bool is_direct(const Piece& p, int n) { return n < FIRSTD_V || p.kc >= KD || (p.tile < 4 && p.kc >= LATE); }
__device__ __forceinline__ size_t stage_off(const Piece& p, int cs) {
  return p.tile >= 8 ? A_BYTES + ((size_t)cs * 16 + (p.tile - 8)) * PIECE : ((size_t)cs * 8 + p.tile) * PIECE;
}
struct Src { const char* base; unsigned ustep, loff, js; };
__device__ __forceinline__ Src src_of(const Args& a, int d, const Piece& p, int lane) {
  Src s;
  const unsigned i = lane & 31, h = lane >> 5;
  if (p.tile >= 8) {
    s.base = (const char*)a.w + ((size_t)(p.kc * 64) * NCOL + (d >> 2) * 2048 + (p.tile - 8) * 128) * 4;
    s.ustep = NCOL * 4; s.loff = h * (4 * NCOL * 4) + i * 16; s.js = 8 * NCOL * 4;
  } else {
    const int row0 = p.tile < 4 ? d * 512 + p.tile * 128 : (((d >> 2) ^ 1) * 4 + (p.tile - 4)) * 512 + (d & 3) * 128;
    s.base = (const char*)a.x + ((size_t)row0 * KSH + p.kc * 64) * 4;
    s.ustep = 32 * KSH * 4; s.loff = i * (KSH * 4) + h * 16; s.js = 32;
  }
  return s;
}
__device__ __forceinline__ void load8(f32x4 (&r)[8], const char* base, unsigned loff, unsigned js) {
  unsigned o[8];
#pragma unroll
  for (int j = 0; j < 8; ++j) o[j] = loff + j * js;
  asm volatile(
      "s_nop 4\n\t"
      "global_load_dwordx4 %0, %8, %16\n\t"
      "global_load_dwordx4 %1, %9, %16\n\t"
      "global_load_dwordx4 %2, %10, %16\n\t"
      "global_load_dwordx4 %3, %11, %16\n\t"
      "global_load_dwordx4 %4, %12, %16\n\t"
      "global_load_dwordx4 %5, %13, %16\n\t"
      "global_load_dwordx4 %6, %14, %16\n\t"
      "global_load_dwordx4 %7, %15, %16"
      : "=&v"(r[0]), "=&v"(r[1]), "=&v"(r[2]), "=&v"(r[3]), "=&v"(r[4]), "=&v"(r[5]), "=&v"(r[6]), "=&v"(r[7])
      : "v"(o[0]), "v"(o[1]), "v"(o[2]), "v"(o[3]), "v"(o[4]), "v"(o[5]), "v"(o[6]), "v"(o[7]), "s"(base)
      : "memory");
}
#define WAIT_R(N, r) asm volatile("s_waitcnt vmcnt(" #N ")" : "+v"(r[0]), "+v"(r[1]), "+v"(r[2]), "+v"(r[3]), \
    "+v"(r[4]), "+v"(r[5]), "+v"(r[6]), "+v"(r[7]) :: "memory")
__device__ __forceinline__ void cvt4(const f32x4 (&r)[8], u32x4 (&fr)[4]) {
#pragma unroll
  for (int c = 0; c < 4; ++c)
    fr[c] = (u32x4){pk(r[0][c], r[1][c]), pk(r[2][c], r[3][c]), pk(r[4][c], r[5][c]), pk(r[6][c], r[7][c])};
}
__device__ __forceinline__ void store4(char* dbase, unsigned voff, const u32x4 (&fr)[4]) {
  asm volatile(
      "s_nop 4\n\t"
      "global_store_dwordx4 %0, %1, %5 sc0 sc1\n\t"
      "global_store_dwordx4 %0, %2, %5 offset:1024 sc0 sc1\n\t"
      "global_store_dwordx4 %0, %3, %5 offset:2048 sc0 sc1\n\t"
      "global_store_dwordx4 %0, %4, %5 offset:3072 sc0 sc1\n\t"
      "s_nop 1"
      :: "v"(voff), "v"(fr[0]), "v"(fr[1]), "v"(fr[2]), "v"(fr[3]), "s"(dbase) : "memory");
}
__device__ __forceinline__ void store_packed(char* dbase, int lane, const Packed& p) {
  asm volatile(
      "s_nop 4\n\t"
      "global_store_dwordx4 %0, %2, %6 sc0 sc1\n\t"
      "global_store_dwordx4 %0, %3, %6 offset:1024 sc0 sc1\n\t"
      "global_store_dwordx4 %0, %4, %6 offset:2048 sc0 sc1\n\t"
      "global_store_dwordx2 %1, %5, %6 offset:3072 sc0 sc1\n\t"
      "s_nop 1"
      :: "v"(lane * 16), "v"(lane * 8), "v"(p.l0), "v"(p.l1), "v"(p.c0), "v"(p.c1), "s"(dbase) : "memory");
}

__device__ __forceinline__ void queue_wave(const Args& a, unsigned call, int d, int slot, int lane,
                                           LAS unsigned* seq, LAS unsigned* prog) {
  unsigned* ctr = (unsigned*)(a.ws[a.rank] + OFF_QCNT + (size_t)d * 128);
  for (unsigned i = 0;; ++i) {
    const unsigned cur = i == 0 ? (unsigned)slot : i == 1 ? (unsigned)(slot + NSLOT)
                                : __builtin_amdgcn_readfirstlane(*(volatile LAS unsigned*)(seq + i));
    if (cur >= 192u) break;
    while (__builtin_amdgcn_readfirstlane(*(volatile LAS unsigned*)prog) <= i) __builtin_amdgcn_s_sleep(2);
    if (lane == 0) {
      const unsigned t = __hip_atomic_fetch_add(ctr, 1u, __ATOMIC_RELAXED, __HIP_MEMORY_SCOPE_AGENT) - (call - 1u) * 192u;
      *(volatile LAS unsigned*)(seq + i + 2) = 2u * NSLOT + t;
    }
    asm volatile("s_waitcnt vmcnt(0) lgkmcnt(0)" ::: "memory");
  }
}

__device__ __forceinline__ unsigned producer(const Args& a, int d, int slot, int u, int lane,
                                             LAS char* scratch, LAS unsigned* cnt, LAS unsigned* seq, LAS unsigned* prog) {
  constexpr int NP = 192;
  char* dws = a.ws[d];
  const unsigned voff = lane * 16;
  f32x4 r[8];
  u32x4 fr[4];
  Piece p = decode(slot);
  Src s = src_of(a, d, p, lane);
  load8(r, s.base + u * s.ustep, s.loff, s.js);
  unsigned call_v;
  asm volatile("global_load_dword %0, %1, %2 sc1" : "=&v"(call_v) : "v"(0u), "s"(a.ws[a.rank] + OFF_EPOCH) : "memory");
  BAR();
  WAIT_R(0, r);
  asm volatile("s_nop 0" : "+v"(call_v));
  const unsigned call = __builtin_amdgcn_readfirstlane(call_v) + 1u;
  char* pstage = dws + OFF_PSTAGE + (size_t)(call & 1u) * PPAR;
  char* fstage = dws + OFF_STAGE + (size_t)(call & 1u) * FPAR;
  char* prevflag = nullptr;
  bool prevdirect = false;
  unsigned long long tnext = 0;
  int i = 0;
  auto publish = [&](int idx) -> int {
    unsigned old = 0;
    if (lane == 0) old = __hip_atomic_fetch_add(cnt + idx, 1u, __ATOMIC_RELAXED, __HIP_MEMORY_SCOPE_WORKGROUP);
    if (__builtin_amdgcn_readfirstlane(old) != 3u) return 0;
    if (lane == 0) { if (prevdirect) st16_sys(prevflag, (u32x4){call, call, call, call}); else st_sys(prevflag, call); }
    return 1;
  };
  int after = 4;
  int n = slot;
  for (; n < NP; ++i) {
    if (u == 0 && lane == 0) *(volatile LAS unsigned*)prog = (unsigned)i + 1u;
    int nx;
    if (i == 0) nx = slot + NSLOT;
    else {
      unsigned v;
      while ((v = __builtin_amdgcn_readfirstlane(*(volatile LAS unsigned*)(seq + i + 1))) == 0xffffffffu) __builtin_amdgcn_s_sleep(1);
      nx = (int)v;
    }
    const int nn = nx < NP ? nx : n;
    const Piece pn = decode(nn);
    const Src sn = src_of(a, d, pn, lane);
    const int cs = a.rank * 8 + p.kc;
    const bool direct = is_direct(p, n);
    asm volatile("s_waitcnt vmcnt(6)\n\ts_cmp_gt_u32 %8, 5\n\ts_cbranch_scc1 1f\n\ts_waitcnt vmcnt(5)\n\t"
                 "s_cmp_gt_u32 %8, 4\n\ts_cbranch_scc1 1f\n\ts_waitcnt vmcnt(4)\n1:"
                 : "+v"(r[0]), "+v"(r[1]), "+v"(r[2]), "+v"(r[3]), "+v"(r[4]), "+v"(r[5]), "+v"(r[6]), "+v"(r[7])
                 : "s"(__builtin_amdgcn_readfirstlane(after)) : "scc", "memory");
    cvt4(r, fr);
    load8(r, sn.base + u * sn.ustep, sn.loff, sn.js);
    Packed pk;
    bool raw = true;
    if (!direct) raw = pk_encode(fr, pk, lane, scratch);
    asm volatile("s_waitcnt vmcnt(8)" ::: "memory");
    after = (direct ? 4 : 5) + (prevflag != nullptr ? publish(i - 1) : 0);
    {
      unsigned long long now = __builtin_amdgcn_s_memrealtime();
      while (now < tnext) { __builtin_amdgcn_s_sleep(1); now = __builtin_amdgcn_s_memrealtime(); }
      tnext = now + (direct ? PACE_D : PACE_P);
    }
    if (direct) {
      store4(fstage + stage_off(p, cs) + u * 4096, voff, fr);
      prevflag = dws + OFF_LFLAG + (size_t)(p.tile * 64 + cs) * 16;
    } else {
      char* dst = pstage + (size_t)((cs * 24 + p.tile) * 4 + u) * PUNIT;
      if (raw) store4(dst, voff, fr); else store_packed(dst, lane, pk);
      if (lane < PK_HDR_LANES) st16_sys(dst + 4096 + lane * 16, pk.hdr);
      prevflag = dws + OFF_PFLAG + (size_t)(p.tile * 64 + cs) * 4;
    }
    prevdirect = direct;
    p = pn; s = sn; n = nx;
  }
  asm volatile("s_waitcnt vmcnt(0)" : "+v"(r[0]) :: "memory");
  publish(i - 1);
  return call;
}

__device__ __forceinline__ void decoder(const Args& a, unsigned call, int p, int j, int gw, int lane, LAS char* scratch) {
  char* myws = a.ws[a.rank];
  const char* pstage = myws + OFF_PSTAGE + (size_t)(call & 1u) * PPAR;
  const int u = j & 3;
  for (int U = gw; U < 768; U += 448) {
    const Piece pc = decode(U >> 2);
    const Src s = src_of(a, a.rank, pc, lane);
    const int cs = a.rank * 8 + pc.kc;
    f32x4 r[8]; u32x4 fr[4];
    load8(r, s.base + (U & 3) * s.ustep, s.loff, s.js);
    WAIT_R(0, r);
    cvt4(r, fr);
    store4(myws + OFF_STAGE + (size_t)(call & 1u) * FPAR + stage_off(pc, cs) + (U & 3) * 4096, lane * 16, fr);
    asm volatile("s_waitcnt vmcnt(0)" ::: "memory");
    if (lane == 0) st_agent(myws + OFF_LFLAG + (size_t)((pc.tile * 64 + cs) * 4 + (U & 3)) * 4, call);
  }
  for (int k = 0; k < 12; ++k) {
    const Piece pc = decode((j >> 2) + 16 * k);
    if (is_direct(pc, (j >> 2) + 16 * k)) continue;
    const int cs = p * 8 + pc.kc;
    const char* flag = myws + OFF_PFLAG + (size_t)(pc.tile * 64 + cs) * 4;
    {
      const unsigned long long t0 = __builtin_amdgcn_s_memrealtime();
      bool bad = false;
      for (;;) {
        const unsigned fv = ld_sys(flag), tm = ld_agent(myws + OFF_TMO);
        if (__builtin_amdgcn_readfirstlane(fv) >= call) break;
        if (__builtin_amdgcn_readfirstlane(tm) == call || __builtin_amdgcn_s_memrealtime() - t0 > DEADLINE) {
          if (lane == 0) st_agent(myws + OFF_TMO, call);
          bad = true;
          break;
        }
        __builtin_amdgcn_s_sleep(2);
      }
      if (bad) return;
    }
    const char* src = pstage + (size_t)((cs * 24 + pc.tile) * 4 + u) * PUNIT;
    Packed pk; unsigned h;
    asm volatile("s_nop 4\n\t"
                 "global_load_dword %4, %7, %8 offset:0 sc0 sc1\n\t"
                 "global_load_dwordx4 %2, %5, %8 offset:2048 sc0 sc1\n\t"
                 "global_load_dwordx2 %3, %6, %8 offset:3072 sc0 sc1\n\t"
                 "global_load_dwordx4 %0, %5, %8 sc0 sc1\n\t"
                 "global_load_dwordx4 %1, %5, %8 offset:1024 sc0 sc1\n\t"
                 "s_waitcnt vmcnt(0)"
                 : "=&v"(pk.l0), "=&v"(pk.l1), "=&v"(pk.c0), "=&v"(pk.c1), "=&v"(h)
                 : "v"(lane * 16), "v"(lane * 8), "v"(4096 + (lane & 31) * 4), "s"(src) : "memory");
    u32x4 fr[4];
    if (__builtin_amdgcn_readfirstlane(h) & 0x80u) {
      asm volatile("global_load_dwordx4 %0, %4, %5 sc0 sc1\n\t"
                   "global_load_dwordx4 %1, %4, %5 offset:1024 sc0 sc1\n\t"
                   "global_load_dwordx4 %2, %4, %5 offset:2048 sc0 sc1\n\t"
                   "global_load_dwordx4 %3, %4, %5 offset:3072 sc0 sc1\n\t"
                   "s_waitcnt vmcnt(0)"
                   : "=&v"(fr[0]), "=&v"(fr[1]), "=&v"(fr[2]), "=&v"(fr[3]) : "v"(lane * 16), "s"(src) : "memory");
    } else {
      pk_decode(pk, h, fr, lane, scratch);
    }
    char* dst = myws + OFF_STAGE + (size_t)(call & 1u) * FPAR + stage_off(pc, cs);
    store4(dst + u * 4096, lane * 16, fr);
    asm volatile("s_waitcnt vmcnt(0)" ::: "memory");
    if (lane == 0) st_agent(myws + OFF_LFLAG + (size_t)((pc.tile * 64 + cs) * 4 + u) * 4, call);
  }
}

__device__ __forceinline__ void expander(const Args& a, unsigned call, int ti, int part, int half, int lane, LAS char* buf) {
  char* myws = a.ws[a.rank];
  const int slot = ti * 4 + part;
  const char* flag = myws + OFF_OFLAG + (size_t)slot * 4;
  bool bad = false;
  {
    const unsigned long long t0 = __builtin_amdgcn_s_memrealtime();
    for (;;) {
      const unsigned fv = ld_sys(flag), tm = ld_agent(myws + OFF_TMO);
      if (__builtin_amdgcn_readfirstlane(fv) >= call) break;
      if (__builtin_amdgcn_readfirstlane(tm) == call || __builtin_amdgcn_s_memrealtime() - t0 > DEADLINE) {
        if (lane == 0) st_agent(myws + OFF_TMO, call);
        bad = true;
        break;
      }
      __builtin_amdgcn_s_sleep(1);
    }
  }
  const char* src = myws + OFF_CSLOT + (size_t)slot * CSLOT + lane * 16;
  if (!bad) {
    __builtin_amdgcn_global_load_lds((const GAS void*)(src + 16384), (LAS void*)(buf + 16384), 16, 0, 17);
    if (half == 0) {
#pragma unroll
      for (int i = 0; i < 5; ++i)
        __builtin_amdgcn_global_load_lds((const GAS void*)(src + i * 1024), (LAS void*)(buf + i * 1024), 16, 0, 17);
    } else {
#pragma unroll
      for (int i = 3; i < 10; ++i)
        __builtin_amdgcn_global_load_lds((const GAS void*)(src + i * 1024), (LAS void*)(buf + i * 1024), 16, 0, 17);
    }
  }
  const unsigned tm = __builtin_amdgcn_readfirstlane(ld_agent(myws + OFF_TMO));
  asm volatile("s_waitcnt vmcnt(0)" ::: "memory");
  bad = bad || tm == call;
  float* o = a.out[a.rank] + (size_t)((ti >> 4) * 128 + part * 32 + half * 16 + 4 * (lane >> 5)) * NCOL +
             ((a.rank >> 2) ^ 1) * 2048 + (ti & 15) * 128 + 4 * (lane & 31);
  if (bad) {
    for (int e = 0; e < 8; ++e)
      st16_sys(o + (size_t)((e & 3) + 8 * (e >> 2)) * NCOL, (u32x4){0x7fc00000u, 0x7fc00000u, 0x7fc00000u, 0x7fc00000u});
    return;
  }
  const u32x2 m = *(const LAS u32x2*)(buf + 16384 + lane * 8);
  const u32x2 cnt = *(const LAS u32x2*)(buf + 16896);
  const unsigned n0 = __builtin_amdgcn_readfirstlane(cnt[0]), total = __builtin_amdgcn_readfirstlane(cnt[1]);
  if (half == 0 ? n0 > 1280u : (n0 < 768u || total > 2560u)) {
#pragma unroll
    for (int i = 0; i < 16; ++i)
      __builtin_amdgcn_global_load_lds((const GAS void*)(src + i * 1024), (LAS void*)(buf + i * 1024), 16, 0, 17);
    asm volatile("s_waitcnt vmcnt(0)" ::: "memory");
  }
  unsigned base = half ? n0 : 0u;
  const unsigned bits = half ? m[0] : m[1];
  unsigned pos[32];
#pragma unroll
  for (int i = 0; i < 32; ++i) {
    const unsigned long long mm = __ballot((int)(bits << i) < 0);
    pos[i] = __builtin_amdgcn_mbcnt_hi((unsigned)(mm >> 32), __builtin_amdgcn_mbcnt_lo((unsigned)mm, base));
    base += (unsigned)__builtin_popcountll(mm);
  }
  float x[32];
#pragma unroll
  for (int i = 0; i < 32; ++i) x[i] = *(const LAS float*)(buf + (pos[i] & 4095u) * 4u);
#pragma unroll
  for (int e = 0; e < 8; ++e) {
    asm volatile("" : "+v"(x[4 * e]), "+v"(x[4 * e + 1]), "+v"(x[4 * e + 2]), "+v"(x[4 * e + 3]));
    f32x4 v;
#pragma unroll
    for (int c = 0; c < 4; ++c) v[c] = (int)(bits << (4 * e + c)) < 0 ? x[4 * e + c] : 0.f;
    st16_sys(o + (size_t)((e & 3) + 8 * (e >> 2)) * NCOL, __builtin_bit_cast(u32x4, v));
  }
}

__global__ void __launch_bounds__(THREADS) gemm_rs_kernel(Args a) {
  __shared__ __attribute__((aligned(16))) char lds[LDS_CTRL + 512];
  const int tid = threadIdx.x, lane = tid & 63;
  const int wv = __builtin_amdgcn_readfirstlane(tid >> 6);
  const int b = blockIdx.x, xcd = b & 7, jb = b >> 3;
  char* myws = a.ws[a.rank];
  unsigned call = 0;

  if (jb < 8) {
    LAS unsigned* sync = (LAS unsigned*)((LAS char*)lds + LDS_CTRL);
    LAS unsigned* seq = (LAS unsigned*)((LAS char*)lds + 4 * CSLOT);
    LAS unsigned* cnt = seq + 256;
    const bool streams = jb < NSLOT && xcd != a.rank;
    if (streams && wv < 4) {
      call = producer(a, xcd, jb, wv, lane, (LAS char*)lds + wv * PK_LDS, cnt, seq, sync + 8);
    } else
    {
      call = __builtin_amdgcn_readfirstlane(ld_agent(myws + OFF_EPOCH)) + 1u;
      if (wv == 7 && lane < 16) sync[lane] = 0u;
      if (wv >= 4) { seq[(wv - 4) * 64 + lane] = 0xffffffffu; cnt[(wv - 4) * 64 + lane] = 0u; }
      asm volatile("s_waitcnt lgkmcnt(0)" ::: "memory");
      BAR();
      if (streams && wv == 4) queue_wave(a, call, xcd, jb, lane, seq, sync + 8);
    }
    BAR();
    expander(a, call, jb * 8 + xcd, wv >> 1, wv & 1, lane, (LAS char*)lds + (wv >> 1) * CSLOT);
  } else if (jb >= 24) {
    call = __builtin_amdgcn_readfirstlane(ld_agent(myws + OFF_EPOCH)) + 1u;
    decoder(a, call, (a.rank + 1 + (jb - 24)) & 7, xcd * 8 + wv, (jb - 24) * 64 + xcd * 8 + wv, lane, (LAS char*)lds + wv * PK_LDS);
    BAR();
  } else {
    const int jc = jb - 8;
    const int rt = (xcd & 3) + (jc >> 3) * 4;
    const int ct = (xcd >> 2) * 8 + (jc & 7);
    call = __builtin_amdgcn_readfirstlane(ld_agent(myws + OFF_EPOCH)) + 1u;
    LAS unsigned* lds_bad = (LAS unsigned*)((LAS char*)lds + LDS_CTRL);
    LAS unsigned* lds_sched = (LAS unsigned*)((LAS char*)lds + LDS_CTRL + 64);
    const char* stage = myws + OFF_STAGE + (size_t)(call & 1u) * FPAR;
    if (wv == 6) {
      const char* fa = myws + OFF_LFLAG + (size_t)(rt * 64 + lane) * 16;
      const char* fb = myws + OFF_LFLAG + (size_t)((8 + ct) * 64 + lane) * 16;
      const char* ptm = myws + OFF_TMO;
      unsigned long long ready = 0, done = 0;
      for (int c = 0; c < 64; ++c) {
        if ((ready & ~done) == 0ull) {
          const unsigned long long t0 = __builtin_amdgcn_s_memrealtime();
          for (;;) {
            u32x4 va, vb; unsigned tm;
            asm volatile("global_load_dwordx4 %0, %3, off sc0 sc1\n\t"
                         "global_load_dwordx4 %1, %4, off sc0 sc1\n\t"
                         "global_load_dword %2, %5, off sc1\n\t"
                         "s_waitcnt vmcnt(0)"
                         : "=&v"(va), "=&v"(vb), "=&v"(tm) : "v"(fa), "v"(fb), "v"(ptm) : "memory");
            const unsigned lo = min(min(min(va[0], va[1]), min(va[2], va[3])), min(min(vb[0], vb[1]), min(vb[2], vb[3])));
            ready = __ballot(lo >= call);
            if ((ready & ~done) != 0ull) break;
            if (__builtin_amdgcn_readfirstlane(tm) == call || __builtin_amdgcn_s_memrealtime() - t0 > DEADLINE) {
              if (lane == 0) st_agent(myws + OFF_TMO, call);
              ready = ~0ull;
              break;
            }
            __builtin_amdgcn_s_sleep(1);
          }
        }
        const int cs = __builtin_ctzll(ready & ~done);
        done |= 1ull << cs;
        if (lane == 0) lds_sched[c & 7] = (unsigned)cs;
        if (c == 63) {
          const unsigned t = __builtin_amdgcn_readfirstlane(ld_agent(myws + OFF_TMO));
          if (lane == 0) *lds_bad = (t == call) ? 1u : 0u;
        }
        asm volatile("s_waitcnt lgkmcnt(0)" ::: "memory");
        BAR();
      }
      for (int i = 0; i <= DIST; ++i) BAR();
      asm volatile("s_waitcnt vmcnt(0)" ::: "memory");
    } else if (wv == 7) {
    } else if (wv >= 4) {
      const int lw = wv - 4;
      const char* gbase = stage + A_BYTES + (size_t)ct * PIECE + lw * 8192 + lane * 16;
      BAR();
      int st = 0;
      for (int c = 0; c < 64 + DIST; ++c) {
        if (c < 64) {
          const int cs = __builtin_amdgcn_readfirstlane(*(volatile LAS unsigned*)(lds_sched + (c & 7))) & 63;
          const char* ga = gbase + (size_t)cs * (16 * PIECE);
          LAS char* la = (LAS char*)lds + st * 16384 + lw * 8192;
#pragma unroll
          for (int f = 0; f < 8; ++f)
            __builtin_amdgcn_global_load_lds((const GAS void*)(ga + f * 1024), (LAS void*)(la + f * 1024), 16, 0, 17);
          if constexpr (DIST == 4) asm volatile("s_waitcnt vmcnt(16)" ::: "memory");
          else asm volatile("s_waitcnt vmcnt(24)" ::: "memory");
        } else if (c + 4 - DIST == 63) {
          asm volatile("s_waitcnt vmcnt(16)" ::: "memory");
        } else if (c + 4 - DIST == 64) {
          asm volatile("s_waitcnt vmcnt(8)" ::: "memory");
        } else {
          asm volatile("s_waitcnt vmcnt(0)" ::: "memory");
        }
        st = st == NSTAGE - 1 ? 0 : st + 1;
        BAR();
      }
    } else {
      f32x16 acc[4];
#pragma unroll
      for (int c = 0; c < 4; ++c)
#pragma unroll
        for (int e = 0; e < 16; ++e) acc[c][e] = 0.f;
      const __amdgpu_buffer_rsrc_t rsa = __builtin_amdgcn_make_buffer_rsrc((void*)(stage + (size_t)rt * PIECE), 0, (int)A_BYTES, 0x00020000);
      const unsigned aoff = wv * 4096 + lane * 16;
      h8 ar[NSTAGE][4];
      h8 bf[2][4];
#define SCHED_READ(C) (*(volatile LAS unsigned*)(lds_sched + ((C) & 7)))
#define LOAD_A(SLOT, SV) do {                                                                            \
        const unsigned cs_ = __builtin_amdgcn_readfirstlane(SV) & 63u;                                   \
        _Pragma("unroll") for (int s_ = 0; s_ < 4; ++s_)                                                 \
          ar[SLOT][s_] = __builtin_bit_cast(h8, __builtin_amdgcn_raw_buffer_load_b128(rsa, aoff + s_ * 1024, cs_ * (unsigned)(8 * PIECE), 17)); \
      } while (0)
#define READ_B(BUF, ST, S) do {                                                                          \
        _Pragma("unroll") for (int c_ = 0; c_ < 4; ++c_)                                                 \
          bf[BUF][c_] = *(const LAS h8*)((const LAS char*)lds + (ST) * 16384 + (S) * 4096 + c_ * 1024 + lane * 16); \
      } while (0)
#define MMA4(SLOT, S) do {                                                                               \
        __builtin_amdgcn_sched_barrier(0);                                                               \
        _Pragma("unroll") for (int c_ = 0; c_ < 4; ++c_)                                                 \
          acc[c_] = __builtin_amdgcn_mfma_f32_32x32x16_f16(ar[SLOT][S], bf[(S) & 1][c_], acc[c_], 0, 0, 0); \
        __builtin_amdgcn_sched_barrier(0);                                                               \
      } while (0)
#define STEP(K, T, MORE) do {                                                                            \
        READ_B(1, K, 1);                                                                                 \
        unsigned sv_ = 0; if (MORE) sv_ = SCHED_READ((T) + DIST);                                        \
        MMA4(K, 0);                                                                                      \
        READ_B(0, K, 2);                                                                                 \
        if (MORE) LOAD_A(((K) + DIST) % NSTAGE, sv_);                                                    \
        MMA4(K, 1);                                                                                      \
        READ_B(1, K, 3);                                                                                 \
        MMA4(K, 2);                                                                                      \
        READ_B(0, ((K) + 1) % NSTAGE, 0);                                                                \
        MMA4(K, 3);                                                                                      \
        asm volatile("s_waitcnt lgkmcnt(4)" ::: "memory");        \
        BAR();                                                                                           \
      } while (0)
      BAR(); LOAD_A(0, SCHED_READ(0)); BAR(); LOAD_A(1, SCHED_READ(1)); BAR(); LOAD_A(2, SCHED_READ(2));
      BAR(); LOAD_A(3, SCHED_READ(3));
      if constexpr (DIST == 5) { BAR(); LOAD_A(4, SCHED_READ(4)); }
      BAR();
      READ_B(0, 0, 0);
      for (int t = 0; t < 54; t += 6) {
        STEP(0, t, true); STEP(1, t + 1, true); STEP(2, t + 2, true);
        STEP(3, t + 3, true); STEP(4, t + 4, true); STEP(5, t + 5, true);
      }
      STEP(0, 54, true); STEP(1, 55, true); STEP(2, 56, true); STEP(3, 57, true); STEP(4, 58, true);
      STEP(5, 59, 59 + DIST <= 63);
      STEP(0, 60, false); STEP(1, 61, false); STEP(2, 62, false); STEP(3, 63, false);
#undef SCHED_READ
#undef LOAD_A
#undef READ_B
#undef MMA4
#undef STEP
      const bool bad = *lds_bad != 0u;
      const int g = a.rank >> 2;
      if (rt < 4) {
        const float qnan = __builtin_nanf("");
        float* o = a.out[a.rank] + (size_t)(rt * 128 + wv * 32 + 4 * (lane >> 5)) * NCOL + g * 2048 + ct * 128 + 4 * (lane & 31);
#pragma unroll
        for (int e = 0; e < 16; ++e) {
          f32x4 v;
#pragma unroll
          for (int c = 0; c < 4; ++c) { const float y = acc[c][e]; v[c] = bad ? qnan : (y < 0.f ? 0.f : y); }
          st16_sys(o + (size_t)((e & 3) + 8 * (e >> 2)) * NCOL, __builtin_bit_cast(u32x4, v));
        }
      } else {
        LAS char* buf = (LAS char*)lds + wv * CSLOT;
        const int q = ((g ^ 1) << 2) + (rt - 4);
        const int slot = ((a.rank & 3) * 16 + ct) * 4 + wv;
        char* dst = a.ws[q & 7] + OFF_CSLOT + (size_t)slot * CSLOT + lane * 16;
        if (bad) {
          const unsigned nb = 0x7fc00000u;
          for (int i = 0; i < 16; ++i) st16_sys(dst + i * 1024, (u32x4){nb, nb, nb, nb});
          if (lane < 32) st16_sys(dst + 16384, (u32x4){~0u, ~0u, ~0u, ~0u});
          if (lane == 32) st16_sys(dst + 16384, (u32x4){2048u, 4096u, 0u, 0u});
        } else {
          unsigned base = 0, sent = 0, n0 = 0, mhi = 0, mlo = 0;
#pragma unroll
          for (int e4 = 0; e4 < 4; ++e4) {
#pragma unroll
            for (int ee = 0; ee < 4; ++ee) {
#pragma unroll
              for (int c = 0; c < 4; ++c) {
                const float y = acc[c][e4 * 4 + ee];
                const bool nz = !(y <= 0.f);
                const unsigned long long m = __ballot(nz);
                unsigned pos = __builtin_amdgcn_mbcnt_hi((unsigned)(m >> 32), __builtin_amdgcn_mbcnt_lo((unsigned)m, base));
                asm volatile("" : "+v"(pos));
                *(LAS float*)(buf + 4u * (nz ? pos : 4288u + lane)) = y;
                if (e4 < 2) mhi = mhi * 2u + (nz ? 1u : 0u); else mlo = mlo * 2u + (nz ? 1u : 0u);
                base += (unsigned)__builtin_popcountll(m);
              }
            }
            if (e4 == 1) n0 = base;
            asm volatile("s_waitcnt vmcnt(0)" ::: "memory");
            for (; sent + 256u <= base; sent += 256u)
              st16_sys(dst + sent * 4u, *(const LAS u32x4*)(buf + sent * 4u + lane * 16));
          }
          if (sent + lane * 4u < base) st16_sys(dst + sent * 4u, *(const LAS u32x4*)(buf + sent * 4u + lane * 16));
          *(LAS u32x2*)(buf + 16384 + lane * 8) = (u32x2){mlo, mhi};
          if (lane == 0) *(LAS u32x2*)(buf + 16896) = (u32x2){n0, base};
          if (lane < 33) st16_sys(dst + 16384, *(const LAS u32x4*)(buf + 16384 + lane * 16));
        }
        asm volatile("s_waitcnt vmcnt(0)" ::: "memory");
        if (lane == 0) st_sys(a.ws[q & 7] + OFF_OFLAG + (size_t)slot * 4, call);
      }
    }
  }
  if (tid == 0) {
    const unsigned t = __hip_atomic_fetch_add((unsigned*)(myws + OFF_ENDCNT), 1u, __ATOMIC_RELAXED, __HIP_MEMORY_SCOPE_AGENT);
    if (t % GRID == GRID - 1) st_agent(myws + OFF_EPOCH, call);
  }
}

}

extern "C" void kernel_launch(int rank, int nranks, ncclComm_t comm,
                              hipStream_t stream,
                              void* const* d_in, const int* in_sizes, int n_in,
                              void* d_out, int out_size,
                              const int* in_dtypes, int out_dtype,
                              void* const* d_out_all,
                              void* const* d_ws_all, size_t ws_bytes) {
  (void)comm; (void)in_dtypes; (void)out_dtype; (void)d_out;
  if (nranks != NR || (unsigned)rank >= (unsigned)NR || n_in < 2 || ws_bytes < WS_NEED ||
      in_sizes[0] != 4096 * KSH || in_sizes[1] != KSH * NCOL || out_size != 512 * NCOL) {
    fprintf(stderr, "gemm_rs: unexpected problem shape\n");
    return;
  }
  Args a;
  for (int p = 0; p < NR; ++p) { a.ws[p] = (char*)d_ws_all[p]; a.out[p] = (float*)d_out_all[p]; }
  a.x = (const float*)d_in[0];
  a.w = (const float*)d_in[1];
  a.rank = rank;
  a.pad = 0;
  gemm_rs_kernel<<<GRID, THREADS, 0, stream>>>(a);
  hipError_t e = hipGetLastError();
  if (e != hipSuccess) fprintf(stderr, "gemm_rs: launch failed: %s\n", hipGetErrorString(e));
}
```

```cpp
#include <hip/hip_runtime.h>
#include <rccl/rccl.h>
#include <cstdio>

namespace {

constexpr int NR = 8;
constexpr int NCOL = 4096;
constexpr int KSH = 512;
#define LATE_V 3
constexpr int LATE = LATE_V;
#define KD_V 7
#define NSLOT_V 7
constexpr int NSLOT = NSLOT_V;
#define PACE_V 200
constexpr unsigned PACE_P = PACE_V * NSLOT / 8, PACE_D = PACE_P * 4096 / 3712;
constexpr int KD = KD_V;

constexpr size_t OFF_EPOCH = 0;
constexpr size_t OFF_ENDCNT = 128;
constexpr size_t OFF_TMO = 256;
constexpr size_t OFF_OFLAG = 1024;
constexpr size_t OFF_QCNT = 2048;
constexpr size_t OFF_PFLAG = 4096;
constexpr size_t OFF_LFLAG = 16384;
constexpr size_t OFF_STAGE = 1u << 20;
constexpr size_t PIECE = 16384;
constexpr size_t A_BYTES = 64 * 8 * PIECE;
constexpr size_t B_BYTES = 64 * 16 * PIECE;
constexpr size_t PUNIT = 4224;
constexpr size_t PPAR = 64 * 24 * 4 * PUNIT;
constexpr size_t FPAR = A_BYTES + B_BYTES;
constexpr size_t OFF_PSTAGE = OFF_STAGE + 2 * FPAR;
constexpr size_t CSLOT = 17408;
constexpr size_t OFF_CSLOT = OFF_PSTAGE + 2 * PPAR;
constexpr size_t WS_NEED = OFF_CSLOT + 256 * CSLOT;

constexpr int NPROD = 64, NCONS = 128, NDEC = 56, GRID = NPROD + NCONS + NDEC;
constexpr int THREADS = 512;
constexpr int NSTAGE = 6;
#define DIST_V 4
constexpr int DIST = DIST_V;
constexpr int LDS_CTRL = 6 * 16384;
constexpr unsigned long long DEADLINE = 30000000ull;

typedef unsigned u32x4 __attribute__((ext_vector_type(4)));
typedef unsigned u32x2 __attribute__((ext_vector_type(2)));
typedef float f32x4 __attribute__((ext_vector_type(4)));
typedef float f32x16 __attribute__((ext_vector_type(16)));
typedef _Float16 h2 __attribute__((ext_vector_type(2)));
typedef _Float16 h8 __attribute__((ext_vector_type(8)));
#define LAS __attribute__((address_space(3)))
#define GAS __attribute__((address_space(1)))

struct Args {
  char* ws[NR];
  float* out[NR];
  const float* x;
  const float* w;
  int rank;
  int pad;
};

__device__ __forceinline__ unsigned ld_sys(const void* p) {
  return __hip_atomic_load((const unsigned*)p, __ATOMIC_RELAXED, __HIP_MEMORY_SCOPE_SYSTEM);
}
__device__ __forceinline__ unsigned ld_agent(const void* p) {
  return __hip_atomic_load((const unsigned*)p, __ATOMIC_RELAXED, __HIP_MEMORY_SCOPE_AGENT);
}
__device__ __forceinline__ void st_sys(void* p, unsigned v) {
  __hip_atomic_store((unsigned*)p, v, __ATOMIC_RELAXED, __HIP_MEMORY_SCOPE_SYSTEM);
}
__device__ __forceinline__ void st_agent(void* p, unsigned v) {
  __hip_atomic_store((unsigned*)p, v, __ATOMIC_RELAXED, __HIP_MEMORY_SCOPE_AGENT);
}
__device__ __forceinline__ void st16_sys(void* p, u32x4 v) {
  asm volatile("global_store_dwordx4 %0, %1, off sc0 sc1\n\ts_nop 1" :: "v"(p), "v"(v) : "memory");
}
__device__ __forceinline__ unsigned pk(float a, float b) {
  h2 h; h.x = (_Float16)a; h.y = (_Float16)b; return __builtin_bit_cast(unsigned, h);
}
#define BAR() do { asm volatile("" ::: "memory"); __builtin_amdgcn_s_barrier(); asm volatile("" ::: "memory"); } while (0)

typedef unsigned short us2 __attribute__((ext_vector_type(2)));
constexpr int PK_UNIT = (int)PUNIT;
constexpr int PK_LDS = 4096 + 128;
#define HDR_LANES 8
constexpr int PK_HDR_LANES = HDR_LANES;
constexpr int PK_MAXESC = 16 * PK_HDR_LANES - 1;
struct Packed { u32x4 l0, l1, c0; u32x2 c1; u32x4 hdr; };

__device__ __forceinline__ unsigned pk_sub(unsigned a, unsigned b) { return __builtin_bit_cast(unsigned, __builtin_bit_cast(us2, a) - __builtin_bit_cast(us2, b)); }
__device__ __forceinline__ unsigned pk_add(unsigned a, unsigned b) { return __builtin_bit_cast(unsigned, __builtin_bit_cast(us2, a) + __builtin_bit_cast(us2, b)); }
__device__ __forceinline__ unsigned pk_min(unsigned a, unsigned b) { return __builtin_bit_cast(unsigned, __builtin_elementwise_min(__builtin_bit_cast(us2, a), __builtin_bit_cast(us2, b))); }
template <int CTRL, int RM> __device__ __forceinline__ unsigned dpp_add(unsigned x) {
  return x + (unsigned)__builtin_amdgcn_update_dpp(0, (int)x, CTRL, RM, 0xf, false);
}
__device__ __forceinline__ unsigned wave_scan(unsigned x) {
  x = dpp_add<0x111, 0xf>(x); x = dpp_add<0x112, 0xf>(x); x = dpp_add<0x114, 0xf>(x); x = dpp_add<0x118, 0xf>(x);
  x = dpp_add<0x142, 0xa>(x); x = dpp_add<0x143, 0xc>(x);
  return x;
}
__device__ __forceinline__ unsigned esc_addr(unsigned b, int lane) {
  const unsigned j = 15u - (b & 15u);
  return (j >> 2) * 1024u + lane * 16u + (j & 3u) * 4u + (b >> 4) * 2u + 1u;
}

__device__ __forceinline__ bool pk_encode(const u32x4 (&fr)[4], Packed& p, int lane, LAS char* lds) {
  unsigned d[16];
#pragma unroll
  for (int j = 0; j < 16; ++j) d[j] = fr[j >> 2][j & 3];
  unsigned s = 0;
#pragma unroll
  for (int j = 0; j < 16; j += 5) s += ((d[j] >> 10) & 31u) + ((d[j] >> 26) & 31u);
  s = __builtin_amdgcn_readlane(wave_scan(s), 63);
  int top = (int)((s + 1388u) >> 9);
  int base = top - 6; base = base < 0 ? 0 : (base > 24 ? 24 : base);
  const unsigned base2 = (unsigned)base * 0x00010001u;
  unsigned q[16], acc = 0;
#pragma unroll
  for (int j = 0; j < 16; ++j) {
    const unsigned x = d[j] >> 10;
    const unsigned c = pk_min(pk_sub(x & 0x001f001fu, base2), 0x00070007u);
    const unsigned c6 = (x & 0x00200020u) | ((d[j] >> 5) & 0x00180018u) | c;
    q[j] = (c6 & 0x3fu) | (c6 >> 10);
    acc = pk_add(pk_add(acc, acc), (pk_add(c, 0x00010001u) >> 3) & 0x00010001u);
  }
#pragma unroll
  for (int i = 0; i < 4; ++i) {
    p.l0[i] = __builtin_amdgcn_perm(d[2 * i + 1], d[2 * i], 0x06040200u);
    p.l1[i] = __builtin_amdgcn_perm(d[2 * i + 9], d[2 * i + 8], 0x06040200u);
  }
  p.c0[0] = q[0] | (q[1] << 12) | (q[2] << 24);
  p.c0[1] = (q[2] >> 8) | (q[3] << 4) | (q[4] << 16) | (q[5] << 28);
  p.c0[2] = (q[5] >> 4) | (q[6] << 8) | (q[7] << 20);
  p.c0[3] = q[8] | (q[9] << 12) | (q[10] << 24);
  p.c1[0] = (q[10] >> 8) | (q[11] << 4) | (q[12] << 16) | (q[13] << 28);
  p.c1[1] = (q[13] >> 4) | (q[14] << 8) | (q[15] << 20);
  const unsigned cnt = (unsigned)__builtin_popcount(acc);
  const unsigned incl = wave_scan(cnt);
  const unsigned total = __builtin_amdgcn_readlane(incl, 63);
  if (total > (unsigned)PK_MAXESC) { p.hdr = (u32x4){0x80u, 0u, 0u, 0u}; return true; }
  LAS char* list = lds + 4096;
  if (total != 0u) {
#pragma unroll
    for (int f = 0; f < 4; ++f) *(LAS u32x4*)(lds + f * 1024 + lane * 16) = fr[f];
    unsigned off = 1u + incl - cnt;
    while (acc != 0u) {
      const unsigned b = (unsigned)__builtin_ctz(acc);
      acc &= acc - 1u;
      const unsigned hb = *(const LAS unsigned char*)(lds + esc_addr(b, lane));
      *(LAS unsigned char*)(list + off) = (unsigned char)((hb >> 2) & 31u);
      ++off;
    }
  }
  if (lane == 0) *(LAS unsigned char*)list = (unsigned char)base;
  p.hdr = *(const LAS u32x4*)(list + (lane & 7) * 16);
  return false;
}

__device__ __forceinline__ void pk_decode(const Packed& p, unsigned h, u32x4 (&fr)[4], int lane, LAS char* lds) {
  const unsigned base2 = (__builtin_amdgcn_readfirstlane(h) & 31u) * 0x00010001u;
  unsigned q[16];
  q[0] = p.c0[0] & 0xfffu; q[1] = (p.c0[0] >> 12) & 0xfffu; q[2] = __builtin_amdgcn_alignbit(p.c0[1], p.c0[0], 24) & 0xfffu;
  q[3] = (p.c0[1] >> 4) & 0xfffu; q[4] = (p.c0[1] >> 16) & 0xfffu; q[5] = __builtin_amdgcn_alignbit(p.c0[2], p.c0[1], 28) & 0xfffu;
  q[6] = (p.c0[2] >> 8) & 0xfffu; q[7] = p.c0[2] >> 20;
  q[8] = p.c0[3] & 0xfffu; q[9] = (p.c0[3] >> 12) & 0xfffu; q[10] = __builtin_amdgcn_alignbit(p.c1[0], p.c0[3], 24) & 0xfffu;
  q[11] = (p.c1[0] >> 4) & 0xfffu; q[12] = (p.c1[0] >> 16) & 0xfffu; q[13] = __builtin_amdgcn_alignbit(p.c1[1], p.c1[0], 28) & 0xfffu;
  q[14] = (p.c1[1] >> 8) & 0xfffu; q[15] = p.c1[1] >> 20;
  unsigned acc = 0;
#pragma unroll
  for (int j = 0; j < 16; ++j) {
    const unsigned c6 = (q[j] | (q[j] << 10)) & 0x003f003fu;
    const unsigned c = c6 & 0x00070007u;
    const unsigned hi8 = ((c6 << 2) & 0x00800080u) | ((c6 >> 3) & 0x00030003u) | (pk_add(c, base2) << 2);
    const unsigned l = (j < 8 ? p.l0 : p.l1)[(j & 7) >> 1];
    fr[j >> 2][j & 3] = __builtin_amdgcn_perm(hi8, l, (j & 1) ? 0x06030402u : 0x06010400u);
    acc = pk_add(pk_add(acc, acc), (pk_add(c, 0x00010001u) >> 3) & 0x00010001u);
  }
  const unsigned cnt = (unsigned)__builtin_popcount(acc);
  const unsigned incl = wave_scan(cnt);
  if (__builtin_amdgcn_readlane(incl, 63) != 0u) {
    LAS char* list = lds + 4096;
    if (lane < 32) *(LAS unsigned*)(list + lane * 4) = h;
#pragma unroll
    for (int f = 0; f < 4; ++f) *(LAS u32x4*)(lds + f * 1024 + lane * 16) = fr[f];
    unsigned off = 1u + incl - cnt;
    while (acc != 0u) {
      const unsigned b = (unsigned)__builtin_ctz(acc);
      acc &= acc - 1u;
      LAS unsigned char* t = (LAS unsigned char*)(lds + esc_addr(b, lane));
      const unsigned e = *(const LAS unsigned char*)(list + off);
      *t = (unsigned char)((*t & 0x83u) | (e << 2));
      ++off;
    }
#pragma unroll
    for (int f = 0; f < 4; ++f) fr[f] = *(const LAS u32x4*)(lds + f * 1024 + lane * 16);
  }
}

struct Piece { int kc, tile; };
__device__ __forceinline__ Piece decode(int n) {
  Piece p;
  constexpr int nf = LATE * 24, n2 = nf + (8 - LATE) * 20;
  int i;
  if (n < nf) { p.kc = n / 24; i = n % 24; }
  else if (n < n2) { const int v = n - nf; p.kc = LATE + v / 20; i = v % 20; }
  else { const int v = n - n2; p.kc = LATE + v / 4; i = 20 + v % 4; }
  p.tile = i < 16 ? 8 + i : (i < 20 ? i - 12 : i - 20);
  return p;
}
#define FIRSTD_V 7
__device__ __forceinline__ bool is_direct(const Piece& p, int n) { return n < FIRSTD_V || p.kc >= KD || (p.tile < 4 && p.kc >= LATE); }
__device__ __forceinline__ size_t stage_off(const Piece& p, int cs) {
  return p.tile >= 8 ? A_BYTES + ((size_t)cs * 16 + (p.tile - 8)) * PIECE : ((size_t)cs * 8 + p.tile) * PIECE;
}
struct Src { const char* base; unsigned ustep, loff, js; };
__device__ __forceinline__ Src src_of(const Args& a, int d, const Piece& p, int lane) {
  Src s;
  const unsigned i = lane & 31, h = lane >> 5;
  if (p.tile >= 8) {
    s.base = (const char*)a.w + ((size_t)(p.kc * 64) * NCOL + (d >> 2) * 2048 + (p.tile - 8) * 128) * 4;
    s.ustep = NCOL * 4; s.loff = h * (4 * NCOL * 4) + i * 16; s.js = 8 * NCOL * 4;
  } else {
    const int row0 = p.tile < 4 ? d * 512 + p.tile * 128 : (((d >> 2) ^ 1) * 4 + (p.tile - 4)) * 512 + (d & 3) * 128;
    s.base = (const char*)a.x + ((size_t)row0 * KSH + p.kc * 64) * 4;
    s.ustep = 32 * KSH * 4; s.loff = i * (KSH * 4) + h * 16; s.js = 32;
  }
  return s;
}
__device__ __forceinline__ void load8(f32x4 (&r)[8], const char* base, unsigned loff, unsigned js) {
  unsigned o[8];
#pragma unroll
  for (int j = 0; j < 8; ++j) o[j] = loff + j * js;
  asm volatile(
      "s_nop 4\n\t"
      "global_load_dwordx4 %0, %8, %16\n\t"
      "global_load_dwordx4 %1, %9, %16\n\t"
      "global_load_dwordx4 %2, %10, %16\n\t"
      "global_load_dwordx4 %3, %11, %16\n\t"
      "global_load_dwordx4 %4, %12, %16\n\t"
      "global_load_dwordx4 %5, %13, %16\n\t"
      "global_load_dwordx4 %6, %14, %16\n\t"
      "global_load_dwordx4 %7, %15, %16"
      : "=&v"(r[0]), "=&v"(r[1]), "=&v"(r[2]), "=&v"(r[3]), "=&v"(r[4]), "=&v"(r[5]), "=&v"(r[6]), "=&v"(r[7])
      : "v"(o[0]), "v"(o[1]), "v"(o[2]), "v"(o[3]), "v"(o[4]), "v"(o[5]), "v"(o[6]), "v"(o[7]), "s"(base)
      : "memory");
}
#define WAIT_R(N, r) asm volatile("s_waitcnt vmcnt(" #N ")" : "+v"(r[0]), "+v"(r[1]), "+v"(r[2]), "+v"(r[3]), \
    "+v"(r[4]), "+v"(r[5]), "+v"(r[6]), "+v"(r[7]) :: "memory")
__device__ __forceinline__ void cvt4(const f32x4 (&r)[8], u32x4 (&fr)[4]) {
#pragma unroll
  for (int c = 0; c < 4; ++c)
    fr[c] = (u32x4){pk(r[0][c], r[1][c]), pk(r[2][c], r[3][c]), pk(r[4][c], r[5][c]), pk(r[6][c], r[7][c])};
}
__device__ __forceinline__ void store4(char* dbase, unsigned voff, const u32x4 (&fr)[4]) {
  asm volatile(
      "s_nop 4\n\t"
      "global_store_dwordx4 %0, %1, %5 sc0 sc1\n\t"
      "global_store_dwordx4 %0, %2, %5 offset:1024 sc0 sc1\n\t"
      "global_store_dwordx4 %0, %3, %5 offset:2048 sc0 sc1\n\t"
      "global_store_dwordx4 %0, %4, %5 offset:3072 sc0 sc1\n\t"
      "s_nop 1"
      :: "v"(voff), "v"(fr[0]), "v"(fr[1]), "v"(fr[2]), "v"(fr[3]), "s"(dbase) : "memory");
}
__device__ __forceinline__ void store_packed(char* dbase, int lane, const Packed& p) {
  asm volatile(
      "s_nop 4\n\t"
      "global_store_dwordx4 %0, %2, %6 sc0 sc1\n\t"
      "global_store_dwordx4 %0, %3, %6 offset:1024 sc0 sc1\n\t"
      "global_store_dwordx4 %0, %4, %6 offset:2048 sc0 sc1\n\t"
      "global_store_dwordx2 %1, %5, %6 offset:3072 sc0 sc1\n\t"
      "s_nop 1"
      :: "v"(lane * 16), "v"(lane * 8), "v"(p.l0), "v"(p.l1), "v"(p.c0), "v"(p.c1), "s"(dbase) : "memory");
}

__device__ __forceinline__ void queue_wave(const Args& a, unsigned call, int d, int slot, int lane,
                                           LAS unsigned* seq, LAS unsigned* prog) {
  unsigned* ctr = (unsigned*)(a.ws[a.rank] + OFF_QCNT + (size_t)d * 128);
  for (unsigned i = 0;; ++i) {
    const unsigned cur = i == 0 ? (unsigned)slot : i == 1 ? (unsigned)(slot + NSLOT)
                                : __builtin_amdgcn_readfirstlane(*(volatile LAS unsigned*)(seq + i));
    if (cur >= 192u) break;
    while (__builtin_amdgcn_readfirstlane(*(volatile LAS unsigned*)prog) <= i) __builtin_amdgcn_s_sleep(2);
    if (lane == 0) {
      const unsigned t = __hip_atomic_fetch_add(ctr, 1u, __ATOMIC_RELAXED, __HIP_MEMORY_SCOPE_AGENT) - (call - 1u) * 192u;
      *(volatile LAS unsigned*)(seq + i + 2) = 2u * NSLOT + t;
    }
    asm volatile("s_waitcnt vmcnt(0) lgkmcnt(0)" ::: "memory");
  }
}

__device__ __forceinline__ unsigned producer(const Args& a, int d, int slot, int u, int lane,
                                             LAS char* scratch, LAS unsigned* cnt, LAS unsigned* seq, LAS unsigned* prog) {
  constexpr int NP = 192;
  char* dws = a.ws[d];
  const unsigned voff = lane * 16;
  f32x4 r[8];
  u32x4 fr[4];
  Piece p = decode(slot);
  Src s = src_of(a, d, p, lane);
  load8(r, s.base + u * s.ustep, s.loff, s.js);
  unsigned call_v;
  asm volatile("global_load_dword %0, %1, %2 sc1" : "=&v"(call_v) : "v"(0u), "s"(a.ws[a.rank] + OFF_EPOCH) : "memory");
  BAR();
  WAIT_R(0, r);
  asm volatile("s_nop 0" : "+v"(call_v));
  const unsigned call = __builtin_amdgcn_readfirstlane(call_v) + 1u;
  char* pstage = dws + OFF_PSTAGE + (size_t)(call & 1u) * PPAR;
  char* fstage = dws + OFF_STAGE + (size_t)(call & 1u) * FPAR;
  char* prevflag = nullptr;
  bool prevdirect = false;
  unsigned long long tnext = 0;
  int i = 0;
  auto publish = [&](int idx) -> int {
    unsigned old = 0;
    if (lane == 0) old = __hip_atomic_fetch_add(cnt + idx, 1u, __ATOMIC_RELAXED, __HIP_MEMORY_SCOPE_WORKGROUP);
    if (__builtin_amdgcn_readfirstlane(old) != 3u) return 0;
    if (lane == 0) { if (prevdirect) st16_sys(prevflag, (u32x4){call, call, call, call}); else st_sys(prevflag, call); }
    return 1;
  };
  int after = 4;
  int n = slot;
  for (; n < NP; ++i) {
    if (u == 0 && lane == 0) *(volatile LAS unsigned*)prog = (unsigned)i + 1u;
    int nx;
    if (i == 0) nx = slot + NSLOT;
    else {
      unsigned v;
      while ((v = __builtin_amdgcn_readfirstlane(*(volatile LAS unsigned*)(seq + i + 1))) == 0xffffffffu) __builtin_amdgcn_s_sleep(1);
      nx = (int)v;
    }
    const int nn = nx < NP ? nx : n;
    const Piece pn = decode(nn);
    const Src sn = src_of(a, d, pn, lane);
    const int cs = a.rank * 8 + p.kc;
    const bool direct = is_direct(p, n);
    asm volatile("s_waitcnt vmcnt(6)\n\ts_cmp_gt_u32 %8, 5\n\ts_cbranch_scc1 1f\n\ts_waitcnt vmcnt(5)\n\t"
                 "s_cmp_gt_u32 %8, 4\n\ts_cbranch_scc1 1f\n\ts_waitcnt vmcnt(4)\n1:"
                 : "+v"(r[0]), "+v"(r[1]), "+v"(r[2]), "+v"(r[3]), "+v"(r[4]), "+v"(r[5]), "+v"(r[6]), "+v"(r[7])
                 : "s"(__builtin_amdgcn_readfirstlane(after)) : "scc", "memory");
    cvt4(r, fr);
    load8(r, sn.base + u * sn.ustep, sn.loff, sn.js);
    Packed pk;
    bool raw = true;
    if (!direct) raw = pk_encode(fr, pk, lane, scratch);
    asm volatile("s_waitcnt vmcnt(8)" ::: "memory");
    after = (direct ? 4 : 5) + (prevflag != nullptr ? publish(i - 1) : 0);
    {
      unsigned long long now = __builtin_amdgcn_s_memrealtime();
      while (now < tnext) { __builtin_amdgcn_s_sleep(1); now = __builtin_amdgcn_s_memrealtime(); }
      tnext = now + (direct ? PACE_D : PACE_P);
    }
    if (direct) {
      store4(fstage + stage_off(p, cs) + u * 4096, voff, fr);
      prevflag = dws + OFF_LFLAG + (size_t)(p.tile * 64 + cs) * 16;
    } else {
      char* dst = pstage + (size_t)((cs * 24 + p.tile) * 4 + u) * PUNIT;
      if (raw) store4(dst, voff, fr); else store_packed(dst, lane, pk);
      if (lane < PK_HDR_LANES) st16_sys(dst + 4096 + lane * 16, pk.hdr);
      prevflag = dws + OFF_PFLAG + (size_t)(p.tile * 64 + cs) * 4;
    }
    prevdirect = direct;
    p = pn; s = sn; n = nx;
  }
  asm volatile("s_waitcnt vmcnt(0)" : "+v"(r[0]) :: "memory");
  publish(i - 1);
  return call;
}

__device__ __forceinline__ void decoder(const Args& a, unsigned call, int p, int j, int gw, int lane, LAS char* scratch) {
  char* myws = a.ws[a.rank];
  const char* pstage = myws + OFF_PSTAGE + (size_t)(call & 1u) * PPAR;
  const int u = j & 3;
  for (int U = gw; U < 768; U += 448) {
    const Piece pc = decode(U >> 2);
    const Src s = src_of(a, a.rank, pc, lane);
    const int cs = a.rank * 8 + pc.kc;
    f32x4 r[8]; u32x4 fr[4];
    load8(r, s.base + (U & 3) * s.ustep, s.loff, s.js);
    WAIT_R(0, r);
    cvt4(r, fr);
    store4(myws + OFF_STAGE + (size_t)(call & 1u) * FPAR + stage_off(pc, cs) + (U & 3) * 4096, lane * 16, fr);
    asm volatile("s_waitcnt vmcnt(0)" ::: "memory");
    if (lane == 0) st_agent(myws + OFF_LFLAG + (size_t)((pc.tile * 64 + cs) * 4 + (U & 3)) * 4, call);
  }
  for (int k = 0; k < 12; ++k) {
    const Piece pc = decode((j >> 2) + 16 * k);
    if (is_direct(pc, (j >> 2) + 16 * k)) continue;
    const int cs = p * 8 + pc.kc;
    const char* flag = myws + OFF_PFLAG + (size_t)(pc.tile * 64 + cs) * 4;
    {
      const unsigned long long t0 = __builtin_amdgcn_s_memrealtime();
      bool bad = false;
      for (;;) {
        const unsigned fv = ld_sys(flag), tm = ld_agent(myws + OFF_TMO);
        if (__builtin_amdgcn_readfirstlane(fv) >= call) break;
        if (__builtin_amdgcn_readfirstlane(tm) == call || __builtin_amdgcn_s_memrealtime() - t0 > DEADLINE) {
          if (lane == 0) st_agent(myws + OFF_TMO, call);
          bad = true;
          break;
        }
        __builtin_amdgcn_s_sleep(2);
      }
      if (bad) return;
    }
    const char* src = pstage + (size_t)((cs * 24 + pc.tile) * 4 + u) * PUNIT;
    Packed pk; unsigned h;
    asm volatile("s_nop 4\n\t"
                 "global_load_dword %4, %7, %8 offset:0 sc0 sc1\n\t"
                 "global_load_dwordx4 %2, %5, %8 offset:2048 sc0 sc1\n\t"
                 "global_load_dwordx2 %3, %6, %8 offset:3072 sc0 sc1\n\t"
                 "global_load_dwordx4 %0, %5, %8 sc0 sc1\n\t"
                 "global_load_dwordx4 %1, %5, %8 offset:1024 sc0 sc1\n\t"
                 "s_waitcnt vmcnt(0)"
                 : "=&v"(pk.l0), "=&v"(pk.l1), "=&v"(pk.c0), "=&v"(pk.c1), "=&v"(h)
                 : "v"(lane * 16), "v"(lane * 8), "v"(4096 + (lane & 31) * 4), "s"(src) : "memory");
    u32x4 fr[4];
    if (__builtin_amdgcn_readfirstlane(h) & 0x80u) {
      asm volatile("global_load_dwordx4 %0, %4, %5 sc0 sc1\n\t"
                   "global_load_dwordx4 %1, %4, %5 offset:1024 sc0 sc1\n\t"
                   "global_load_dwordx4 %2, %4, %5 offset:2048 sc0 sc1\n\t"
                   "global_load_dwordx4 %3, %4, %5 offset:3072 sc0 sc1\n\t"
                   "s_waitcnt vmcnt(0)"
                   : "=&v"(fr[0]), "=&v"(fr[1]), "=&v"(fr[2]), "=&v"(fr[3]) : "v"(lane * 16), "s"(src) : "memory");
    } else {
      pk_decode(pk, h, fr, lane, scratch);
    }
    char* dst = myws + OFF_STAGE + (size_t)(call & 1u) * FPAR + stage_off(pc, cs);
    store4(dst + u * 4096, lane * 16, fr);
    asm volatile("s_waitcnt vmcnt(0)" ::: "memory");
    if (lane == 0) st_agent(myws + OFF_LFLAG + (size_t)((pc.tile * 64 + cs) * 4 + u) * 4, call);
  }
}

__device__ __forceinline__ void expander(const Args& a, unsigned call, int ti, int part, int half, int lane, LAS char* buf) {
  char* myws = a.ws[a.rank];
  const int slot = ti * 4 + part;
  const char* flag = myws + OFF_OFLAG + (size_t)slot * 4;
  bool bad = false;
  {
    const unsigned long long t0 = __builtin_amdgcn_s_memrealtime();
    for (;;) {
      const unsigned fv = ld_sys(flag), tm = ld_agent(myws + OFF_TMO);
      if (__builtin_amdgcn_readfirstlane(fv) >= call) break;
      if (__builtin_amdgcn_readfirstlane(tm) == call || __builtin_amdgcn_s_memrealtime() - t0 > DEADLINE) {
        if (lane == 0) st_agent(myws + OFF_TMO, call);
        bad = true;
        break;
      }
      __builtin_amdgcn_s_sleep(1);
    }
  }
  const char* src = myws + OFF_CSLOT + (size_t)slot * CSLOT + lane * 16;
  if (!bad) {
    __builtin_amdgcn_global_load_lds((const GAS void*)(src + 16384), (LAS void*)(buf + 16384), 16, 0, 17);
    if (half == 0) {
#pragma unroll
      for (int i = 0; i < 5; ++i)
        __builtin_amdgcn_global_load_lds((const GAS void*)(src + i * 1024), (LAS void*)(buf + i * 1024), 16, 0, 17);
    } else {
#pragma unroll
      for (int i = 3; i < 10; ++i)
        __builtin_amdgcn_global_load_lds((const GAS void*)(src + i * 1024), (LAS void*)(buf + i * 1024), 16, 0, 17);
    }
  }
  const unsigned tm = __builtin_amdgcn_readfirstlane(ld_agent(myws + OFF_TMO));
  asm volatile("s_waitcnt vmcnt(0)" ::: "memory");
  bad = bad || tm == call;
  float* o = a.out[a.rank] + (size_t)((ti >> 4) * 128 + part * 32 + half * 16 + 4 * (lane >> 5)) * NCOL +
             ((a.rank >> 2) ^ 1) * 2048 + (ti & 15) * 128 + 4 * (lane & 31);
  if (bad) {
    for (int e = 0; e < 8; ++e)
      st16_sys(o + (size_t)((e & 3) + 8 * (e >> 2)) * NCOL, (u32x4){0x7fc00000u, 0x7fc00000u, 0x7fc00000u, 0x7fc00000u});
    return;
  }
  const u32x2 m = *(const LAS u32x2*)(buf + 16384 + lane * 8);
  const u32x2 cnt = *(const LAS u32x2*)(buf + 16896);
  const unsigned n0 = __builtin_amdgcn_readfirstlane(cnt[0]), total = __builtin_amdgcn_readfirstlane(cnt[1]);
  if (half == 0 ? n0 > 1280u : (n0 < 768u || total > 2560u)) {
#pragma unroll
    for (int i = 0; i < 16; ++i)
      __builtin_amdgcn_global_load_lds((const GAS void*)(src + i * 1024), (LAS void*)(buf + i * 1024), 16, 0, 17);
    asm volatile("s_waitcnt vmcnt(0)" ::: "memory");
  }
  unsigned base = half ? n0 : 0u;
  const unsigned bits = half ? m[0] : m[1];
  unsigned pos[32];
#pragma unroll
  for (int i = 0; i < 32; ++i) {
    const unsigned long long mm = __ballot((int)(bits << i) < 0);
    pos[i] = __builtin_amdgcn_mbcnt_hi((unsigned)(mm >> 32), __builtin_amdgcn_mbcnt_lo((unsigned)mm, base));
    base += (unsigned)__builtin_popcountll(mm);
  }
  float x[32];
#pragma unroll
  for (int i = 0; i < 32; ++i) x[i] = *(const LAS float*)(buf + (pos[i] & 4095u) * 4u);
#pragma unroll
  for (int e = 0; e < 8; ++e) {
    asm volatile("" : "+v"(x[4 * e]), "+v"(x[4 * e + 1]), "+v"(x[4 * e + 2]), "+v"(x[4 * e + 3]));
    f32x4 v;
#pragma unroll
    for (int c = 0; c < 4; ++c) v[c] = (int)(bits << (4 * e + c)) < 0 ? x[4 * e + c] : 0.f;
    st16_sys(o + (size_t)((e & 3) + 8 * (e >> 2)) * NCOL, __builtin_bit_cast(u32x4, v));
  }
}

__global__ void __launch_bounds__(THREADS) gemm_rs_kernel(Args a) {
  __shared__ __attribute__((aligned(16))) char lds[LDS_CTRL + 512];
  unsigned long long pc0;
  asm volatile("s_getpc_b64 %0" : "=s"(pc0));
  const int tid = threadIdx.x, lane = tid & 63;
  const int wv = __builtin_amdgcn_readfirstlane(tid >> 6);
  const int b = blockIdx.x, xcd = b & 7, jb = b >> 3;
  char* myws = a.ws[a.rank];
  unsigned call = 0;

  if (jb < 8) {
    LAS unsigned* sync = (LAS unsigned*)((LAS char*)lds + LDS_CTRL);
    LAS unsigned* seq = (LAS unsigned*)((LAS char*)lds + 4 * CSLOT);
    LAS unsigned* cnt = seq + 256;
    const bool streams = jb < NSLOT && xcd != a.rank;
    if (streams && wv < 4) {
      call = producer(a, xcd, jb, wv, lane, (LAS char*)lds + wv * PK_LDS, cnt, seq, sync + 8);
    } else
    {
      if (wv == 7 && jb == 0) {
        const char* pc = (const char*)(pc0 & ~127ull);
#pragma unroll
        for (int i = 0; i < 4; ++i)
          __builtin_amdgcn_global_load_lds((const GAS void*)(pc + (i * 64 + lane) * 128), (LAS void*)(sync + 64), 4, 0, 0);
        if (lane < 8) __builtin_amdgcn_global_load_lds((const GAS void*)(pc + (256 + lane) * 128), (LAS void*)(sync + 64), 4, 0, 0);
      }
      call = __builtin_amdgcn_readfirstlane(ld_agent(myws + OFF_EPOCH)) + 1u;
      if (wv == 7 && lane < 16) sync[lane] = 0u;
      if (wv >= 4) { seq[(wv - 4) * 64 + lane] = 0xffffffffu; cnt[(wv - 4) * 64 + lane] = 0u; }
      asm volatile("s_waitcnt lgkmcnt(0)" ::: "memory");
      BAR();
      if (streams && wv == 4) queue_wave(a, call, xcd, jb, lane, seq, sync + 8);
    }
    BAR();
    expander(a, call, jb * 8 + xcd, wv >> 1, wv & 1, lane, (LAS char*)lds + (wv >> 1) * CSLOT);
  } else if (jb >= 24) {
    call = __builtin_amdgcn_readfirstlane(ld_agent(myws + OFF_EPOCH)) + 1u;
    decoder(a, call, (a.rank + 1 + (jb - 24)) & 7, xcd * 8 + wv, (jb - 24) * 64 + xcd * 8 + wv, lane, (LAS char*)lds + wv * PK_LDS);
    BAR();
  } else {
    const int jc = jb - 8;
    const int rt = (xcd & 3) + (jc >> 3) * 4;
    const int ct = (xcd >> 2) * 8 + (jc & 7);
    call = __builtin_amdgcn_readfirstlane(ld_agent(myws + OFF_EPOCH)) + 1u;
    LAS unsigned* lds_bad = (LAS unsigned*)((LAS char*)lds + LDS_CTRL);
    LAS unsigned* lds_sched = (LAS unsigned*)((LAS char*)lds + LDS_CTRL + 64);
    const char* stage = myws + OFF_STAGE + (size_t)(call & 1u) * FPAR;
    if (wv == 6) {
      const char* fa = myws + OFF_LFLAG + (size_t)(rt * 64 + lane) * 16;
      const char* fb = myws + OFF_LFLAG + (size_t)((8 + ct) * 64 + lane) * 16;
      const char* ptm = myws + OFF_TMO;
      unsigned long long ready = 0, done = 0;
      for (int c = 0; c < 64; ++c) {
        if ((ready & ~done) == 0ull) {
          const unsigned long long t0 = __builtin_amdgcn_s_memrealtime();
          for (;;) {
            u32x4 va, vb; unsigned tm;
            asm volatile("global_load_dwordx4 %0, %3, off sc0 sc1\n\t"
                         "global_load_dwordx4 %1, %4, off sc0 sc1\n\t"
                         "global_load_dword %2, %5, off sc1\n\t"
                         "s_waitcnt vmcnt(0)"
                         : "=&v"(va), "=&v"(vb), "=&v"(tm) : "v"(fa), "v"(fb), "v"(ptm) : "memory");
            const unsigned lo = min(min(min(va[0], va[1]), min(va[2], va[3])), min(min(vb[0], vb[1]), min(vb[2], vb[3])));
            ready = __ballot(lo >= call);
            if ((ready & ~done) != 0ull) break;
            if (__builtin_amdgcn_readfirstlane(tm) == call || __builtin_amdgcn_s_memrealtime() - t0 > DEADLINE) {
              if (lane == 0) st_agent(myws + OFF_TMO, call);
              ready = ~0ull;
              break;
            }
            __builtin_amdgcn_s_sleep(1);
          }
        }
        const int cs = __builtin_ctzll(ready & ~done);
        done |= 1ull << cs;
        if (lane == 0) lds_sched[c & 7] = (unsigned)cs;
        if (c == 63) {
          const unsigned t = __builtin_amdgcn_readfirstlane(ld_agent(myws + OFF_TMO));
          if (lane == 0) *lds_bad = (t == call) ? 1u : 0u;
        }
        asm volatile("s_waitcnt lgkmcnt(0)" ::: "memory");
        BAR();
      }
      for (int i = 0; i <= DIST; ++i) BAR();
      asm volatile("s_waitcnt vmcnt(0)" ::: "memory");
    } else if (wv == 7) {
    } else if (wv >= 4) {
      const int lw = wv - 4;
      const char* gbase = stage + A_BYTES + (size_t)ct * PIECE + lw * 8192 + lane * 16;
      BAR();
      int st = 0;
      for (int c = 0; c < 64 + DIST; ++c) {
        if (c < 64) {
          const int cs = __builtin_amdgcn_readfirstlane(*(volatile LAS unsigned*)(lds_sched + (c & 7))) & 63;
          const char* ga = gbase + (size_t)cs * (16 * PIECE);
          LAS char* la = (LAS char*)lds + st * 16384 + lw * 8192;
#pragma unroll
          for (int f = 0; f < 8; ++f)
            __builtin_amdgcn_global_load_lds((const GAS void*)(ga + f * 1024), (LAS void*)(la + f * 1024), 16, 0, 17);
          if constexpr (DIST == 3) asm volatile("s_waitcnt vmcnt(8)" ::: "memory");
          else if constexpr (DIST == 4) asm volatile("s_waitcnt vmcnt(16)" ::: "memory");
          else asm volatile("s_waitcnt vmcnt(24)" ::: "memory");
        } else if (63 - c + DIST - 2 == 2) {
          asm volatile("s_waitcnt vmcnt(16)" ::: "memory");
        } else if (63 - c + DIST - 2 == 1) {
          asm volatile("s_waitcnt vmcnt(8)" ::: "memory");
        } else {
          asm volatile("s_waitcnt vmcnt(0)" ::: "memory");
        }
        st = st == NSTAGE - 1 ? 0 : st + 1;
        BAR();
      }
    } else {
      f32x16 acc[4];
#pragma unroll
      for (int c = 0; c < 4; ++c)
#pragma unroll
        for (int e = 0; e < 16; ++e) acc[c][e] = 0.f;
      const __amdgpu_buffer_rsrc_t rsa = __builtin_amdgcn_make_buffer_rsrc((void*)(stage + (size_t)rt * PIECE), 0, (int)A_BYTES, 0x00020000);
      const unsigned aoff = wv * 4096 + lane * 16;
      h8 ar[NSTAGE][4];
      h8 bf[2][4];
#define SCHED_READ(C) (*(volatile LAS unsigned*)(lds_sched + ((C) & 7)))
#define LOAD_A(SLOT, SV) do {                                                                            \
        const unsigned cs_ = __builtin_amdgcn_readfirstlane(SV) & 63u;                                   \
        _Pragma("unroll") for (int s_ = 0; s_ < 4; ++s_)                                                 \
          ar[SLOT][s_] = __builtin_bit_cast(h8, __builtin_amdgcn_raw_buffer_load_b128(rsa, aoff + s_ * 1024, cs_ * (unsigned)(8 * PIECE), 17)); \
      } while (0)
#define READ_B(BUF, ST, S) do {                                                                          \
        _Pragma("unroll") for (int c_ = 0; c_ < 4; ++c_)                                                 \
          bf[BUF][c_] = *(const LAS h8*)((const LAS char*)lds + (ST) * 16384 + (S) * 4096 + c_ * 1024 + lane * 16); \
      } while (0)
#define MMA4(SLOT, S) do {                                                                               \
        __builtin_amdgcn_sched_barrier(0);                                                               \
        _Pragma("unroll") for (int c_ = 0; c_ < 4; ++c_)                                                 \
          acc[c_] = __builtin_amdgcn_mfma_f32_32x32x16_f16(ar[SLOT][S], bf[(S) & 1][c_], acc[c_], 0, 0, 0); \
        __builtin_amdgcn_sched_barrier(0);                                                               \
      } while (0)
#define STEP(K, T, MORE) do {                                                                            \
        READ_B(1, K, 1);                                                                                 \
        unsigned sv_ = 0; if (MORE) sv_ = SCHED_READ((T) + DIST);                                        \
        MMA4(K, 0);                                                                                      \
        READ_B(0, K, 2);                                                                                 \
        if (MORE) LOAD_A(((K) + DIST) % NSTAGE, sv_);                                                    \
        MMA4(K, 1);                                                                                      \
        READ_B(1, K, 3);                                                                                 \
        MMA4(K, 2);                                                                                      \
        READ_B(0, ((K) + 1) % NSTAGE, 0);                                                                \
        MMA4(K, 3);                                                                                      \
        asm volatile("s_waitcnt lgkmcnt(4)" ::: "memory");        \
        BAR();                                                                                           \
      } while (0)
      BAR(); LOAD_A(0, SCHED_READ(0)); BAR(); LOAD_A(1, SCHED_READ(1)); BAR(); LOAD_A(2, SCHED_READ(2));
      if constexpr (DIST >= 4) { BAR(); LOAD_A(3, SCHED_READ(3)); }
      if constexpr (DIST == 5) { BAR(); LOAD_A(4, SCHED_READ(4)); }
      BAR();
      READ_B(0, 0, 0);
      for (int t = 0; t < 54; t += 6) {
        STEP(0, t, true); STEP(1, t + 1, true); STEP(2, t + 2, true);
        STEP(3, t + 3, true); STEP(4, t + 4, true); STEP(5, t + 5, true);
      }
      STEP(0, 54, true); STEP(1, 55, true); STEP(2, 56, true); STEP(3, 57, true); STEP(4, 58, true);
      STEP(5, 59, 59 + DIST <= 63);
      STEP(0, 60, 60 + DIST <= 63); STEP(1, 61, false); STEP(2, 62, false); STEP(3, 63, false);
#undef SCHED_READ
#undef LOAD_A
#undef READ_B
#undef MMA4
#undef STEP
      const bool bad = *lds_bad != 0u;
      const int g = a.rank >> 2;
      if (rt < 4) {
        const float qnan = __builtin_nanf("");
        float* o = a.out[a.rank] + (size_t)(rt * 128 + wv * 32 + 4 * (lane >> 5)) * NCOL + g * 2048 + ct * 128 + 4 * (lane & 31);
#pragma unroll
        for (int e = 0; e < 16; ++e) {
          f32x4 v;
#pragma unroll
          for (int c = 0; c < 4; ++c) { const float y = acc[c][e]; v[c] = bad ? qnan : (y < 0.f ? 0.f : y); }
          st16_sys(o + (size_t)((e & 3) + 8 * (e >> 2)) * NCOL, __builtin_bit_cast(u32x4, v));
        }
      } else {
        LAS char* buf = (LAS char*)lds + wv * CSLOT;
        const int q = ((g ^ 1) << 2) + (rt - 4);
        const int slot = ((a.rank & 3) * 16 + ct) * 4 + wv;
        char* dst = a.ws[q & 7] + OFF_CSLOT + (size_t)slot * CSLOT + lane * 16;
        if (bad) {
          const unsigned nb = 0x7fc00000u;
          for (int i = 0; i < 16; ++i) st16_sys(dst + i * 1024, (u32x4){nb, nb, nb, nb});
          if (lane < 32) st16_sys(dst + 16384, (u32x4){~0u, ~0u, ~0u, ~0u});
          if (lane == 32) st16_sys(dst + 16384, (u32x4){2048u, 4096u, 0u, 0u});
        } else {
          unsigned base = 0, sent = 0, n0 = 0, mhi = 0, mlo = 0;
#pragma unroll
          for (int e4 = 0; e4 < 4; ++e4) {
#pragma unroll
            for (int ee = 0; ee < 4; ++ee) {
              unsigned long long m[4];
              unsigned bs[4], pos[4];
#pragma unroll
              for (int c = 0; c < 4; ++c) m[c] = __ballot(!(acc[c][e4 * 4 + ee] <= 0.f));
#pragma unroll
              for (int c = 0; c < 4; ++c) { bs[c] = base; base += (unsigned)__builtin_popcountll(m[c]); }
#pragma unroll
              for (int c = 0; c < 4; ++c)
                pos[c] = __builtin_amdgcn_mbcnt_hi((unsigned)(m[c] >> 32), __builtin_amdgcn_mbcnt_lo((unsigned)m[c], 0u)) + bs[c];
              asm volatile("" : "+v"(pos[0]), "+v"(pos[1]), "+v"(pos[2]), "+v"(pos[3]));
#pragma unroll
              for (int c = 0; c < 4; ++c) {
                const float y = acc[c][e4 * 4 + ee];
                const bool nz = !(y <= 0.f);
                *(LAS float*)(buf + 4u * (nz ? pos[c] : 4288u + lane)) = y;
                if (e4 < 2) asm volatile("v_addc_co_u32_e64 %0, vcc, %0, %0, %1" : "+v"(mhi) : "s"(m[c]) : "vcc");
                else asm volatile("v_addc_co_u32_e64 %0, vcc, %0, %0, %1" : "+v"(mlo) : "s"(m[c]) : "vcc");
              }
            }
            if (e4 == 1) n0 = base;
            asm volatile("s_waitcnt vmcnt(0)" ::: "memory");
            for (; sent + 256u <= base; sent += 256u)
              st16_sys(dst + sent * 4u, *(const LAS u32x4*)(buf + sent * 4u + lane * 16));
          }
          if (sent + lane * 4u < base) st16_sys(dst + sent * 4u, *(const LAS u32x4*)(buf + sent * 4u + lane * 16));
          *(LAS u32x2*)(buf + 16384 + lane * 8) = (u32x2){mlo, mhi};
          if (lane == 0) *(LAS u32x2*)(buf + 16896) = (u32x2){n0, base};
          if (lane < 33) st16_sys(dst + 16384, *(const LAS u32x4*)(buf + 16384 + lane * 16));
        }
        asm volatile("s_waitcnt vmcnt(0)" ::: "memory");
        if (lane == 0) st_sys(a.ws[q & 7] + OFF_OFLAG + (size_t)slot * 4, call);
      }
    }
  }
  if (tid == 0) {
    const unsigned t = __hip_atomic_fetch_add((unsigned*)(myws + OFF_ENDCNT), 1u, __ATOMIC_RELAXED, __HIP_MEMORY_SCOPE_AGENT);
    if (t % GRID == GRID - 1) st_agent(myws + OFF_EPOCH, call);
  }
}

}

extern "C" void kernel_launch(int rank, int nranks, ncclComm_t comm,
                              hipStream_t stream,
                              void* const* d_in, const int* in_sizes, int n_in,
                              void* d_out, int out_size,
                              const int* in_dtypes, int out_dtype,
                              void* const* d_out_all,
                              void* const* d_ws_all, size_t ws_bytes) {
  (void)comm; (void)in_dtypes; (void)out_dtype; (void)d_out;
  if (nranks != NR || (unsigned)rank >= (unsigned)NR || n_in < 2 || ws_bytes < WS_NEED ||
      in_sizes[0] != 4096 * KSH || in_sizes[1] != KSH * NCOL || out_size != 512 * NCOL) {
    fprintf(stderr, "gemm_rs: unexpected problem shape\n");
    return;
  }
  Args a;
  for (int p = 0; p < NR; ++p) { a.ws[p] = (char*)d_ws_all[p]; a.out[p] = (float*)d_out_all[p]; }
  a.x = (const float*)d_in[0];
  a.w = (const float*)d_in[1];
  a.rank = rank;
  a.pad = 0;
  gemm_rs_kernel<<<GRID, THREADS, 0, stream>>>(a);
  hipError_t e = hipGetLastError();
  if (e != hipSuccess) fprintf(stderr, "gemm_rs: launch failed: %s\n", hipGetErrorString(e));
}
```

```cpp
#include <hip/hip_runtime.h>
#include <rccl/rccl.h>
#include <cstdio>

namespace {

constexpr int NR = 8;
constexpr int NCOL = 4096;
constexpr int KSH = 512;
constexpr int LATE = 3;
constexpr int NSLOT = 8;
constexpr unsigned PACE_P = 200 * NSLOT / 8, PACE_D = PACE_P * 4096 / 3712;
constexpr int KD = 7;

constexpr size_t OFF_EPOCH = 0;
constexpr size_t OFF_ENDCNT = 128;
constexpr size_t OFF_TMO = 256;
constexpr size_t OFF_OFLAG = 1024;
constexpr size_t OFF_QCNT = 2048;
constexpr size_t OFF_PFLAG = 4096;
constexpr size_t OFF_LFLAG = 16384;
constexpr size_t OFF_STAGE = 1u << 20;
constexpr size_t PIECE = 16384;
constexpr size_t A_BYTES = 64 * 8 * PIECE;
constexpr size_t B_BYTES = 64 * 16 * PIECE;
constexpr size_t PUNIT = 4224;
constexpr size_t PPAR = 64 * 24 * 4 * PUNIT;
constexpr size_t FPAR = A_BYTES + B_BYTES;
constexpr size_t OFF_PSTAGE = OFF_STAGE + 2 * FPAR;
constexpr size_t CSLOT = 17408;
constexpr size_t OFF_CSLOT = OFF_PSTAGE + 2 * PPAR;
constexpr size_t WS_NEED = OFF_CSLOT + 256 * CSLOT;

constexpr int NPROD = 64, NCONS = 128, NDEC = 56, GRID = NPROD + NCONS + NDEC;
constexpr int THREADS = 512;
constexpr int NSTAGE = 6;
constexpr int DIST = 4;
constexpr int LDS_CTRL = 6 * 16384;
constexpr unsigned long long DEADLINE = 30000000ull;

typedef unsigned u32x4 __attribute__((ext_vector_type(4)));
typedef unsigned u32x2 __attribute__((ext_vector_type(2)));
typedef float f32x4 __attribute__((ext_vector_type(4)));
typedef float f32x16 __attribute__((ext_vector_type(16)));
typedef _Float16 h2 __attribute__((ext_vector_type(2)));
typedef _Float16 h8 __attribute__((ext_vector_type(8)));
#define LAS __attribute__((address_space(3)))
#define GAS __attribute__((address_space(1)))

struct Args {
  char* myws;
  float* myout;
  const float* x;
  const float* w;
  int rank;
  int pad[7];
  char* ws[NR];
};

__device__ __forceinline__ unsigned ld_sys(const void* p) {
  return __hip_atomic_load((const unsigned*)p, __ATOMIC_RELAXED, __HIP_MEMORY_SCOPE_SYSTEM);
}
__device__ __forceinline__ unsigned ld_agent(const void* p) {
  return __hip_atomic_load((const unsigned*)p, __ATOMIC_RELAXED, __HIP_MEMORY_SCOPE_AGENT);
}
__device__ __forceinline__ void st_sys(void* p, unsigned v) {
  __hip_atomic_store((unsigned*)p, v, __ATOMIC_RELAXED, __HIP_MEMORY_SCOPE_SYSTEM);
}
__device__ __forceinline__ void st_agent(void* p, unsigned v) {
  __hip_atomic_store((unsigned*)p, v, __ATOMIC_RELAXED, __HIP_MEMORY_SCOPE_AGENT);
}
__device__ __forceinline__ void st16_sys(void* p, u32x4 v) {
  asm volatile("global_store_dwordx4 %0, %1, off sc0 sc1\n\ts_nop 1" :: "v"(p), "v"(v) : "memory");
}
__device__ __forceinline__ unsigned pk(float a, float b) {
  h2 h; h.x = (_Float16)a; h.y = (_Float16)b; return __builtin_bit_cast(unsigned, h);
}
#define BAR() do { asm volatile("" ::: "memory"); __builtin_amdgcn_s_barrier(); asm volatile("" ::: "memory"); } while (0)

typedef unsigned short us2 __attribute__((ext_vector_type(2)));
constexpr int PK_UNIT = (int)PUNIT;
constexpr int PK_LDS = 4096 + 128;
constexpr int PK_HDR_LANES = 8;
constexpr int PK_MAXESC = 16 * PK_HDR_LANES - 1;
struct Packed { u32x4 l0, l1, c0; u32x2 c1; u32x4 hdr; };

__device__ __forceinline__ unsigned pk_sub(unsigned a, unsigned b) { return __builtin_bit_cast(unsigned, __builtin_bit_cast(us2, a) - __builtin_bit_cast(us2, b)); }
__device__ __forceinline__ unsigned pk_add(unsigned a, unsigned b) { return __builtin_bit_cast(unsigned, __builtin_bit_cast(us2, a) + __builtin_bit_cast(us2, b)); }
__device__ __forceinline__ unsigned pk_min(unsigned a, unsigned b) { return __builtin_bit_cast(unsigned, __builtin_elementwise_min(__builtin_bit_cast(us2, a), __builtin_bit_cast(us2, b))); }
template <int CTRL, int RM> __device__ __forceinline__ unsigned dpp_add(unsigned x) {
  return x + (unsigned)__builtin_amdgcn_update_dpp(0, (int)x, CTRL, RM, 0xf, false);
}
__device__ __forceinline__ unsigned wave_scan(unsigned x) {
  x = dpp_add<0x111, 0xf>(x); x = dpp_add<0x112, 0xf>(x); x = dpp_add<0x114, 0xf>(x); x = dpp_add<0x118, 0xf>(x);
  x = dpp_add<0x142, 0xa>(x); x = dpp_add<0x143, 0xc>(x);
  return x;
}
__device__ __forceinline__ unsigned esc_addr(unsigned b, int lane) {
  const unsigned j = 15u - (b & 15u);
  return (j >> 2) * 1024u + lane * 16u + (j & 3u) * 4u + (b >> 4) * 2u + 1u;
}

__device__ __forceinline__ bool pk_encode(const u32x4 (&fr)[4], Packed& p, int lane, LAS char* lds) {
  unsigned d[16];
#pragma unroll
  for (int j = 0; j < 16; ++j) d[j] = fr[j >> 2][j & 3];
  unsigned s = 0;
#pragma unroll
  for (int j = 0; j < 16; j += 5) s += ((d[j] >> 10) & 31u) + ((d[j] >> 26) & 31u);
  s = __builtin_amdgcn_readlane(wave_scan(s), 63);
  int top = (int)((s + 1388u) >> 9);
  int base = top - 6; base = base < 0 ? 0 : (base > 24 ? 24 : base);
  const unsigned base2 = (unsigned)base * 0x00010001u;
  unsigned q[16], acc = 0;
#pragma unroll
  for (int j = 0; j < 16; ++j) {
    const unsigned x = d[j] >> 10;
    const unsigned c = pk_min(pk_sub(x & 0x001f001fu, base2), 0x00070007u);
    const unsigned c6 = (x & 0x00200020u) | ((d[j] >> 5) & 0x00180018u) | c;
    q[j] = (c6 & 0x3fu) | (c6 >> 10);
    acc = pk_add(pk_add(acc, acc), (pk_add(c, 0x00010001u) >> 3) & 0x00010001u);
  }
#pragma unroll
  for (int i = 0; i < 4; ++i) {
    p.l0[i] = __builtin_amdgcn_perm(d[2 * i + 1], d[2 * i], 0x06040200u);
    p.l1[i] = __builtin_amdgcn_perm(d[2 * i + 9], d[2 * i + 8], 0x06040200u);
  }
  p.c0[0] = q[0] | (q[1] << 12) | (q[2] << 24);
  p.c0[1] = (q[2] >> 8) | (q[3] << 4) | (q[4] << 16) | (q[5] << 28);
  p.c0[2] = (q[5] >> 4) | (q[6] << 8) | (q[7] << 20);
  p.c0[3] = q[8] | (q[9] << 12) | (q[10] << 24);
  p.c1[0] = (q[10] >> 8) | (q[11] << 4) | (q[12] << 16) | (q[13] << 28);
  p.c1[1] = (q[13] >> 4) | (q[14] << 8) | (q[15] << 20);
  const unsigned cnt = (unsigned)__builtin_popcount(acc);
  const unsigned incl = wave_scan(cnt);
  const unsigned total = __builtin_amdgcn_readlane(incl, 63);
  if (total > (unsigned)PK_MAXESC) { p.hdr = (u32x4){0x80u, 0u, 0u, 0u}; return true; }
  LAS char* list = lds + 4096;
  if (total != 0u) {
#pragma unroll
    for (int f = 0; f < 4; ++f) *(LAS u32x4*)(lds + f * 1024 + lane * 16) = fr[f];
    unsigned off = 1u + incl - cnt;
    while (acc != 0u) {
      const unsigned b = (unsigned)__builtin_ctz(acc);
      acc &= acc - 1u;
      const unsigned hb = *(const LAS unsigned char*)(lds + esc_addr(b, lane));
      *(LAS unsigned char*)(list + off) = (unsigned char)((hb >> 2) & 31u);
      ++off;
    }
  }
  if (lane == 0) *(LAS unsigned char*)list = (unsigned char)base;
  p.hdr = *(const LAS u32x4*)(list + (lane & 7) * 16);
  return false;
}

__device__ __forceinline__ void pk_decode(const Packed& p, unsigned h, u32x4 (&fr)[4], int lane, LAS char* lds) {
  const unsigned base2 = (__builtin_amdgcn_readfirstlane(h) & 31u) * 0x00010001u;
  unsigned q[16];
  q[0] = p.c0[0] & 0xfffu; q[1] = (p.c0[0] >> 12) & 0xfffu; q[2] = __builtin_amdgcn_alignbit(p.c0[1], p.c0[0], 24) & 0xfffu;
  q[3] = (p.c0[1] >> 4) & 0xfffu; q[4] = (p.c0[1] >> 16) & 0xfffu; q[5] = __builtin_amdgcn_alignbit(p.c0[2], p.c0[1], 28) & 0xfffu;
  q[6] = (p.c0[2] >> 8) & 0xfffu; q[7] = p.c0[2] >> 20;
  q[8] = p.c0[3] & 0xfffu; q[9] = (p.c0[3] >> 12) & 0xfffu; q[10] = __builtin_amdgcn_alignbit(p.c1[0], p.c0[3], 24) & 0xfffu;
  q[11] = (p.c1[0] >> 4) & 0xfffu; q[12] = (p.c1[0] >> 16) & 0xfffu; q[13] = __builtin_amdgcn_alignbit(p.c1[1], p.c1[0], 28) & 0xfffu;
  q[14] = (p.c1[1] >> 8) & 0xfffu; q[15] = p.c1[1] >> 20;
  unsigned acc = 0;
#pragma unroll
  for (int j = 0; j < 16; ++j) {
    const unsigned c6 = (q[j] | (q[j] << 10)) & 0x003f003fu;
    const unsigned c = c6 & 0x00070007u;
    const unsigned hi8 = ((c6 << 2) & 0x00800080u) | ((c6 >> 3) & 0x00030003u) | (pk_add(c, base2) << 2);
    const unsigned l = (j < 8 ? p.l0 : p.l1)[(j & 7) >> 1];
    fr[j >> 2][j & 3] = __builtin_amdgcn_perm(hi8, l, (j & 1) ? 0x06030402u : 0x06010400u);
    acc = pk_add(pk_add(acc, acc), (pk_add(c, 0x00010001u) >> 3) & 0x00010001u);
  }
  const unsigned cnt = (unsigned)__builtin_popcount(acc);
  const unsigned incl = wave_scan(cnt);
  if (__builtin_amdgcn_readlane(incl, 63) != 0u) {
    LAS char* list = lds + 4096;
    if (lane < 32) *(LAS unsigned*)(list + lane * 4) = h;
#pragma unroll
    for (int f = 0; f < 4; ++f) *(LAS u32x4*)(lds + f * 1024 + lane * 16) = fr[f];
    unsigned off = 1u + incl - cnt;
    while (acc != 0u) {
      const unsigned b = (unsigned)__builtin_ctz(acc);
      acc &= acc - 1u;
      LAS unsigned char* t = (LAS unsigned char*)(lds + esc_addr(b, lane));
      const unsigned e = *(const LAS unsigned char*)(list + off);
      *t = (unsigned char)((*t & 0x83u) | (e << 2));
      ++off;
    }
#pragma unroll
    for (int f = 0; f < 4; ++f) fr[f] = *(const LAS u32x4*)(lds + f * 1024 + lane * 16);
  }
}

struct Piece { int kc, tile; };
__device__ __forceinline__ Piece decode(int n) {
  Piece p;
  constexpr int nf = LATE * 24, n2 = nf + (8 - LATE) * 20;
  int i;
  if (n < nf) { p.kc = n / 24; i = n % 24; }
  else if (n < n2) { const int v = n - nf; p.kc = LATE + v / 20; i = v % 20; }
  else { const int v = n - n2; p.kc = LATE + v / 4; i = 20 + v % 4; }
  p.tile = i < 16 ? 8 + i : (i < 20 ? i - 12 : i - 20);
  return p;
}
constexpr int FIRSTD = 8;
__device__ __forceinline__ bool is_direct(const Piece& p, int n) { return n < FIRSTD || p.kc >= KD || (p.tile < 4 && p.kc >= LATE); }
__device__ __forceinline__ size_t stage_off(const Piece& p, int cs) {
  return p.tile >= 8 ? A_BYTES + ((size_t)cs * 16 + (p.tile - 8)) * PIECE : ((size_t)cs * 8 + p.tile) * PIECE;
}
struct Src { const char* base; unsigned ustep, loff, js; };
__device__ __forceinline__ Src src_of(const Args& a, int d, const Piece& p, int lane) {
  Src s;
  const unsigned i = lane & 31, h = lane >> 5;
  if (p.tile >= 8) {
    s.base = (const char*)a.w + ((size_t)(p.kc * 64) * NCOL + (d >> 2) * 2048 + (p.tile - 8) * 128) * 4;
    s.ustep = NCOL * 4; s.loff = h * (4 * NCOL * 4) + i * 16; s.js = 8 * NCOL * 4;
  } else {
    const int row0 = p.tile < 4 ? d * 512 + p.tile * 128 : (((d >> 2) ^ 1) * 4 + (p.tile - 4)) * 512 + (d & 3) * 128;
    s.base = (const char*)a.x + ((size_t)row0 * KSH + p.kc * 64) * 4;
    s.ustep = 32 * KSH * 4; s.loff = i * (KSH * 4) + h * 16; s.js = 32;
  }
  return s;
}
__device__ __forceinline__ void load8(f32x4 (&r)[8], const char* base, unsigned loff, unsigned js) {
  unsigned o[8];
#pragma unroll
  for (int j = 0; j < 8; ++j) o[j] = loff + j * js;
  asm volatile(
      "s_nop 4\n\t"
      "global_load_dwordx4 %0, %8, %16\n\t"
      "global_load_dwordx4 %1, %9, %16\n\t"
      "global_load_dwordx4 %2, %10, %16\n\t"
      "global_load_dwordx4 %3, %11, %16\n\t"
      "global_load_dwordx4 %4, %12, %16\n\t"
      "global_load_dwordx4 %5, %13, %16\n\t"
      "global_load_dwordx4 %6, %14, %16\n\t"
      "global_load_dwordx4 %7, %15, %16"
      : "=&v"(r[0]), "=&v"(r[1]), "=&v"(r[2]), "=&v"(r[3]), "=&v"(r[4]), "=&v"(r[5]), "=&v"(r[6]), "=&v"(r[7])
      : "v"(o[0]), "v"(o[1]), "v"(o[2]), "v"(o[3]), "v"(o[4]), "v"(o[5]), "v"(o[6]), "v"(o[7]), "s"(base)
      : "memory");
}
#define WAIT_R(N, r) asm volatile("s_waitcnt vmcnt(" #N ")" : "+v"(r[0]), "+v"(r[1]), "+v"(r[2]), "+v"(r[3]), \
    "+v"(r[4]), "+v"(r[5]), "+v"(r[6]), "+v"(r[7]) :: "memory")
__device__ __forceinline__ void cvt4(const f32x4 (&r)[8], u32x4 (&fr)[4]) {
#pragma unroll
  for (int c = 0; c < 4; ++c)
    fr[c] = (u32x4){pk(r[0][c], r[1][c]), pk(r[2][c], r[3][c]), pk(r[4][c], r[5][c]), pk(r[6][c], r[7][c])};
}
__device__ __forceinline__ void store4(char* dbase, unsigned voff, const u32x4 (&fr)[4]) {
  asm volatile(
      "s_nop 4\n\t"
      "global_store_dwordx4 %0, %1, %5 sc0 sc1\n\t"
      "global_store_dwordx4 %0, %2, %5 offset:1024 sc0 sc1\n\t"
      "global_store_dwordx4 %0, %3, %5 offset:2048 sc0 sc1\n\t"
      "global_store_dwordx4 %0, %4, %5 offset:3072 sc0 sc1\n\t"
      "s_nop 1"
      :: "v"(voff), "v"(fr[0]), "v"(fr[1]), "v"(fr[2]), "v"(fr[3]), "s"(dbase) : "memory");
}
__device__ __forceinline__ void store_packed(char* dbase, int lane, const Packed& p) {
  asm volatile(
      "s_nop 4\n\t"
      "global_store_dwordx4 %0, %2, %6 sc0 sc1\n\t"
      "global_store_dwordx4 %0, %3, %6 offset:1024 sc0 sc1\n\t"
      "global_store_dwordx4 %0, %4, %6 offset:2048 sc0 sc1\n\t"
      "global_store_dwordx2 %1, %5, %6 offset:3072 sc0 sc1\n\t"
      "s_nop 1"
      :: "v"(lane * 16), "v"(lane * 8), "v"(p.l0), "v"(p.l1), "v"(p.c0), "v"(p.c1), "s"(dbase) : "memory");
}

__device__ __forceinline__ void queue_wave(const Args& a, unsigned call, int d, int slot, int lane,
                                           LAS unsigned* seq, LAS unsigned* prog) {
  unsigned* ctr = (unsigned*)(a.myws + OFF_QCNT + (size_t)d * 128);
  for (unsigned i = 0;; ++i) {
    const unsigned cur = i == 0 ? (unsigned)slot : i == 1 ? (unsigned)(slot + NSLOT)
                                : __builtin_amdgcn_readfirstlane(*(volatile LAS unsigned*)(seq + i));
    if (cur >= 192u) break;
    while (__builtin_amdgcn_readfirstlane(*(volatile LAS unsigned*)prog) <= i) __builtin_amdgcn_s_sleep(2);
    if (lane == 0) {
      const unsigned t = __hip_atomic_fetch_add(ctr, 1u, __ATOMIC_RELAXED, __HIP_MEMORY_SCOPE_AGENT) - (call - 1u) * 192u;
      *(volatile LAS unsigned*)(seq + i + 2) = 2u * NSLOT + t;
    }
    asm volatile("s_waitcnt vmcnt(0) lgkmcnt(0)" ::: "memory");
  }
}

__device__ __forceinline__ unsigned producer(const Args& a, int d, int slot, int u, int lane,
                                             LAS char* scratch, LAS unsigned* cnt, LAS unsigned* seq, LAS unsigned* prog) {
  constexpr int NP = 192;
  char* dws = a.ws[d];
  const unsigned voff = lane * 16;
  f32x4 r[8];
  u32x4 fr[4];
  Piece p = decode(slot);
  Src s = src_of(a, d, p, lane);
  load8(r, s.base + u * s.ustep, s.loff, s.js);
  unsigned call_v;
  asm volatile("global_load_dword %0, %1, %2 sc1" : "=&v"(call_v) : "v"(0u), "s"(a.myws + OFF_EPOCH) : "memory");
  BAR();
  WAIT_R(0, r);
  asm volatile("s_nop 0" : "+v"(call_v));
  const unsigned call = __builtin_amdgcn_readfirstlane(call_v) + 1u;
  char* pstage = dws + OFF_PSTAGE + (size_t)(call & 1u) * PPAR;
  char* fstage = dws + OFF_STAGE + (size_t)(call & 1u) * FPAR;
  char* prevflag = nullptr;
  bool prevdirect = false;
  unsigned long long tnext = 0;
  int i = 0;
  auto publish = [&](int idx) -> int {
    unsigned old = 0;
    if (lane == 0) old = __hip_atomic_fetch_add(cnt + idx, 1u, __ATOMIC_RELAXED, __HIP_MEMORY_SCOPE_WORKGROUP);
    if (__builtin_amdgcn_readfirstlane(old) != 3u) return 0;
    if (lane == 0) { if (prevdirect) st16_sys(prevflag, (u32x4){call, call, call, call}); else st_sys(prevflag, call); }
    return 1;
  };
  int after = 4;
  int n = slot;
  for (; n < NP; ++i) {
    if (u == 0 && lane == 0) *(volatile LAS unsigned*)prog = (unsigned)i + 1u;
    int nx;
    if (i == 0) nx = slot + NSLOT;
    else {
      unsigned v;
      while ((v = __builtin_amdgcn_readfirstlane(*(volatile LAS unsigned*)(seq + i + 1))) == 0xffffffffu) __builtin_amdgcn_s_sleep(1);
      nx = (int)v;
    }
    const int nn = nx < NP ? nx : n;
    const Piece pn = decode(nn);
    const Src sn = src_of(a, d, pn, lane);
    const int cs = a.rank * 8 + p.kc;
    const bool direct = is_direct(p, n);
    asm volatile("s_waitcnt vmcnt(6)\n\ts_cmp_gt_u32 %8, 5\n\ts_cbranch_scc1 1f\n\ts_waitcnt vmcnt(5)\n\t"
                 "s_cmp_gt_u32 %8, 4\n\ts_cbranch_scc1 1f\n\ts_waitcnt vmcnt(4)\n1:"
                 : "+v"(r[0]), "+v"(r[1]), "+v"(r[2]), "+v"(r[3]), "+v"(r[4]), "+v"(r[5]), "+v"(r[6]), "+v"(r[7])
                 : "s"(__builtin_amdgcn_readfirstlane(after)) : "scc", "memory");
    cvt4(r, fr);
    load8(r, sn.base + u * sn.ustep, sn.loff, sn.js);
    Packed pk;
    bool raw = true;
    if (!direct) raw = pk_encode(fr, pk, lane, scratch);
    asm volatile("s_waitcnt vmcnt(8)" ::: "memory");
    after = (direct ? 4 : 5) + (prevflag != nullptr ? publish(i - 1) : 0);
    {
      unsigned long long now = __builtin_amdgcn_s_memrealtime();
      while (now < tnext) { __builtin_amdgcn_s_sleep(1); now = __builtin_amdgcn_s_memrealtime(); }
      tnext = now + (direct ? PACE_D : PACE_P);
    }
    if (direct) {
      store4(fstage + stage_off(p, cs) + u * 4096, voff, fr);
      prevflag = dws + OFF_LFLAG + (size_t)(p.tile * 64 + cs) * 16;
    } else {
      char* dst = pstage + (size_t)((cs * 24 + p.tile) * 4 + u) * PUNIT;
      if (raw) store4(dst, voff, fr); else store_packed(dst, lane, pk);
      if (lane < PK_HDR_LANES) st16_sys(dst + 4096 + lane * 16, pk.hdr);
      prevflag = dws + OFF_PFLAG + (size_t)(p.tile * 64 + cs) * 4;
    }
    prevdirect = direct;
    p = pn; s = sn; n = nx;
  }
  asm volatile("s_waitcnt vmcnt(0)" : "+v"(r[0]) :: "memory");
  publish(i - 1);
  return call;
}

__device__ __forceinline__ void decoder(const Args& a, unsigned call, int p, int j, int gw, int lane, LAS char* scratch) {
  char* myws = a.myws;
  const char* pstage = myws + OFF_PSTAGE + (size_t)(call & 1u) * PPAR;
  const int u = j & 3;
  for (int U = gw; U < 768; U += 448) {
    const Piece pc = decode(U >> 2);
    const Src s = src_of(a, a.rank, pc, lane);
    const int cs = a.rank * 8 + pc.kc;
    f32x4 r[8]; u32x4 fr[4];
    load8(r, s.base + (U & 3) * s.ustep, s.loff, s.js);
    WAIT_R(0, r);
    cvt4(r, fr);
    store4(myws + OFF_STAGE + (size_t)(call & 1u) * FPAR + stage_off(pc, cs) + (U & 3) * 4096, lane * 16, fr);
    asm volatile("s_waitcnt vmcnt(0)" ::: "memory");
    if (lane == 0) st_agent(myws + OFF_LFLAG + (size_t)((pc.tile * 64 + cs) * 4 + (U & 3)) * 4, call);
  }
  for (int k = 0; k < 12; ++k) {
    const Piece pc = decode((j >> 2) + 16 * k);
    if (is_direct(pc, (j >> 2) + 16 * k)) continue;
    const int cs = p * 8 + pc.kc;
    const char* flag = myws + OFF_PFLAG + (size_t)(pc.tile * 64 + cs) * 4;
    {
      const unsigned long long t0 = __builtin_amdgcn_s_memrealtime();
      bool bad = false;
      for (;;) {
        const unsigned fv = ld_sys(flag), tm = ld_agent(myws + OFF_TMO);
        if (__builtin_amdgcn_readfirstlane(fv) >= call) break;
        if (__builtin_amdgcn_readfirstlane(tm) == call || __builtin_amdgcn_s_memrealtime() - t0 > DEADLINE) {
          if (lane == 0) st_agent(myws + OFF_TMO, call);
          bad = true;
          break;
        }
        __builtin_amdgcn_s_sleep(1);
      }
      if (bad) return;
    }
    const char* src = pstage + (size_t)((cs * 24 + pc.tile) * 4 + u) * PUNIT;
    Packed pk; unsigned h;
    asm volatile("s_nop 4\n\t"
                 "global_load_dword %4, %7, %8 offset:0 sc0 sc1\n\t"
                 "global_load_dwordx4 %2, %5, %8 offset:2048 sc0 sc1\n\t"
                 "global_load_dwordx2 %3, %6, %8 offset:3072 sc0 sc1\n\t"
                 "global_load_dwordx4 %0, %5, %8 sc0 sc1\n\t"
                 "global_load_dwordx4 %1, %5, %8 offset:1024 sc0 sc1\n\t"
                 "s_waitcnt vmcnt(0)"
                 : "=&v"(pk.l0), "=&v"(pk.l1), "=&v"(pk.c0), "=&v"(pk.c1), "=&v"(h)
                 : "v"(lane * 16), "v"(lane * 8), "v"(4096 + (lane & 31) * 4), "s"(src) : "memory");
    u32x4 fr[4];
    if (__builtin_amdgcn_readfirstlane(h) & 0x80u) {
      asm volatile("global_load_dwordx4 %0, %4, %5 sc0 sc1\n\t"
                   "global_load_dwordx4 %1, %4, %5 offset:1024 sc0 sc1\n\t"
                   "global_load_dwordx4 %2, %4, %5 offset:2048 sc0 sc1\n\t"
                   "global_load_dwordx4 %3, %4, %5 offset:3072 sc0 sc1\n\t"
                   "s_waitcnt vmcnt(0)"
                   : "=&v"(fr[0]), "=&v"(fr[1]), "=&v"(fr[2]), "=&v"(fr[3]) : "v"(lane * 16), "s"(src) : "memory");
    } else {
      pk_decode(pk, h, fr, lane, scratch);
    }
    char* dst = myws + OFF_STAGE + (size_t)(call & 1u) * FPAR + stage_off(pc, cs);
    store4(dst + u * 4096, lane * 16, fr);
    asm volatile("s_waitcnt vmcnt(0)" ::: "memory");
    if (lane == 0) st_agent(myws + OFF_LFLAG + (size_t)((pc.tile * 64 + cs) * 4 + u) * 4, call);
  }
}

__device__ __forceinline__ void expander(const Args& a, unsigned call, int ti, int part, int half, int lane, LAS char* buf) {
  char* myws = a.myws;
  const int slot = ti * 4 + part;
  const char* flag = myws + OFF_OFLAG + (size_t)slot * 4;
  bool bad = false;
  {
    const unsigned long long t0 = __builtin_amdgcn_s_memrealtime();
    for (;;) {
      const unsigned fv = ld_sys(flag), tm = ld_agent(myws + OFF_TMO);
      if (__builtin_amdgcn_readfirstlane(fv) >= call) break;
      if (__builtin_amdgcn_readfirstlane(tm) == call || __builtin_amdgcn_s_memrealtime() - t0 > DEADLINE) {
        if (lane == 0) st_agent(myws + OFF_TMO, call);
        bad = true;
        break;
      }
      __builtin_amdgcn_s_sleep(1);
    }
  }
  const char* src = myws + OFF_CSLOT + (size_t)slot * CSLOT + lane * 16;
  if (!bad) {
    __builtin_amdgcn_global_load_lds((const GAS void*)(src + 16384), (LAS void*)(buf + 16384), 16, 0, 17);
    if (half == 0) {
#pragma unroll
      for (int i = 0; i < 5; ++i)
        __builtin_amdgcn_global_load_lds((const GAS void*)(src + i * 1024), (LAS void*)(buf + i * 1024), 16, 0, 17);
    } else {
#pragma unroll
      for (int i = 3; i < 10; ++i)
        __builtin_amdgcn_global_load_lds((const GAS void*)(src + i * 1024), (LAS void*)(buf + i * 1024), 16, 0, 17);
    }
  }
  const unsigned tm = __builtin_amdgcn_readfirstlane(ld_agent(myws + OFF_TMO));
  asm volatile("s_waitcnt vmcnt(0)" ::: "memory");
  bad = bad || tm == call;
  float* o = a.myout + (size_t)((ti >> 4) * 128 + part * 32 + half * 16 + 4 * (lane >> 5)) * NCOL +
             ((a.rank >> 2) ^ 1) * 2048 + (ti & 15) * 128 + 4 * (lane & 31);
  if (bad) {
    for (int e = 0; e < 8; ++e)
      st16_sys(o + (size_t)((e & 3) + 8 * (e >> 2)) * NCOL, (u32x4){0x7fc00000u, 0x7fc00000u, 0x7fc00000u, 0x7fc00000u});
    return;
  }
  const u32x2 m = *(const LAS u32x2*)(buf + 16384 + lane * 8);
  const u32x2 cnt = *(const LAS u32x2*)(buf + 16896);
  const unsigned n0 = __builtin_amdgcn_readfirstlane(cnt[0]), total = __builtin_amdgcn_readfirstlane(cnt[1]);
  if (half == 0 ? n0 > 1280u : (n0 < 768u || total > 2560u)) {
#pragma unroll
    for (int i = 0; i < 16; ++i)
      __builtin_amdgcn_global_load_lds((const GAS void*)(src + i * 1024), (LAS void*)(buf + i * 1024), 16, 0, 17);
    asm volatile("s_waitcnt vmcnt(0)" ::: "memory");
  }
  unsigned base = half ? n0 : 0u;
  const unsigned bits = half ? m[0] : m[1];
  unsigned pos[32];
#pragma unroll
  for (int i = 0; i < 32; ++i) {
    const unsigned long long mm = __ballot((int)(bits << i) < 0);
    pos[i] = __builtin_amdgcn_mbcnt_hi((unsigned)(mm >> 32), __builtin_amdgcn_mbcnt_lo((unsigned)mm, base));
    base += (unsigned)__builtin_popcountll(mm);
  }
  float x[32];
#pragma unroll
  for (int i = 0; i < 32; ++i) x[i] = *(const LAS float*)(buf + (pos[i] & 4095u) * 4u);
#pragma unroll
  for (int e = 0; e < 8; ++e) {
    asm volatile("" : "+v"(x[4 * e]), "+v"(x[4 * e + 1]), "+v"(x[4 * e + 2]), "+v"(x[4 * e + 3]));
    f32x4 v;
#pragma unroll
    for (int c = 0; c < 4; ++c) v[c] = (int)(bits << (4 * e + c)) < 0 ? x[4 * e + c] : 0.f;
    st16_sys(o + (size_t)((e & 3) + 8 * (e >> 2)) * NCOL, __builtin_bit_cast(u32x4, v));
  }
}

__global__ void __launch_bounds__(THREADS) gemm_rs_kernel(Args a) {
  __shared__ __attribute__((aligned(16))) char lds[LDS_CTRL + 512];
  unsigned long long pc0;
  asm volatile("s_getpc_b64 %0" : "=s"(pc0));
  const int tid = threadIdx.x, lane = tid & 63;
  const int wv = __builtin_amdgcn_readfirstlane(tid >> 6);
  const int b = blockIdx.x, xcd = b & 7, jb = b >> 3;
  char* myws = a.myws;
  asm volatile("" :: "s"(a.myws), "s"(a.x), "s"(a.w), "s"(a.rank), "s"(a.ws[xcd]));
  unsigned call = 0;

  if (__builtin_expect(jb < 8, 1)) {
    LAS unsigned* sync = (LAS unsigned*)((LAS char*)lds + LDS_CTRL);
    LAS unsigned* seq = (LAS unsigned*)((LAS char*)lds + 4 * CSLOT);
    LAS unsigned* cnt = seq + 256;
    const bool streams = jb < NSLOT && xcd != a.rank;
    if (__builtin_expect(streams && wv < 4, 1)) {
      call = producer(a, xcd, jb, wv, lane, (LAS char*)lds + wv * PK_LDS, cnt, seq, sync + 8);
    } else {
      auto code_to_l2 = [&]() {
        const char* pc = (const char*)(pc0 & ~127ull);
#pragma unroll
        for (int i = 0; i < 4; ++i)
          __builtin_amdgcn_global_load_lds((const GAS void*)(pc + (i * 64 + lane) * 128), (LAS void*)(sync + 64), 4, 0, 0);
        if (lane < 8) __builtin_amdgcn_global_load_lds((const GAS void*)(pc + (256 + lane) * 128), (LAS void*)(sync + 64), 4, 0, 0);
      };
      if (wv == 7 && jb == 0) code_to_l2();
      call = __builtin_amdgcn_readfirstlane(ld_agent(myws + OFF_EPOCH)) + 1u;
      if (wv == 7 && lane < 16) sync[lane] = 0u;
      if (wv >= 4) { seq[(wv - 4) * 64 + lane] = 0xffffffffu; cnt[(wv - 4) * 64 + lane] = 0u; }
      asm volatile("s_waitcnt lgkmcnt(0)" ::: "memory");
      BAR();
      if (streams && wv == 4) queue_wave(a, call, xcd, jb, lane, seq, sync + 8);
    }
    BAR();
    expander(a, call, jb * 8 + xcd, wv >> 1, wv & 1, lane, (LAS char*)lds + (wv >> 1) * CSLOT);
  } else if (jb >= 24) {
    call = __builtin_amdgcn_readfirstlane(ld_agent(myws + OFF_EPOCH)) + 1u;
    decoder(a, call, (a.rank + 1 + (jb - 24)) & 7, xcd * 8 + wv, (jb - 24) * 64 + xcd * 8 + wv, lane, (LAS char*)lds + wv * PK_LDS);
    BAR();
  } else {
    const int jc = jb - 8;
    const int rt = (xcd & 3) + (jc >> 3) * 4;
    const int ct = (xcd >> 2) * 8 + (jc & 7);
    call = __builtin_amdgcn_readfirstlane(ld_agent(myws + OFF_EPOCH)) + 1u;
    LAS unsigned* lds_bad = (LAS unsigned*)((LAS char*)lds + LDS_CTRL);
    LAS unsigned* lds_sched = (LAS unsigned*)((LAS char*)lds + LDS_CTRL + 64);
    const char* stage = myws + OFF_STAGE + (size_t)(call & 1u) * FPAR;
    if (wv == 6) {
      const char* fa = myws + OFF_LFLAG + (size_t)(rt * 64 + lane) * 16;
      const char* fb = myws + OFF_LFLAG + (size_t)((8 + ct) * 64 + lane) * 16;
      const char* ptm = myws + OFF_TMO;
      unsigned long long ready = 0, done = 0;
      for (int c = 0; c < 64; ++c) {
        if ((ready & ~done) == 0ull) {
          const unsigned long long t0 = __builtin_amdgcn_s_memrealtime();
          for (;;) {
            u32x4 va, vb; unsigned tm;
            asm volatile("global_load_dwordx4 %0, %3, off sc0 sc1\n\t"
                         "global_load_dwordx4 %1, %4, off sc0 sc1\n\t"
                         "global_load_dword %2, %5, off sc1\n\t"
                         "s_waitcnt vmcnt(0)"
                         : "=&v"(va), "=&v"(vb), "=&v"(tm) : "v"(fa), "v"(fb), "v"(ptm) : "memory");
            const unsigned lo = min(min(min(va[0], va[1]), min(va[2], va[3])), min(min(vb[0], vb[1]), min(vb[2], vb[3])));
            ready = __ballot(lo >= call);
            if ((ready & ~done) != 0ull) break;
            if (__builtin_amdgcn_readfirstlane(tm) == call || __builtin_amdgcn_s_memrealtime() - t0 > DEADLINE) {
              if (lane == 0) st_agent(myws + OFF_TMO, call);
              ready = ~0ull;
              break;
            }
            __builtin_amdgcn_s_sleep(1);
          }
        }
        const int cs = __builtin_ctzll(ready & ~done);
        done |= 1ull << cs;
        if (lane == 0) lds_sched[c & 7] = (unsigned)cs;
        if (c == 63) {
          const unsigned t = __builtin_amdgcn_readfirstlane(ld_agent(myws + OFF_TMO));
          if (lane == 0) *lds_bad = (t == call) ? 1u : 0u;
        }
        asm volatile("s_waitcnt lgkmcnt(0)" ::: "memory");
        BAR();
      }
      for (int i = 0; i <= DIST; ++i) BAR();
      asm volatile("s_waitcnt vmcnt(0)" ::: "memory");
    } else if (wv == 7) {
    } else if (wv >= 4) {
      const int lw = wv - 4;
      const char* gbase = stage + A_BYTES + (size_t)ct * PIECE + lw * 8192 + lane * 16;
      BAR();
      int st = 0;
      for (int c = 0; c < 64 + DIST; ++c) {
        if (c < 64) {
          const int cs = __builtin_amdgcn_readfirstlane(*(volatile LAS unsigned*)(lds_sched + (c & 7))) & 63;
          const char* ga = gbase + (size_t)cs * (16 * PIECE);
          LAS char* la = (LAS char*)lds + st * 16384 + lw * 8192;
#pragma unroll
          for (int f = 0; f < 8; ++f)
            __builtin_amdgcn_global_load_lds((const GAS void*)(ga + f * 1024), (LAS void*)(la + f * 1024), 16, 0, 17);
          if constexpr (DIST == 3) asm volatile("s_waitcnt vmcnt(8)" ::: "memory");
          else if constexpr (DIST == 4) asm volatile("s_waitcnt vmcnt(16)" ::: "memory");
          else asm volatile("s_waitcnt vmcnt(24)" ::: "memory");
        } else if (63 - c + DIST - 2 == 2) {
          asm volatile("s_waitcnt vmcnt(16)" ::: "memory");
        } else if (63 - c + DIST - 2 == 1) {
          asm volatile("s_waitcnt vmcnt(8)" ::: "memory");
        } else {
          asm volatile("s_waitcnt vmcnt(0)" ::: "memory");
        }
        st = st == NSTAGE - 1 ? 0 : st + 1;
        BAR();
      }
    } else {
      f32x16 acc[4];
#pragma unroll
      for (int c = 0; c < 4; ++c)
#pragma unroll
        for (int e = 0; e < 16; ++e) acc[c][e] = 0.f;
      const __amdgpu_buffer_rsrc_t rsa = __builtin_amdgcn_make_buffer_rsrc((void*)(stage + (size_t)rt * PIECE), 0, (int)A_BYTES, 0x00020000);
      const unsigned aoff = wv * 4096 + lane * 16;
      h8 ar[NSTAGE][4];
      h8 bf[2][4];
#define SCHED_READ(C) (*(volatile LAS unsigned*)(lds_sched + ((C) & 7)))
#define LOAD_A(SLOT, SV) do {                                                                            \
        const unsigned cs_ = __builtin_amdgcn_readfirstlane(SV) & 63u;                                   \
        _Pragma("unroll") for (int s_ = 0; s_ < 4; ++s_)                                                 \
          ar[SLOT][s_] = __builtin_bit_cast(h8, __builtin_amdgcn_raw_buffer_load_b128(rsa, aoff + s_ * 1024, cs_ * (unsigned)(8 * PIECE), 17)); \
      } while (0)
#define READ_B(BUF, ST, S) do {                                                                          \
        _Pragma("unroll") for (int c_ = 0; c_ < 4; ++c_)                                                 \
          bf[BUF][c_] = *(const LAS h8*)((const LAS char*)lds + (ST) * 16384 + (S) * 4096 + c_ * 1024 + lane * 16); \
      } while (0)
#define MMA4(SLOT, S) do {                                                                               \
        __builtin_amdgcn_sched_barrier(0);                                                               \
        _Pragma("unroll") for (int c_ = 0; c_ < 4; ++c_)                                                 \
          acc[c_] = __builtin_amdgcn_mfma_f32_32x32x16_f16(ar[SLOT][S], bf[(S) & 1][c_], acc[c_], 0, 0, 0); \
        __builtin_amdgcn_sched_barrier(0);                                                               \
      } while (0)
#define STEP(K, T, MORE) do {                                                                            \
        READ_B(1, K, 1);                                                                                 \
        unsigned sv_ = 0; if (MORE) sv_ = SCHED_READ((T) + DIST);                                        \
        MMA4(K, 0);                                                                                      \
        READ_B(0, K, 2);                                                                                 \
        if (MORE) LOAD_A(((K) + DIST) % NSTAGE, sv_);                                                    \
        MMA4(K, 1);                                                                                      \
        READ_B(1, K, 3);                                                                                 \
        MMA4(K, 2);                                                                                      \
        READ_B(0, ((K) + 1) % NSTAGE, 0);                                                                \
        MMA4(K, 3);                                                                                      \
        asm volatile("s_waitcnt lgkmcnt(4)" ::: "memory");        \
        BAR();                                                                                           \
      } while (0)
      BAR(); LOAD_A(0, SCHED_READ(0)); BAR(); LOAD_A(1, SCHED_READ(1)); BAR(); LOAD_A(2, SCHED_READ(2));
      if constexpr (DIST >= 4) { BAR(); LOAD_A(3, SCHED_READ(3)); }
      if constexpr (DIST == 5) { BAR(); LOAD_A(4, SCHED_READ(4)); }
      BAR();
      READ_B(0, 0, 0);
      for (int t = 0; t < 54; t += 6) {
        STEP(0, t, true); STEP(1, t + 1, true); STEP(2, t + 2, true);
        STEP(3, t + 3, true); STEP(4, t + 4, true); STEP(5, t + 5, true);
      }
      STEP(0, 54, true); STEP(1, 55, true); STEP(2, 56, true); STEP(3, 57, true); STEP(4, 58, true);
      STEP(5, 59, 59 + DIST <= 63);
      STEP(0, 60, 60 + DIST <= 63); STEP(1, 61, false); STEP(2, 62, false); STEP(3, 63, false);
#undef SCHED_READ
#undef LOAD_A
#undef READ_B
#undef MMA4
#undef STEP
      const bool bad = *lds_bad != 0u;
      const int g = a.rank >> 2;
      if (rt < 4) {
        const float qnan = __builtin_nanf("");
        float* o = a.myout + (size_t)(rt * 128 + wv * 32 + 4 * (lane >> 5)) * NCOL + g * 2048 + ct * 128 + 4 * (lane & 31);
#pragma unroll
        for (int e = 0; e < 16; ++e) {
          f32x4 v;
#pragma unroll
          for (int c = 0; c < 4; ++c) { const float y = acc[c][e]; v[c] = bad ? qnan : (y < 0.f ? 0.f : y); }
          st16_sys(o + (size_t)((e & 3) + 8 * (e >> 2)) * NCOL, __builtin_bit_cast(u32x4, v));
        }
      } else {
        LAS char* buf = (LAS char*)lds + wv * CSLOT;
        const int q = ((g ^ 1) << 2) + (rt - 4);
        const int slot = ((a.rank & 3) * 16 + ct) * 4 + wv;
        char* dst = a.ws[q & 7] + OFF_CSLOT + (size_t)slot * CSLOT + lane * 16;
        if (bad) {
          const unsigned nb = 0x7fc00000u;
          for (int i = 0; i < 16; ++i) st16_sys(dst + i * 1024, (u32x4){nb, nb, nb, nb});
          if (lane < 32) st16_sys(dst + 16384, (u32x4){~0u, ~0u, ~0u, ~0u});
          if (lane == 32) st16_sys(dst + 16384, (u32x4){2048u, 4096u, 0u, 0u});
        } else {
          unsigned base = 0, sent = 0, n0 = 0, mhi = 0, mlo = 0;
#pragma unroll
          for (int e4 = 0; e4 < 4; ++e4) {
#pragma unroll
            for (int ee = 0; ee < 4; ++ee) {
              unsigned long long m[4];
              unsigned bs[4], pos[4];
#pragma unroll
              for (int c = 0; c < 4; ++c) m[c] = __ballot(!(acc[c][e4 * 4 + ee] <= 0.f));
#pragma unroll
              for (int c = 0; c < 4; ++c) { bs[c] = base; base += (unsigned)__builtin_popcountll(m[c]); }
#pragma unroll
              for (int c = 0; c < 4; ++c)
                pos[c] = __builtin_amdgcn_mbcnt_hi((unsigned)(m[c] >> 32), __builtin_amdgcn_mbcnt_lo((unsigned)m[c], 0u)) + bs[c];
              asm volatile("" : "+v"(pos[0]), "+v"(pos[1]), "+v"(pos[2]), "+v"(pos[3]));
#pragma unroll
              for (int c = 0; c < 4; ++c) {
                const float y = acc[c][e4 * 4 + ee];
                const bool nz = !(y <= 0.f);
                *(LAS float*)(buf + 4u * (nz ? pos[c] : 4288u + lane)) = y;
                if (e4 < 2) asm volatile("v_addc_co_u32_e64 %0, vcc, %0, %0, %1" : "+v"(mhi) : "s"(m[c]) : "vcc");
                else asm volatile("v_addc_co_u32_e64 %0, vcc, %0, %0, %1" : "+v"(mlo) : "s"(m[c]) : "vcc");
              }
            }
            if (e4 == 1) n0 = base;
            asm volatile("s_waitcnt vmcnt(0)" ::: "memory");
            for (; sent + 256u <= base; sent += 256u)
              st16_sys(dst + sent * 4u, *(const LAS u32x4*)(buf + sent * 4u + lane * 16));
          }
          if (sent + lane * 4u < base) st16_sys(dst + sent * 4u, *(const LAS u32x4*)(buf + sent * 4u + lane * 16));
          *(LAS u32x2*)(buf + 16384 + lane * 8) = (u32x2){mlo, mhi};
          if (lane == 0) *(LAS u32x2*)(buf + 16896) = (u32x2){n0, base};
          if (lane < 33) st16_sys(dst + 16384, *(const LAS u32x4*)(buf + 16384 + lane * 16));
        }
        asm volatile("s_waitcnt vmcnt(0)" ::: "memory");
        if (lane == 0) st_sys(a.ws[q & 7] + OFF_OFLAG + (size_t)slot * 4, call);
      }
    }
  }
  if (tid == 0) {
    const unsigned t = __hip_atomic_fetch_add((unsigned*)(myws + OFF_ENDCNT), 1u, __ATOMIC_RELAXED, __HIP_MEMORY_SCOPE_AGENT);
    if (t % GRID == GRID - 1) st_agent(myws + OFF_EPOCH, call);
  }
}

}

extern "C" void kernel_launch(int rank, int nranks, ncclComm_t comm,
                              hipStream_t stream,
                              void* const* d_in, const int* in_sizes, int n_in,
                              void* d_out, int out_size,
                              const int* in_dtypes, int out_dtype,
                              void* const* d_out_all,
                              void* const* d_ws_all, size_t ws_bytes) {
  (void)comm; (void)in_dtypes; (void)out_dtype; (void)d_out_all;
  if (nranks != NR || (unsigned)rank >= (unsigned)NR || n_in < 2 || ws_bytes < WS_NEED ||
      in_sizes[0] != 4096 * KSH || in_sizes[1] != KSH * NCOL || out_size != 512 * NCOL) {
    fprintf(stderr, "gemm_rs: unexpected problem shape\n");
    return;
  }
  Args a;
  for (int p = 0; p < NR; ++p) a.ws[p] = (char*)d_ws_all[p];
  a.myws = a.ws[rank];
  a.myout = (float*)d_out;
  a.x = (const float*)d_in[0];
  a.w = (const float*)d_in[1];
  a.rank = rank;
  for (int i = 0; i < 7; ++i) a.pad[i] = 0;
  gemm_rs_kernel<<<GRID, THREADS, 0, stream>>>(a);
  hipError_t e = hipGetLastError();
  if (e != hipSuccess) fprintf(stderr, "gemm_rs: launch failed: %s\n", hipGetErrorString(e));
}
```
